```python
import numpy as np

kernel_name = "gpu_ag_gemm_m4096_k4096_n4096_e4m3e4m3_dqrelu"


ABSMAX_RTOL = 0.03

COMPUTE_DTYPE = "float32"
EPILOGUE = "dequant_relu"
IN_QUANT = None
OUT_QUANT = None
IN_FP8 = "e4m3e4m3"
IN_DTYPES = {"x": "e4m3", "w_mat": "e4m3",
             "scale_x": "float32", "scale_w": "float32"}
WORLD_SIZE = 8

def _snap_fp8(x, mant_bits, max_abs, min_normal):
    a = np.abs(x.astype(np.float64))
    _m, _e = np.frexp(a)
    q = np.where(a >= min_normal, np.ldexp(1.0, _e - 1 - mant_bits),
                 min_normal * 2.0 ** -mant_bits)
    snapped = np.minimum(np.round(a / q) * q, max_abs)
    return (np.sign(x) * snapped).astype(np.float32)

def setup_inputs():

    x = _snap_fp8(np.random.randn(4096, 4096).astype(np.float32), 3, 448.0, 2.0 ** -6)
    w_mat = _snap_fp8(np.random.randn(4096, 4096).astype(np.float32), 3, 448.0, 2.0 ** -6)
    scale_x = (0.02 * (1.0 + np.random.rand(1))).astype(np.float32)
    scale_w = (0.02 * (1.0 + np.random.rand(1))).astype(np.float32)
    return {"x": x, "w_mat": w_mat, "scale_x": scale_x, "scale_w": scale_w}

def reference(x, w_mat, scale_x, scale_w):
    acc = (x @ w_mat).astype(np.float32)
    y = (acc * (scale_x * scale_w)).astype(np.float32)
    return np.maximum(y, 0.0).astype(np.float32)
```

```cpp
#include <hip/hip_runtime.h>
#include <rccl/rccl.h>
#include <cstdio>
#include <cstdint>

#define HIP_CHECK(expr) do { hipError_t _e = (expr); if (_e != hipSuccess) { \
    fprintf(stderr, "HIP error %s at %s:%d\n", hipGetErrorString(_e), __FILE__, __LINE__); \
    return; } } while (0)

#define LAS __attribute__((address_space(3)))
typedef unsigned u32x4 __attribute__((ext_vector_type(4)));
typedef int v8i __attribute__((ext_vector_type(8)));
typedef int v2i __attribute__((ext_vector_type(2)));
typedef float v4f __attribute__((ext_vector_type(4)));

namespace {

constexpr int NR = 8;
constexpr int KDIM = 4096;
constexpr int NLOC = 512;
constexpr int ROWS = 512;
constexpr unsigned SLOT_BYTES = ROWS * KDIM;
constexpr size_t DATA_PAR = (size_t)NR * SLOT_BYTES;
constexpr size_t FLAG_OFF = 2 * DATA_PAR;
constexpr size_t CTR_OFF = FLAG_OFF + 64 * 1024;
constexpr size_t TMO_OFF = CTR_OFF + 256 * 8 * 64;
constexpr size_t DBG_OFF = TMO_OFF + 4096;
constexpr size_t WS_NEED = DBG_OFF + 256 * 8 * 32 * 8;
constexpr unsigned long long DEADLINE = 30000000ull;

constexpr int NGRP = 16;
constexpr int STAGE = 256 * 128;
constexpr int LDS_CTL = 4 * STAGE;
constexpr int LDS_BYTES = LDS_CTL + 256;
constexpr int XST_OFF = 2 * STAGE;
constexpr int WIN = 2;
constexpr int NDIRECT = 2;

struct Params {
    const unsigned char* x;
    const unsigned char* w;
    const float* sx;
    const float* sw;
    float* out;
    unsigned char* ws[NR];
    int rank;
    int pad;
};

__device__ __forceinline__ unsigned long long rt() { return __builtin_amdgcn_s_memrealtime(); }
__device__ __forceinline__ unsigned uni(unsigned v) { return __builtin_amdgcn_readfirstlane(v); }
template <int N> __device__ __forceinline__ void waitvm() { asm volatile("s_waitcnt vmcnt(%0)" :: "i"(N) : "memory"); }
__device__ __forceinline__ void st16_sys(void* p, u32x4 v) {
    asm volatile("global_store_dwordx4 %0, %1, off sc0 sc1\n\ts_nop 1" :: "v"(p), "v"(v) : "memory");
}
__device__ __forceinline__ void glds16(const void* gsrc, unsigned lds_dst) {
    unsigned keep;
    asm volatile("s_mov_b32 %0, m0\n\ts_mov_b32 m0, %2\n\ts_nop 0\n\tglobal_load_lds_dwordx4 %1, off\n\ts_mov_b32 m0, %0"
                 : "=&s"(keep) : "v"(gsrc), "s"(lds_dst) : "memory");
}
__device__ __forceinline__ void lds_st(unsigned addr, unsigned v) {
    asm volatile("ds_write_b32 %0, %1" :: "v"(addr), "v"(v) : "memory");
}
__device__ __forceinline__ unsigned lds_ld(unsigned addr) {
    unsigned v;
    asm volatile("ds_read_b32 %0, %1\n\ts_waitcnt lgkmcnt(0)" : "=v"(v) : "v"(addr) : "memory");
    return uni(v);
}
__device__ __forceinline__ u32x4 lds_ld16(unsigned addr) {
    u32x4 v;
    asm volatile("ds_read_b128 %0, %1\n\ts_waitcnt lgkmcnt(0)" : "=v"(v) : "v"(addr) : "memory");
    return v;
}
__device__ __forceinline__ unsigned ld_sys(const unsigned* p) { return __hip_atomic_load(p, __ATOMIC_RELAXED, __HIP_MEMORY_SCOPE_SYSTEM); }
__device__ __forceinline__ unsigned ld_agent(const unsigned* p) { return __hip_atomic_load(p, __ATOMIC_RELAXED, __HIP_MEMORY_SCOPE_AGENT); }
__device__ __forceinline__ void st_agent(unsigned* p, unsigned v) { __hip_atomic_store(p, v, __ATOMIC_RELAXED, __HIP_MEMORY_SCOPE_AGENT); }

__device__ __forceinline__ bool wait_flag(const unsigned* flag, unsigned want, unsigned* tmo, unsigned call) {
    if (uni(ld_sys(flag)) >= want) return true;
    const unsigned long long t0 = rt();
    for (unsigned spins = 1;; ++spins) {
        __builtin_amdgcn_s_sleep(1);
        if (uni(ld_sys(flag)) >= want) return true;
        if ((spins & 15u) == 0u) {
            if (uni(ld_agent(tmo)) == call || rt() - t0 > DEADLINE) { st_agent(tmo, call); return false; }
        }
    }
}
__device__ __forceinline__ bool wait_lds(unsigned addr, unsigned want, unsigned* tmo, unsigned call) {
    if (lds_ld(addr) >= want) return true;
    const unsigned long long t0 = rt();
    for (unsigned spins = 1;; ++spins) {
        __builtin_amdgcn_s_sleep(1);
        if (lds_ld(addr) >= want) return true;
        if ((spins & 255u) == 0u) {
            if (uni(ld_agent(tmo)) == call || rt() - t0 > DEADLINE) { st_agent(tmo, call); return false; }
        }
    }
}

#define STAMP(i) do { } while (0)

__global__ void __launch_bounds__(384, 2) ag_gemm(Params p) {
    extern __shared__ __attribute__((aligned(16))) unsigned char smem[];
    const int lane = threadIdx.x & 63;
    const int wave = uni(threadIdx.x >> 6);
    const int b = blockIdx.x;
    const int rt_ = b & 7;
    const int ct = (b >> 3) & 3;
    const int srcsel = b >> 5;
    const int src = (p.rank + srcsel) & (NR - 1);
    const bool local = srcsel == 0;
    const unsigned lds0 = (unsigned)(size_t)(LAS unsigned char*)smem;
    const unsigned ctl = lds0 + LDS_CTL;
    unsigned char* me = p.ws[p.rank];
    unsigned* tmo = (unsigned*)(me + TMO_OFF);
    STAMP(0);

    unsigned* ctr = (unsigned*)(me + CTR_OFF + (b * 8 + wave) * 64);
    const unsigned call_v = ld_agent(ctr) + 1u;

    const int g = lane >> 4, r = lane & 15;

    if (wave == 5) {
        if (!local) {
            if (lane == 0) st_agent(ctr, call_v);
            return;
        }
        const int rb = rt_ * 4 + ct;
        const unsigned char* xs = p.x + (size_t)(rb * 16 + r) * KDIM + 16 * g;
        u32x4 d0[NDIRECT * 4];
#pragma unroll
        for (int i = 0; i < NDIRECT * 4; ++i) d0[i] = *(const u32x4*)(xs + (i >> 1) * 128 + (i & 1) * 64);
        __builtin_amdgcn_s_barrier();
#pragma unroll
        for (int i = 0; i < NDIRECT * 4; ++i) asm volatile("" : "+v"(d0[i]));
        const unsigned call = uni(call_v);
        if (lane == 0) st_agent(ctr, call);
        STAMP(1);
        const size_t doff = (size_t)(call & 1u) * DATA_PAR + (size_t)p.rank * SLOT_BYTES + (size_t)rb * 65536 + lane * 16;
        unsigned char* dst[NR - 1];
#pragma unroll
        for (int d = 0; d < NR - 1; ++d) {
            int q = d + (rb % 7); if (q >= 7) q -= 7;
            dst[d] = p.ws[(p.rank + 1 + q) & (NR - 1)] + doff;
        }
        unsigned* fl = (unsigned*)(p.ws[(p.rank + 1 + (lane < 7 ? lane : 0)) & (NR - 1)] + FLAG_OFF + (size_t)(p.rank * 32 + rb) * 128);
        const unsigned fbase = call * 64u;
#pragma unroll
        for (int grp = 0; grp < NDIRECT; ++grp) {
#pragma unroll
            for (int d = 0; d < NR - 1; ++d)
#pragma unroll
                for (int i = 0; i < 4; ++i) st16_sys(dst[d] + grp * 4096 + i * 1024, d0[grp * 4 + i]);
        }
        for (int w = 0; w < 4; ++w)
            if (!wait_lds(ctl + 20 + 4 * w, 1u, tmo, call)) return;
        for (int grp = NDIRECT; grp < NGRP; ++grp) {
            u32x4 v[4];
#pragma unroll
            for (int i = 0; i < 4; ++i) v[i] = lds_ld16(lds0 + XST_OFF + grp * 4096 + i * 1024 + lane * 16);
            waitvm<(WIN - 1) * 28>();
            if (lane < 7) __hip_atomic_store(fl, fbase + (unsigned)(grp - WIN + 1), __ATOMIC_RELAXED, __HIP_MEMORY_SCOPE_SYSTEM);
#pragma unroll
            for (int d = 0; d < NR - 1; ++d)
#pragma unroll
                for (int i = 0; i < 4; ++i) st16_sys(dst[d] + grp * 4096 + i * 1024, v[i]);
        }
        waitvm<28>();
        if (lane < 7) __hip_atomic_store(fl, fbase + (unsigned)(NGRP - 1), __ATOMIC_RELAXED, __HIP_MEMORY_SCOPE_SYSTEM);
        waitvm<1>();
        if (lane < 7) __hip_atomic_store(fl, fbase + (unsigned)NGRP, __ATOMIC_RELAXED, __HIP_MEMORY_SCOPE_SYSTEM);
        STAMP(2);
        return;
    }

    const int NS = local ? 2 : 4;

    if (wave == 4) {
        const unsigned char* wsrc = p.w + (size_t)(lane >> 3) * NLOC + ct * 128;
        const unsigned c0 = (unsigned)(((lane & 7) ^ ((lane >> 4) & 3)) << 4);
        const unsigned c1 = c0 ^ 64u;
        if (lane < 16) lds_st(ctl + lane * 4, 0u);
        asm volatile("s_waitcnt lgkmcnt(0)" ::: "memory");
#pragma unroll 4
        for (int i = 0; i < 32; ++i) glds16(wsrc + (size_t)i * 8 * NLOC + (((i >> 1) & 1) ? c1 : c0), lds0 + i * 1024);
        __builtin_amdgcn_s_barrier();
        const unsigned call = uni(call_v);
        if (lane == 0) st_agent(ctr, call);
        bool ok = true;
        for (int st = 1; st < NGRP; ++st) {
            if (st >= NS) {
                const unsigned need = (unsigned)(st - NS + 1);
                bool fr = true;
                for (int w = 0; w < 4; ++w) fr = fr && lds_ld(ctl + 4 + 4 * w) >= need;
                if (!fr) {
                    waitvm<0>();
                    lds_st(ctl, (unsigned)st);
                    for (int w = 0; w < 4; ++w) ok = ok && wait_lds(ctl + 4 + 4 * w, need, tmo, call);
                }
            }
            const unsigned slot = lds0 + (unsigned)(st & (NS - 1)) * STAGE;
            const unsigned char* ws_ = wsrc + (size_t)st * 256 * NLOC;
#pragma unroll 4
            for (int i = 0; i < 32; ++i) glds16(ws_ + (size_t)i * 8 * NLOC + (((i >> 1) & 1) ? c1 : c0), slot + i * 1024);
            waitvm<32>();
            lds_st(ctl, (unsigned)st);
        }
        waitvm<0>();
        lds_st(ctl, (unsigned)NGRP);
        asm volatile("s_waitcnt lgkmcnt(0)" ::: "memory");
        STAMP(2);
        return;
    }

    const int rb = rt_ * 4 + wave;
    if (local) {
        const unsigned char* xs = p.x + (size_t)((rt_ * 4 + ct) * 16 + r) * KDIM + 16 * g;
        __builtin_amdgcn_s_barrier();
        for (int grp = NDIRECT + wave; grp < NGRP; grp += 4) {
#pragma unroll
            for (int i = 0; i < 4; ++i) glds16(xs + grp * 256 + (i >> 1) * 128 + (i & 1) * 64, lds0 + XST_OFF + grp * 4096 + i * 1024);
        }
        waitvm<0>();
        lds_st(ctl + 20 + 4 * wave, 1u);
    } else {
        __builtin_amdgcn_s_barrier();
    }
    const unsigned call = uni(call_v);
    if (lane == 0) st_agent(ctr, call);
    STAMP(1);

    const unsigned char* xbase = local ? p.x : me + (size_t)(call & 1u) * DATA_PAR + (size_t)src * SLOT_BYTES;
    const __amdgpu_buffer_rsrc_t xr = __builtin_amdgcn_make_buffer_rsrc((void*)xbase, 0, (int)SLOT_BYTES, 0x00020000);
    const int voff = local ? (rb * 16 + r) * KDIM + 16 * g : rb * 65536 + lane * 16;
    const int kstride = local ? 128 : 2048, hstride = local ? 64 : 1024;
    const unsigned* flag = (const unsigned*)(me + FLAG_OFF + (size_t)(src * 32 + rb) * 128);
    const unsigned fbase = call * 64u;

    const unsigned q = (unsigned)(lane & 15) >> 1, pp = lane & 1;
    const unsigned swz = ((q >> 1) & 3u) | ((unsigned)(g & 1) << 2);
    const unsigned abase = (16u * g + q) * 128u + 8u * pp + (swz << 4);

    v4f acc[8];
#pragma unroll
    for (int c = 0; c < 8; ++c) acc[c] = v4f{0.f, 0.f, 0.f, 0.f};

    bool ok = true;
    u32x4 xa[4], xb[4];
    auto load_x = [&](u32x4* dstv, int kg) {
#pragma unroll
        for (int i = 0; i < 4; ++i)
            dstv[i] = __builtin_amdgcn_raw_buffer_load_b128(xr, voff, (2 * kg + (i >> 1)) * kstride + (i & 1) * hstride, 17);
    };
    auto compute = [&](const u32x4* xv, int kg) {
        ok = ok && wait_lds(ctl, (unsigned)(kg + 1), tmo, call);
        const unsigned sb = lds0 + (unsigned)(kg & (NS - 1)) * STAGE + abase;
#pragma unroll
        for (int j = 0; j < 2; ++j) {
            v8i xf;
            xf[0] = xv[2 * j][0]; xf[1] = xv[2 * j][1]; xf[2] = xv[2 * j][2]; xf[3] = xv[2 * j][3];
            xf[4] = xv[2 * j + 1][0]; xf[5] = xv[2 * j + 1][1]; xf[6] = xv[2 * j + 1][2]; xf[7] = xv[2 * j + 1][3];
#pragma unroll
            for (int c = 0; c < 8; ++c) {
                LAS unsigned char* a = (LAS unsigned char*)(size_t)((sb ^ (c << 4)) + j * 16384);
                v2i w0 = __builtin_amdgcn_ds_read_tr8_b64_v2i32((LAS v2i*)(a));
                v2i w1 = __builtin_amdgcn_ds_read_tr8_b64_v2i32((LAS v2i*)(a + 1024));
                v2i w2 = __builtin_amdgcn_ds_read_tr8_b64_v2i32((LAS v2i*)(a + 8192));
                v2i w3 = __builtin_amdgcn_ds_read_tr8_b64_v2i32((LAS v2i*)(a + 8192 + 1024));
                v8i wf = {w0[0], w0[1], w1[0], w1[1], w2[0], w2[1], w3[0], w3[1]};
                acc[c] = __builtin_amdgcn_mfma_scale_f32_16x16x128_f8f6f4(wf, xf, acc[c], 0, 0, 0, 0, 0, 0);
            }
        }
        asm volatile("s_waitcnt lgkmcnt(0)" ::: "memory");
        lds_st(ctl + 4 + 4 * wave, (unsigned)(kg + 1));
    };

    for (int kg = 0; kg < NGRP; kg += 2) {
        if (!local) ok = ok && wait_flag(flag, fbase + kg + 1, tmo, call);
        asm volatile("" ::: "memory");
        if (ok) load_x(xa, kg);
        if (kg > 0) compute(xb, kg - 1);
        if (!local) ok = ok && wait_flag(flag, fbase + kg + 2, tmo, call);
        asm volatile("" ::: "memory");
        if (ok) load_x(xb, kg + 1);
        compute(xa, kg);
    }
    STAMP(2);
    compute(xb, NGRP - 1);
    STAMP(3);

    const float scale = p.sx[0] * p.sw[0];
    const bool bad = uni(ld_agent(tmo)) == call;
    float* o = p.out + (size_t)(src * ROWS + rb * 16 + r) * NLOC + ct * 128 + 4 * g;
#pragma unroll
    for (int c = 0; c < 8; ++c) {
        v4f y = acc[c] * scale;
#pragma unroll
        for (int j = 0; j < 4; ++j) y[j] = y[j] < 0.f ? 0.f : y[j];
        if (bad) y = v4f{__builtin_nanf(""), __builtin_nanf(""), __builtin_nanf(""), __builtin_nanf("")};
        *(v4f*)(o + 16 * c) = y;
    }
    STAMP(4);
}

}

extern "C" void kernel_launch(int rank, int nranks, ncclComm_t comm, hipStream_t stream,
                              void* const* d_in, const int* in_sizes, int n_in,
                              void* d_out, int out_size,
                              const int* in_dtypes, int out_dtype,
                              void* const* d_out_all,
                              void* const* d_ws_all, size_t ws_bytes) {
    (void)comm; (void)in_dtypes; (void)out_dtype; (void)d_out_all;
    if (nranks != NR || (unsigned)rank >= (unsigned)NR || n_in != 4 || ws_bytes < WS_NEED ||
        in_sizes[0] != ROWS * KDIM || in_sizes[1] != KDIM * NLOC || out_size != NR * ROWS * NLOC) {
        fprintf(stderr, "kernel_launch: unexpected problem shape\n");
        return;
    }
    static bool init = false;
    if (!init) {
        HIP_CHECK(hipFuncSetAttribute((const void*)ag_gemm, hipFuncAttributeMaxDynamicSharedMemorySize, LDS_BYTES));
        init = true;
    }
    Params p;
    p.x = (const unsigned char*)d_in[0];
    p.w = (const unsigned char*)d_in[1];
    p.sx = (const float*)d_in[2];
    p.sw = (const float*)d_in[3];
    p.out = (float*)d_out;
    for (int i = 0; i < NR; ++i) p.ws[i] = (unsigned char*)d_ws_all[i];
    p.rank = rank;
    p.pad = 0;
    hipLaunchKernelGGL(ag_gemm, dim3(256), dim3(384), LDS_BYTES, stream, p);
    HIP_CHECK(hipGetLastError());
}
```

```cpp
#include <hip/hip_runtime.h>
#include <rccl/rccl.h>
#include <cstdio>
#include <cstdint>

#define HIP_CHECK(expr) do { hipError_t _e = (expr); if (_e != hipSuccess) { \
    fprintf(stderr, "HIP error %s at %s:%d\n", hipGetErrorString(_e), __FILE__, __LINE__); \
    return; } } while (0)

#define PACK 1
#define LSLEEP 12
#define PACE 3750
#define PACEOFF 0
#define FSLEEP 4
#define FASTN 1
#define POLLSLEEP 1
#define LAS __attribute__((address_space(3)))
typedef unsigned u32x4 __attribute__((ext_vector_type(4)));
typedef unsigned long long u64;
typedef int v8i __attribute__((ext_vector_type(8)));
typedef int v2i __attribute__((ext_vector_type(2)));
typedef float v4f __attribute__((ext_vector_type(4)));

namespace {

constexpr int NR = 8;
constexpr int KDIM = 4096;
constexpr int NLOC = 512;
constexpr int ROWS = 512;
constexpr unsigned SLOT_BYTES = ROWS * KDIM;
constexpr size_t DATA_PAR = (size_t)NR * SLOT_BYTES;
constexpr size_t FLAG_OFF = 2 * DATA_PAR;
constexpr size_t CTR_OFF = FLAG_OFF + 64 * 1024;
constexpr size_t TMO_OFF = CTR_OFF + 256 * 8 * 64;
constexpr size_t DBG_OFF = TMO_OFF + 4096;
constexpr size_t WS_NEED = DBG_OFF + 256 * 8 * 32 * 8;
constexpr u64 DEADLINE = 30000000ull;

constexpr int NGRP = 16;
constexpr int NS = 4;
constexpr int STAGE = 256 * 128;
constexpr int LDS_CTL = NS * STAGE;
constexpr int LDS_SCR = LDS_CTL + 256;
constexpr int SCR_BYTES = 1024 + 64;
constexpr int LDS_BYTES = LDS_SCR + 5 * SCR_BYTES;
#define WNUM 24
constexpr int WN = WNUM;

struct Params {
    const unsigned char* x;
    const unsigned char* w;
    const float* sx;
    const float* sw;
    float* out;
    unsigned char* me;
    unsigned char* peer[NR - 1];
    int rank;
    int pad;
};

__device__ __forceinline__ u64 rt() { return __builtin_amdgcn_s_memrealtime(); }
__device__ __forceinline__ unsigned uni(unsigned v) { return __builtin_amdgcn_readfirstlane(v); }
template <int N> __device__ __forceinline__ void waitvm() { asm volatile("s_waitcnt vmcnt(%0)" :: "i"(N) : "memory"); }
__device__ __forceinline__ void st16_sys(void* p, u32x4 v) {
    asm volatile("global_store_dwordx4 %0, %1, off sc0 sc1\n\ts_nop 1" :: "v"(p), "v"(v) : "memory");
}
__device__ __forceinline__ void st16_sys_masked(void* p, u32x4 v, u64 mask) {
    u64 keep;
    asm volatile("s_mov_b64 %0, exec\n\ts_mov_b64 exec, %3\n\tglobal_store_dwordx4 %1, %2, off sc0 sc1\n\ts_mov_b64 exec, %0\n\ts_nop 1"
                 : "=&s"(keep) : "v"(p), "v"(v), "s"(mask) : "memory");
}
__device__ __forceinline__ void glds16(const void* gsrc, unsigned lds_dst) {
    unsigned keep;
    asm volatile("s_mov_b32 %0, m0\n\ts_mov_b32 m0, %2\n\ts_nop 0\n\tglobal_load_lds_dwordx4 %1, off\n\ts_mov_b32 m0, %0"
                 : "=&s"(keep) : "v"(gsrc), "s"(lds_dst) : "memory");
}
__device__ __forceinline__ void lds_st(unsigned addr, unsigned v) {
    asm volatile("ds_write_b32 %0, %1" :: "v"(addr), "v"(v) : "memory");
}
__device__ __forceinline__ void lds_st16(unsigned addr, u32x4 v) {
    asm volatile("ds_write_b128 %0, %1" :: "v"(addr), "v"(v) : "memory");
}
__device__ __forceinline__ void lds_or(unsigned addr, unsigned v) {
    asm volatile("ds_or_b32 %0, %1" :: "v"(addr), "v"(v) : "memory");
}
__device__ __forceinline__ unsigned lds_ldv(unsigned addr) {
    unsigned v;
    asm volatile("ds_read_b32 %0, %1\n\ts_waitcnt lgkmcnt(0)" : "=v"(v) : "v"(addr) : "memory");
    return v;
}
__device__ __forceinline__ unsigned lds_ld(unsigned addr) { return uni(lds_ldv(addr)); }
__device__ __forceinline__ void lds_ld16x4(unsigned addr, u32x4* v) {
    asm volatile("ds_read_b128 %0, %4\n\tds_read_b128 %1, %4 offset:1024\n\tds_read_b128 %2, %4 offset:2048\n\t"
                 "ds_read_b128 %3, %4 offset:3072\n\ts_waitcnt lgkmcnt(0)"
                 : "=&v"(v[0]), "=&v"(v[1]), "=&v"(v[2]), "=&v"(v[3]) : "v"(addr) : "memory");
}
__device__ __forceinline__ unsigned ld_agent(const unsigned* p) { return __hip_atomic_load(p, __ATOMIC_RELAXED, __HIP_MEMORY_SCOPE_AGENT); }
__device__ __forceinline__ void st_agent(unsigned* p, unsigned v) { __hip_atomic_store(p, v, __ATOMIC_RELAXED, __HIP_MEMORY_SCOPE_AGENT); }
__device__ __forceinline__ u64 ld_sys64(const u64* p) { return __hip_atomic_load(p, __ATOMIC_RELAXED, __HIP_MEMORY_SCOPE_SYSTEM); }
__device__ __forceinline__ void st_sys64(u64* p, u64 v) { __hip_atomic_store(p, v, __ATOMIC_RELAXED, __HIP_MEMORY_SCOPE_SYSTEM); }

__device__ __forceinline__ bool wait_flag(const u64* flag, unsigned want, unsigned& lo, unsigned& hi, unsigned* tmo, unsigned call) {
    u64 v = ld_sys64(flag);
    lo = uni((unsigned)v); hi = uni((unsigned)(v >> 32));
    if (lo >= want) return true;
    const u64 t0 = rt();
    for (unsigned spins = 1;; ++spins) {
        __builtin_amdgcn_s_sleep(POLLSLEEP);
        v = ld_sys64(flag);
        lo = uni((unsigned)v); hi = uni((unsigned)(v >> 32));
        if (lo >= want) return true;
        if ((spins & 15u) == 0u) {
            if (uni(ld_agent(tmo)) == call || rt() - t0 > DEADLINE) { st_agent(tmo, call); return false; }
        }
    }
}
__device__ __forceinline__ bool wait_flag_fast(const u64* flag, unsigned want, unsigned& lo, unsigned& hi, unsigned* tmo, unsigned call) {
    u64 v0 = ld_sys64(flag);
    __builtin_amdgcn_s_sleep(FSLEEP);
    u64 v1 = ld_sys64(flag);
    __builtin_amdgcn_s_sleep(FSLEEP);
    u64 v2 = ld_sys64(flag);
    __builtin_amdgcn_s_sleep(FSLEEP);
    u64 v3 = ld_sys64(flag);
    const u64 t0 = rt();
    for (unsigned spins = 1;; ++spins) {
#define POLL1(v) lo = uni((unsigned)v); hi = uni((unsigned)(v >> 32)); if (lo >= want) return true; v = ld_sys64(flag); __builtin_amdgcn_s_sleep(FSLEEP);
        POLL1(v0) POLL1(v1) POLL1(v2) POLL1(v3)
#undef POLL1
        if ((spins & 15u) == 0u) {
            if (uni(ld_agent(tmo)) == call || rt() - t0 > DEADLINE) { st_agent(tmo, call); return false; }
        }
    }
}
template <int SLEEP = 1>
__device__ __forceinline__ bool wait_lds(unsigned addr, unsigned want, unsigned& got, unsigned* tmo, unsigned call) {
    got = lds_ld(addr);
    if (got >= want) return true;
    const u64 t0 = rt();
    for (unsigned spins = 1;; ++spins) {
        __builtin_amdgcn_s_sleep(SLEEP);
        got = lds_ld(addr);
        if (got >= want) return true;
        if ((spins & 255u) == 0u) {
            if (uni(ld_agent(tmo)) == call || rt() - t0 > DEADLINE) { st_agent(tmo, call); return false; }
        }
    }
}

__device__ __forceinline__ unsigned perm(unsigned hi, unsigned lo, unsigned sel) { return __builtin_amdgcn_perm(hi, lo, sel); }
__device__ __forceinline__ unsigned bfi(unsigned m, unsigned a, unsigned b) { return (m & a) | (~m & b); }
__device__ __forceinline__ unsigned wave_scan(unsigned v) {
    v += __builtin_amdgcn_update_dpp(0u, v, 0x111, 0xf, 0xf, true);
    v += __builtin_amdgcn_update_dpp(0u, v, 0x112, 0xf, 0xf, true);
    v += __builtin_amdgcn_update_dpp(0u, v, 0x114, 0xf, 0xf, true);
    v += __builtin_amdgcn_update_dpp(0u, v, 0x118, 0xf, 0xf, true);
    v += __builtin_amdgcn_update_dpp(0u, v, 0x142, 0xa, 0xf, false);
    v += __builtin_amdgcn_update_dpp(0u, v, 0x143, 0xc, 0xf, false);
    return v;
}

__device__ __forceinline__ unsigned pack_group(const unsigned* R, unsigned slot, unsigned lut, int lane, bool force_raw) {
    unsigned nl = 0;
    if (!force_raw) {
        unsigned s = 0;
#pragma unroll
        for (int i = 0; i < 16; i += 5) s = __builtin_amdgcn_udot4((R[i] >> 3) & 0x0F0F0F0Fu, 0x01010101u, s, false);
        const unsigned tot = (unsigned)__builtin_amdgcn_readlane((int)wave_scan(s), 63);
        const unsigned a = tot > 102u ? (tot - 102u) >> 10 : 0u;
        const unsigned K = ((5u - a) & 15u) * 0x01010101u;
        unsigned bad = 0, N[8], P[4], n[2];
        u64 acc[2];
#pragma unroll
        for (int j = 0; j < 2; ++j) {
            u64 ac = 0;
            unsigned pos = 0;
#pragma unroll
            for (int q = 0; q < 2; ++q) {
                unsigned pw = 0;
#pragma unroll
                for (int i = 0; i < 4; ++i) {
                    const unsigned r = R[8 * j + 4 * q + i];
                    const unsigned D = (((r >> 3) & 0x0F0F0F0Fu) + K) & 0x0F0F0F0Fu;
                    bad |= (D + 0x05050505u) & 0x10101010u;
                    const unsigned C = perm(0x820180FFu, 0xFFFFFFFFu, D) & 0x03030303u;
                    const unsigned f = perm(0u, 0x01000000u, C);
                    const unsigned t = (D >> 3) & 0x01010101u;
                    const unsigned S = D - (t + (t << 1));
                    const unsigned m = __builtin_amdgcn_udot4(f, 0x08040201u, 0u, false);
                    const unsigned sel = lds_ldv(lut + m * 4);
                    const unsigned c4 = perm(0u, S, sel);
                    const unsigned w = __builtin_amdgcn_udot4(c4, 0x00400801u, (c4 >> 24) << 9, false);
                    ac |= (u64)w << (pos & 63u);
                    pos += 3u * (unsigned)__builtin_popcount(f);
                    pw |= C << (2 * i);
                }
                P[2 * j + q] = pw;
            }
#pragma unroll
            for (int m2 = 0; m2 < 4; ++m2) {
                const unsigned ra = R[8 * j + 2 * m2], rb = R[8 * j + 2 * m2 + 1];
                N[4 * j + m2] = bfi(0x0F0F0F0Fu, bfi(0x08080808u, ra >> 4, ra), bfi(0x70707070u, rb << 4, rb));
            }
            acc[j] = ac; n[j] = pos;
        }
        const unsigned cnt = n[0] | (n[1] << 16);
        const unsigned inc = wave_scan(cnt);
        const unsigned tot2 = (unsigned)__builtin_amdgcn_readlane((int)inc, 63);
        const unsigned t0 = tot2 & 0xffffu, t1 = tot2 >> 16;
        const bool lane_bad = bad != 0u || n[0] > 63u || n[1] > 63u;
        const unsigned bits = 32u + t0 + t1;
        if (!__any(lane_bad) && bits <= 8192u) {
            const unsigned exc = inc - cnt;
            lds_st16(slot + lane * 16, u32x4{N[0], N[1], N[2], N[3]});
            lds_st16(slot + 1024 + lane * 16, u32x4{N[4], N[5], N[6], N[7]});
            lds_st16(slot + 2048 + lane * 16, u32x4{P[0], P[1], P[2], P[3]});
            lds_st16(slot + 3072 + lane * 16, u32x4{lane == 0 ? a : 0u, 0u, 0u, 0u});
#pragma unroll
            for (int j = 0; j < 2; ++j) {
                const unsigned off = 32u + (j ? t0 + (exc >> 16) : (exc & 0xffffu));
                const unsigned sh = off & 31u, ad = slot + 3072 + (off >> 5) * 4;
                const u64 lo = acc[j] << sh;
                const unsigned hi = (unsigned)((acc[j] >> 32) >> (32u - sh));
                lds_or(ad, (unsigned)lo);
                lds_or(ad + 4, (unsigned)(lo >> 32));
                if ((off >> 5) + 2 < 256u) lds_or(ad + 8, hi);
            }
            nl = ((bits + 511u) >> 9) * 4u;
        }
    }
    if (nl == 0) {
#pragma unroll
        for (int i = 0; i < 4; ++i) lds_st16(slot + i * 1024 + lane * 16, u32x4{R[4 * i], R[4 * i + 1], R[4 * i + 2], R[4 * i + 3]});
    }
    return nl;
}

__device__ __forceinline__ unsigned opaque(unsigned k) { asm("" : "+s"(k)); return k; }
__device__ __forceinline__ unsigned sel3(unsigned m, unsigned a, unsigned b) { return __builtin_amdgcn_bitop3_b32(m, a, b, 0xCA); }
__device__ __forceinline__ void unpack_group(const u32x4* v, unsigned scr, int lane, v8i& x0, v8i& x1) {
    lds_st16(scr + lane * 16, v[3]);
    const unsigned a = (unsigned)__builtin_amdgcn_readlane((int)v[3][0], 0) & 15u;
    const unsigned TP = (((a * 0x01010101u) + 0x00020100u) & 0x0F0F0F0Fu) << 3;
    const unsigned bs = ((a - 5u) & 15u) * 0x01010101u;
    const unsigned TL = ((bs + 0x03020100u) & 0x0F0F0F0Fu) << 3;
    const unsigned TH = ((bs + 0x0A090804u) & 0x0F0F0F0Fu) << 3;
    const unsigned M03 = opaque(0x03030303u), M01 = opaque(0x01010101u), M07 = opaque(0x07070707u), K3 = 0x010101u;
    const unsigned MFF = opaque(0xFF000000u), M16 = opaque(0x0000FFFFu), M8 = opaque(0x00FF00FFu), M80 = opaque(0x80808080u), M78 = opaque(0x78787878u);
    unsigned C[16], FM[16], IX[16], nb[2][9];
#pragma unroll
    for (int j = 0; j < 2; ++j) {
        nb[j][0] = 0;
#pragma unroll
        for (int i = 0; i < 8; ++i) {
            const unsigned c = (v[2][2 * j + (i >> 2)] >> (2 * (i & 3))) & M03;
            const unsigned fm = perm(0u, MFF, c);
            const unsigned f1 = fm & M01;
            C[8 * j + i] = c; FM[8 * j + i] = fm;
            IX[8 * j + i] = perm(__umul24(f1, K3), 0u, 0x0605040Cu);
            nb[j][i + 1] = __builtin_amdgcn_udot4(f1, M03, nb[j][i], false);
        }
    }
    const unsigned cnt = nb[0][8] | (nb[1][8] << 16);
    const unsigned inc = wave_scan(cnt);
    const unsigned t0 = (unsigned)__builtin_amdgcn_readlane((int)inc, 63) & 0xffffu;
    const unsigned exc = inc - cnt;
    u64 win[2];
    {
        const unsigned off0 = 32u + (exc & 0xffffu), off1 = 32u + t0 + (exc >> 16);
        unsigned d2, e2;
        asm volatile("ds_read2_b32 %0, %4 offset1:1\n\tds_read_b32 %1, %4 offset:8\n\t"
                     "ds_read2_b32 %2, %5 offset1:1\n\tds_read_b32 %3, %5 offset:8\n\ts_waitcnt lgkmcnt(0)"
                     : "=&v"(win[0]), "=&v"(d2), "=&v"(win[1]), "=&v"(e2) : "v"(scr + (off0 >> 5) * 4), "v"(scr + (off1 >> 5) * 4) : "memory");
        const unsigned d0 = (unsigned)win[0], d1 = (unsigned)(win[0] >> 32), e0 = (unsigned)win[1], e1 = (unsigned)(win[1] >> 32);
        win[0] = (u64)__builtin_amdgcn_alignbit(d1, d0, off0) | ((u64)__builtin_amdgcn_alignbit(d2, d1, off0) << 32);
        win[1] = (u64)__builtin_amdgcn_alignbit(e1, e0, off1) | ((u64)__builtin_amdgcn_alignbit(e2, e1, off1) << 32);
    }
    unsigned out[16];
#pragma unroll
    for (int j = 0; j < 2; ++j) {
#pragma unroll
        for (int i = 0; i < 8; ++i) {
            const unsigned w = (unsigned)(win[j] >> nb[j][i]);
            const unsigned u = sel3(M16, w, w << 10);
            const unsigned sp = sel3(M8, u, u << 5) & M07;
            const unsigned es = perm(TH, TL, perm(0u, sp, IX[8 * j + i]));
            const unsigned e = sel3(FM[8 * j + i], es, perm(0u, TP, C[8 * j + i]));
            const unsigned n = v[j][i >> 1];
            const unsigned sm = (i & 1) ? sel3(M80, n, n >> 4) : sel3(M80, n << 4, n);
            out[8 * j + i] = sel3(M78, e, sm);
        }
    }
    x0 = v8i{(int)out[0], (int)out[1], (int)out[2], (int)out[3], (int)out[4], (int)out[5], (int)out[6], (int)out[7]};
    x1 = v8i{(int)out[8], (int)out[9], (int)out[10], (int)out[11], (int)out[12], (int)out[13], (int)out[14], (int)out[15]};
}
__device__ __forceinline__ unsigned compact_sel(unsigned m) {
    unsigned s = 0x0C0C0C0Cu, k = 0;
#pragma unroll
    for (unsigned b = 0; b < 4; ++b)
        if (m & (1u << b)) { s = (s & ~(0xFFu << (8 * k))) | (b << (8 * k)); ++k; }
    return s;
}

#define STAMP(i) do { } while (0)

__global__ void __attribute__((amdgpu_flat_work_group_size(384, 384), amdgpu_waves_per_eu(2, 2))) ag_gemm(Params p) {
    extern __shared__ __attribute__((aligned(16))) unsigned char smem[];
    const int lane = threadIdx.x & 63;
    const int wave = uni(threadIdx.x >> 6);
    const int b = blockIdx.x;
    const int rt_ = b & 7;
    const int ct = (b >> 3) & 3;
    const int srcsel = b >> 5;
    const unsigned lds0 = (unsigned)(size_t)(LAS unsigned char*)smem;
    const unsigned ctl = lds0 + LDS_CTL;
    unsigned char* me = p.me;
    unsigned* tmo = (unsigned*)(me + TMO_OFF);
    unsigned* ctr = (unsigned*)(me + CTR_OFF + (b * 8 + wave) * 64);
    STAMP(0);
    const int g = lane >> 4, r = lane & 15;

    if (srcsel == 0) {
        const int rb = rt_ * 4 + ct;
        const unsigned char* xs = p.x + (size_t)(rb * 16 + r) * KDIM + 16 * g;
        if (wave == 5) {
            u32x4 d0[4];
#pragma unroll
            for (int i = 0; i < 4; ++i) d0[i] = *(const u32x4*)(xs + (i >> 1) * 128 + (i & 1) * 64);
            const unsigned call_v = ld_agent(ctr);
            unsigned plo = 0, phi = 0;
#pragma unroll
            for (int d = 0; d < NR - 1; ++d) {
                plo = lane == d ? (unsigned)(uintptr_t)p.peer[d] : plo;
                phi = lane == d ? (unsigned)((uintptr_t)p.peer[d] >> 32) : phi;
            }
            __builtin_amdgcn_s_barrier();
#pragma unroll
            for (int i = 0; i < 4; ++i) asm volatile("" : "+v"(d0[i]));
            const unsigned call = uni(call_v) + 1u;
            if (lane == 0) st_agent(ctr, call);
            STAMP(1);
            const size_t doff = (size_t)(call & 1u) * DATA_PAR + (size_t)p.rank * SLOT_BYTES + (size_t)rb * 65536 + lane * 16;
            unsigned char* dst[NR - 1];
            const int rot = rb % 7;
#pragma unroll
            for (int d = 0; d < NR - 1; ++d) {
                int q = d + rot; if (q >= 7) q -= 7;
                const uintptr_t a = (uintptr_t)(unsigned)__builtin_amdgcn_readlane(plo, q) | ((uintptr_t)(unsigned)__builtin_amdgcn_readlane(phi, q) << 32);
                dst[d] = (unsigned char*)a + doff;
            }
            u64* fl = (u64*)((((uintptr_t)phi << 32) | plo) + FLAG_OFF + (size_t)(p.rank * 32 + rb) * 128);
            const unsigned fbase = call * 64u;
            unsigned rawmask = 1u;
            const u64 tstart = rt();
            unsigned sent = 4096;
#pragma unroll
            for (int d = 0; d < NR - 1; ++d) {
#pragma unroll
                for (int i = 0; i < 4; ++i) st16_sys(dst[d] + i * 1024, d0[i]);
                if (4 * (d + 1) > WN) waitvm<WN>();
            }
            for (int grp = 1; grp < NGRP; ++grp) {
                unsigned meta;
                if (!wait_lds(ctl + 4 * grp, 1u, meta, tmo, call)) return;
                STAMP(8 + grp);
                const unsigned nl = meta & 0xffu;
                rawmask |= (nl == 0u ? 1u : 0u) << grp;
                const u64 em = (nl == 0u || nl >= 64u) ? ~0ull : ((1ull << nl) - 1ull);
                u32x4 v[4];
                lds_ld16x4(lds0 + grp * 4096 + lane * 16, v);
                if (PACE) {
                    const u64 due = tstart + (u64)(long long)(PACEOFF) + (((u64)sent * PACE) >> 16);
                    while (rt() < due) __builtin_amdgcn_s_sleep(1);
                    sent += nl ? 3072u + nl * 16u : 4096u;
                }
#pragma unroll
                for (int d = 0; d < NR - 1; ++d) {
#pragma unroll
                    for (int i = 0; i < 3; ++i) st16_sys(dst[d] + grp * 4096 + i * 1024, v[i]);
                    st16_sys_masked(dst[d] + grp * 4096 + 3072, v[3], em);
                    waitvm<WN>();
                    if (4 * (d + 1) == WN && lane < 7) st_sys64(fl, ((u64)rawmask << 32) | (fbase + (unsigned)grp));
                }
            }
            waitvm<0>();
            if (lane < 7) st_sys64(fl, ((u64)rawmask << 32) | (fbase + (unsigned)NGRP));
            STAMP(2);
            return;
        }
        const unsigned lut = lds0 + LDS_SCR;
        const unsigned call_v = ld_agent(ctr);
        if (wave == 4) {
            if (lane < 32) lds_st(ctl + lane * 4, 0u);
            if (lane < 16) lds_st(lut + lane * 4, compact_sel(lane));
            asm volatile("s_waitcnt lgkmcnt(0)" ::: "memory");
        }
        __builtin_amdgcn_s_barrier();
        u32x4 rv[3][4];
#pragma unroll
        for (int t = 0; t < 3; ++t)
#pragma unroll
            for (int i = 0; i < 4; ++i) rv[t][i] = *(const u32x4*)(xs + (1 + wave + 5 * t) * 256 + (i >> 1) * 128 + (i & 1) * 64);
#pragma unroll
        for (int t = 0; t < 3; ++t) {
            const int grp = 1 + wave + 5 * t;
            unsigned R[16];
#pragma unroll
            for (int i = 0; i < 16; ++i) R[i] = rv[t][i >> 2][i & 3];
            const unsigned nl = pack_group(R, lds0 + grp * 4096, lut, lane, !PACK || grp == NGRP - 1);
            lds_st(ctl + 4 * grp, 0x100u | nl);
            if (t == 0) STAMP(1);
        }
        if (lane == 0) st_agent(ctr, uni(call_v) + 1u);
        STAMP(2);
        return;
    }

    const bool has_local = srcsel <= 4;
    if (wave == 4) {
        const unsigned char* wsrc = p.w + (size_t)(lane >> 3) * NLOC + ct * 128;
        const unsigned c0 = (unsigned)(((lane & 7) ^ ((lane >> 4) & 3)) << 4);
        const unsigned c1 = c0 ^ 64u;
        if (lane < 32) lds_st(ctl + lane * 4, (lane == 5 && !has_local) ? 0x7fffffffu : 0u);
        asm volatile("s_waitcnt lgkmcnt(0)" ::: "memory");
#pragma unroll 4
        for (int i = 0; i < 32; ++i) glds16(wsrc + (size_t)i * 8 * NLOC + (((i >> 1) & 1) ? c1 : c0), lds0 + i * 1024);
        const unsigned call_v = ld_agent(ctr);
        __builtin_amdgcn_s_barrier();
        const unsigned call = uni(call_v) + 1u;
        if (lane == 0) st_agent(ctr, call);
        bool ok = true;
        for (int st = 1; st < NGRP; ++st) {
            if (st >= NS) {
                const unsigned need = (unsigned)(st - NS + 1);
                bool fr = true;
                for (int w = 0; w < 5; ++w) fr = fr && lds_ld(ctl + 4 + 4 * w) >= need;
                if (!fr) {
                    waitvm<0>();
                    lds_st(ctl, (unsigned)st);
                    unsigned got;
                    for (int w = 0; w < 5; ++w) ok = ok && wait_lds<LSLEEP>(ctl + 4 + 4 * w, need, got, tmo, call);
                }
            }
            const unsigned slot = lds0 + (unsigned)(st & (NS - 1)) * STAGE;
            const unsigned char* ws_ = wsrc + (size_t)st * 256 * NLOC;
#pragma unroll 4
            for (int i = 0; i < 32; ++i) glds16(ws_ + (size_t)i * 8 * NLOC + (((i >> 1) & 1) ? c1 : c0), slot + i * 1024);
            waitvm<32>();
            lds_st(ctl, (unsigned)st);
        }
        waitvm<0>();
        lds_st(ctl, (unsigned)NGRP);
        asm volatile("s_waitcnt lgkmcnt(0)" ::: "memory");
        STAMP(2);
        return;
    }

    const bool local = wave == 5;
    const unsigned call_v = ld_agent(ctr);
    const float scale = p.sx[0] * p.sw[0];
    __builtin_amdgcn_s_barrier();
    const unsigned call = uni(call_v) + 1u;
    if (lane == 0) st_agent(ctr, call);
    if (local && !has_local) return;
    STAMP(1);

    const int src = local ? p.rank : ((p.rank + srcsel) & (NR - 1));
    const int rb = local ? (srcsel - 1) * 8 + rt_ : rt_ * 4 + wave;
    const unsigned char* xbase = local ? p.x : me + (size_t)(call & 1u) * DATA_PAR + (size_t)src * SLOT_BYTES;
    const __amdgpu_buffer_rsrc_t xr = __builtin_amdgcn_make_buffer_rsrc((void*)xbase, 0, (int)SLOT_BYTES, 0x00020000);
    const int voff = local ? (rb * 16 + r) * KDIM + 16 * g : rb * 65536 + lane * 16;
    const int kstride = local ? 128 : 2048, hstride = local ? 64 : 1024;
    const u64* flag = (const u64*)(me + FLAG_OFF + (size_t)(src * 32 + rb) * 128);
    const unsigned fbase = call * 64u;
    const unsigned done_addr = ctl + 4 + 4 * (local ? 4 : wave);
    const unsigned scr = lds0 + LDS_SCR + (local ? 4 : wave) * SCR_BYTES;

    const unsigned q = (unsigned)(lane & 15) >> 1, pp = lane & 1;
    const unsigned swz = ((q >> 1) & 3u) | ((unsigned)(g & 1) << 2);
    const unsigned abase = lds0 + (16u * g + q) * 128u + 8u * pp + (swz << 4);

    v4f acc[8];
#pragma unroll
    for (int c = 0; c < 8; ++c) acc[c] = v4f{0.f, 0.f, 0.f, 0.f};

    bool ok = true;
    unsigned tmo_v = 0, got;
    auto load_w1 = [&](unsigned sb, int c) -> v8i {
        LAS unsigned char* a = (LAS unsigned char*)(size_t)(sb ^ (c << 4));
        v2i w0 = __builtin_amdgcn_ds_read_tr8_b64_v2i32((LAS v2i*)(a));
        v2i w1 = __builtin_amdgcn_ds_read_tr8_b64_v2i32((LAS v2i*)(a + 1024));
        v2i w2 = __builtin_amdgcn_ds_read_tr8_b64_v2i32((LAS v2i*)(a + 8192));
        v2i w3 = __builtin_amdgcn_ds_read_tr8_b64_v2i32((LAS v2i*)(a + 8192 + 1024));
        return v8i{w0[0], w0[1], w1[0], w1[1], w2[0], w2[1], w3[0], w3[1]};
    };
    v8i wf[4];
    ok = wait_lds(ctl, 1u, got, tmo, call);
#pragma unroll
    for (int c = 0; c < 4; ++c) wf[c] = load_w1(abase, c);

    unsigned flo = local ? ~0u : 0u, fhi = local ? ~0u : 0u;
    auto load_x = [&](u32x4* xv, int kg, bool have) {
        const int vo = have ? voff : 0x7ff00000;
#pragma unroll
        for (int i = 0; i < 4; ++i)
            xv[i] = __builtin_amdgcn_raw_buffer_load_b128(xr, vo, (2 * kg + (i >> 1)) * kstride + (i & 1) * hstride, 17);
    };
    auto compute = [&](const u32x4* xv, int kg, bool raw) {
        v8i x0, x1;
        if (raw) {
            x0 = v8i{(int)xv[0][0], (int)xv[0][1], (int)xv[0][2], (int)xv[0][3], (int)xv[1][0], (int)xv[1][1], (int)xv[1][2], (int)xv[1][3]};
            x1 = v8i{(int)xv[2][0], (int)xv[2][1], (int)xv[2][2], (int)xv[2][3], (int)xv[3][0], (int)xv[3][1], (int)xv[3][2], (int)xv[3][3]};
        } else {
            unpack_group(xv, scr, lane, x0, x1);
        }
        const unsigned sb = abase + (unsigned)(kg & (NS - 1)) * STAGE;
#pragma unroll
        for (int c = 0; c < 8; ++c) {
            const v8i cur = wf[c & 3];
            acc[c] = __builtin_amdgcn_mfma_scale_f32_16x16x128_f8f6f4(cur, x0, acc[c], 0, 0, 0, 0, 0, 0);
            wf[c & 3] = c < 4 ? load_w1(sb, c + 4) : load_w1(sb + 16384, c - 4);
            __builtin_amdgcn_sched_group_barrier(0x008, 1, 0);
            __builtin_amdgcn_sched_group_barrier(0x100, 4, 0);
        }
#pragma unroll
        for (int c = 0; c < 4; ++c) {
            const v8i cur = wf[c];
            acc[c] = __builtin_amdgcn_mfma_scale_f32_16x16x128_f8f6f4(cur, x1, acc[c], 0, 0, 0, 0, 0, 0);
            wf[c] = load_w1(sb + 16384, c + 4);
            __builtin_amdgcn_sched_group_barrier(0x008, 1, 1);
            __builtin_amdgcn_sched_group_barrier(0x100, 4, 1);
        }
        lds_st(done_addr, (unsigned)(kg + 1));
        const int kn = kg + 1 < NGRP ? kg + 1 : kg;
        if (kg + 1 < NGRP) ok = ok && wait_lds<4>(ctl, (unsigned)(kn + 1), got, tmo, call);
        const unsigned sn = abase + (unsigned)(kn & (NS - 1)) * STAGE;
#pragma unroll
        for (int c = 4; c < 8; ++c) {
            const v8i cur = wf[c & 3];
            acc[c] = __builtin_amdgcn_mfma_scale_f32_16x16x128_f8f6f4(cur, x1, acc[c], 0, 0, 0, 0, 0, 0);
            wf[c & 3] = load_w1(sn, c - 4);
            __builtin_amdgcn_sched_group_barrier(0x008, 1, 2);
            __builtin_amdgcn_sched_group_barrier(0x100, 4, 2);
        }
    };
    auto step = [&](const u32x4* cur, u32x4* nxt, int kg) {
        const bool more = kg + 1 < NGRP;
        const unsigned need = fbase + (unsigned)kg + 2u;
        const bool raw = ((fhi >> kg) & 1u) || !ok;
        const bool early = more && ok && flo >= need;
        load_x(nxt, kg + 1, early);
        const u64 fv = ld_sys64(flag);
        compute(cur, kg, raw);
        if (more && !early) {
            if (!local) {
                flo = uni((unsigned)fv); fhi = uni((unsigned)(fv >> 32));
                if (flo < need) {
                    if (kg + FASTN >= NGRP - 1) ok = ok && wait_flag_fast(flag, need, flo, fhi, tmo, call);
                    else ok = ok && wait_flag(flag, need, flo, fhi, tmo, call);
                }
            }
            asm volatile("" ::: "memory");
            if (kg + 2 == NGRP) { STAMP(2); }
            load_x(nxt, kg + 1, ok);
        } else if (!local) {
            const unsigned l2 = uni((unsigned)fv);
            if (l2 > flo) { flo = l2; fhi = uni((unsigned)(fv >> 32)); }
        }
        if (kg + 2 == NGRP) tmo_v = ld_agent(tmo);
    };
    u32x4 xa[4], xb[4];
    if (!local) ok = ok && wait_flag(flag, fbase + 1u, flo, fhi, tmo, call);
    asm volatile("" ::: "memory");
    load_x(xa, 0, ok);
    for (int kg = 0; kg < NGRP; kg += 2) {
        step(xa, xb, kg);
        step(xb, xa, kg + 1);
    }
    STAMP(3);

    const bool bad = uni(tmo_v) == call;
    float* o = p.out + (size_t)(src * ROWS + rb * 16 + r) * NLOC + ct * 128 + 4 * g;
#pragma unroll
    for (int c = 0; c < 8; ++c) {
        v4f y = acc[c] * scale;
#pragma unroll
        for (int j = 0; j < 4; ++j) y[j] = y[j] < 0.f ? 0.f : y[j];
        if (bad) y = v4f{__builtin_nanf(""), __builtin_nanf(""), __builtin_nanf(""), __builtin_nanf("")};
        *(v4f*)(o + 16 * c) = y;
    }
    STAMP(4);
}

}

extern "C" void kernel_launch(int rank, int nranks, ncclComm_t comm, hipStream_t stream,
                              void* const* d_in, const int* in_sizes, int n_in,
                              void* d_out, int out_size,
                              const int* in_dtypes, int out_dtype,
                              void* const* d_out_all,
                              void* const* d_ws_all, size_t ws_bytes) {
    (void)comm; (void)in_dtypes; (void)out_dtype; (void)d_out_all;
    if (nranks != NR || (unsigned)rank >= (unsigned)NR || n_in != 4 || ws_bytes < WS_NEED ||
        in_sizes[0] != ROWS * KDIM || in_sizes[1] != KDIM * NLOC || out_size != NR * ROWS * NLOC) {
        fprintf(stderr, "kernel_launch: unexpected problem shape\n");
        return;
    }
    static bool init = false;
    if (!init) {
        HIP_CHECK(hipFuncSetAttribute((const void*)ag_gemm, hipFuncAttributeMaxDynamicSharedMemorySize, LDS_BYTES));
        init = true;
    }
    Params p;
    p.x = (const unsigned char*)d_in[0];
    p.w = (const unsigned char*)d_in[1];
    p.sx = (const float*)d_in[2];
    p.sw = (const float*)d_in[3];
    p.out = (float*)d_out;
    p.me = (unsigned char*)d_ws_all[rank];
    for (int d = 0; d < NR - 1; ++d) p.peer[d] = (unsigned char*)d_ws_all[(rank + 1 + d) % NR];
    p.rank = rank;
    p.pad = 0;
    hipLaunchKernelGGL(ag_gemm, dim3(256), dim3(384), LDS_BYTES, stream, p);
    HIP_CHECK(hipGetLastError());
}
```

```cpp
#include <hip/hip_runtime.h>
#include <rccl/rccl.h>
#include <cstdio>
#include <cstdint>

#define HIP_CHECK(expr) do { hipError_t _e = (expr); if (_e != hipSuccess) { \
    fprintf(stderr, "HIP error %s at %s:%d\n", hipGetErrorString(_e), __FILE__, __LINE__); \
    return; } } while (0)

#define PACK 1
#define LSLEEP 12
#define PACE 3750
#define PACEOFF 0
#define OUTMODE 2
#define LAS __attribute__((address_space(3)))
typedef unsigned u32x4 __attribute__((ext_vector_type(4)));
typedef unsigned long long u64;
typedef int v8i __attribute__((ext_vector_type(8)));
typedef int v2i __attribute__((ext_vector_type(2)));
typedef float v4f __attribute__((ext_vector_type(4)));

namespace {

constexpr int NR = 8;
constexpr int KDIM = 4096;
constexpr int NLOC = 512;
constexpr int ROWS = 512;
constexpr unsigned SLOT_BYTES = ROWS * KDIM;
constexpr size_t DATA_PAR = (size_t)NR * SLOT_BYTES;
constexpr size_t FB_OFF = 2 * DATA_PAR;
constexpr size_t FB_PAR = (size_t)NR * 32 * 8192;
constexpr size_t FLAG_OFF = FB_OFF + 2 * FB_PAR;
constexpr size_t CTR_OFF = FLAG_OFF + 64 * 1024;
constexpr size_t TMO_OFF = CTR_OFF + 256 * 8 * 64;
constexpr size_t DBG_OFF = TMO_OFF + 4096;
constexpr size_t WS_NEED = DBG_OFF + 256 * 8 * 32 * 8;
constexpr u64 DEADLINE = 30000000ull;

constexpr int NGRP = 16;
constexpr int NFLAG = NGRP - 2;
#define SMASK 0x7F7F7F7Fu
constexpr int NS = 4;
constexpr int STAGE = 256 * 128;
constexpr int LDS_CTL = NS * STAGE;
constexpr int LDS_SCR = LDS_CTL + 256;
constexpr int SCR_BYTES = 1024 + 64;
constexpr int LDS_BYTES = LDS_SCR + 5 * SCR_BYTES;
#define WNUM 24
constexpr int WN = WNUM;

struct Params {
    const unsigned char* x;
    const unsigned char* w;
    const float* sx;
    const float* sw;
    float* out;
    unsigned char* me;
    unsigned char* peer[NR - 1];
    int rank;
    int pad;
};

__device__ __forceinline__ u64 rt() { return __builtin_amdgcn_s_memrealtime(); }
__device__ __forceinline__ unsigned uni(unsigned v) { return __builtin_amdgcn_readfirstlane(v); }
template <int N> __device__ __forceinline__ void waitvm() { asm volatile("s_waitcnt vmcnt(%0)" :: "i"(N) : "memory"); }
__device__ __forceinline__ void st16_sys(void* p, u32x4 v) {
    asm volatile("global_store_dwordx4 %0, %1, off sc0 sc1\n\ts_nop 1" :: "v"(p), "v"(v) : "memory");
}
__device__ __forceinline__ void st16_sys_masked(void* p, u32x4 v, u64 mask) {
    u64 keep;
    asm volatile("s_mov_b64 %0, exec\n\ts_mov_b64 exec, %3\n\tglobal_store_dwordx4 %1, %2, off sc0 sc1\n\ts_mov_b64 exec, %0\n\ts_nop 1"
                 : "=&s"(keep) : "v"(p), "v"(v), "s"(mask) : "memory");
}
__device__ __forceinline__ void glds16(const void* gsrc, unsigned lds_dst) {
    unsigned keep;
    asm volatile("s_mov_b32 %0, m0\n\ts_mov_b32 m0, %2\n\ts_nop 0\n\tglobal_load_lds_dwordx4 %1, off\n\ts_mov_b32 m0, %0"
                 : "=&s"(keep) : "v"(gsrc), "s"(lds_dst) : "memory");
}
__device__ __forceinline__ void lds_st(unsigned addr, unsigned v) {
    asm volatile("ds_write_b32 %0, %1" :: "v"(addr), "v"(v) : "memory");
}
__device__ __forceinline__ void lds_st16(unsigned addr, u32x4 v) {
    asm volatile("ds_write_b128 %0, %1" :: "v"(addr), "v"(v) : "memory");
}
__device__ __forceinline__ void lds_or(unsigned addr, unsigned v) {
    asm volatile("ds_or_b32 %0, %1" :: "v"(addr), "v"(v) : "memory");
}
__device__ __forceinline__ unsigned lds_ldv(unsigned addr) {
    unsigned v;
    asm volatile("ds_read_b32 %0, %1\n\ts_waitcnt lgkmcnt(0)" : "=v"(v) : "v"(addr) : "memory");
    return v;
}
__device__ __forceinline__ unsigned lds_ld(unsigned addr) { return uni(lds_ldv(addr)); }
__device__ __forceinline__ void lds_ld16x4(unsigned addr, u32x4* v) {
    asm volatile("ds_read_b128 %0, %4\n\tds_read_b128 %1, %4 offset:1024\n\tds_read_b128 %2, %4 offset:2048\n\t"
                 "ds_read_b128 %3, %4 offset:3072\n\ts_waitcnt lgkmcnt(0)"
                 : "=&v"(v[0]), "=&v"(v[1]), "=&v"(v[2]), "=&v"(v[3]) : "v"(addr) : "memory");
}
__device__ __forceinline__ unsigned ld_agent(const unsigned* p) { return __hip_atomic_load(p, __ATOMIC_RELAXED, __HIP_MEMORY_SCOPE_AGENT); }
__device__ __forceinline__ void st_agent(unsigned* p, unsigned v) { __hip_atomic_store(p, v, __ATOMIC_RELAXED, __HIP_MEMORY_SCOPE_AGENT); }
__device__ __forceinline__ u64 ld_sys64(const u64* p) { return __hip_atomic_load(p, __ATOMIC_RELAXED, __HIP_MEMORY_SCOPE_SYSTEM); }
__device__ __forceinline__ void st_sys64(u64* p, u64 v) { __hip_atomic_store(p, v, __ATOMIC_RELAXED, __HIP_MEMORY_SCOPE_SYSTEM); }

template <int SLEEP = 1>
__device__ __forceinline__ bool wait_lds(unsigned addr, unsigned want, unsigned& got, unsigned* tmo, unsigned call) {
    got = lds_ld(addr);
    if (got >= want) return true;
    const u64 t0 = rt();
    for (unsigned spins = 1;; ++spins) {
        __builtin_amdgcn_s_sleep(SLEEP);
        got = lds_ld(addr);
        if (got >= want) return true;
        if ((spins & 255u) == 0u) {
            if (uni(ld_agent(tmo)) == call || rt() - t0 > DEADLINE) { st_agent(tmo, call); return false; }
        }
    }
}

__device__ __forceinline__ unsigned perm(unsigned hi, unsigned lo, unsigned sel) { return __builtin_amdgcn_perm(hi, lo, sel); }
__device__ __forceinline__ unsigned bfi(unsigned m, unsigned a, unsigned b) { return (m & a) | (~m & b); }
__device__ __forceinline__ unsigned wave_scan(unsigned v) {
    v += __builtin_amdgcn_update_dpp(0u, v, 0x111, 0xf, 0xf, true);
    v += __builtin_amdgcn_update_dpp(0u, v, 0x112, 0xf, 0xf, true);
    v += __builtin_amdgcn_update_dpp(0u, v, 0x114, 0xf, 0xf, true);
    v += __builtin_amdgcn_update_dpp(0u, v, 0x118, 0xf, 0xf, true);
    v += __builtin_amdgcn_update_dpp(0u, v, 0x142, 0xa, 0xf, false);
    v += __builtin_amdgcn_update_dpp(0u, v, 0x143, 0xc, 0xf, false);
    return v;
}

__device__ __forceinline__ unsigned pack_group(const unsigned* R, unsigned slot, unsigned lut, int lane, bool force_raw) {
    unsigned nl = 0;
    if (!force_raw) {
        unsigned s = 0;
#pragma unroll
        for (int i = 0; i < 16; i += 5) s = __builtin_amdgcn_udot4((R[i] >> 3) & 0x0F0F0F0Fu, 0x01010101u, s, false);
        const unsigned tot = (unsigned)__builtin_amdgcn_readlane((int)wave_scan(s), 63);
        const unsigned a = tot > 102u ? (tot - 102u) >> 10 : 0u;
        const unsigned K = ((5u - a) & 15u) * 0x01010101u;
        unsigned bad = 0, N[8], P[4], n[2];
        u64 acc[2];
#pragma unroll
        for (int j = 0; j < 2; ++j) {
            u64 ac = 0;
            unsigned pos = 0;
#pragma unroll
            for (int q = 0; q < 2; ++q) {
                unsigned pw = 0;
#pragma unroll
                for (int i = 0; i < 4; ++i) {
                    const unsigned r = R[8 * j + 4 * q + i];
                    const unsigned D = (((r >> 3) & 0x0F0F0F0Fu) + K) & 0x0F0F0F0Fu;
                    bad |= (D + 0x05050505u) & 0x10101010u;
                    const unsigned C = perm(0x820180FFu, 0xFFFFFFFFu, D) & 0x03030303u;
                    const unsigned f = perm(0u, 0x01000000u, C);
                    const unsigned t = (D >> 3) & 0x01010101u;
                    const unsigned S = D - (t + (t << 1));
                    const unsigned m = __builtin_amdgcn_udot4(f, 0x08040201u, 0u, false);
                    const unsigned sel = lds_ldv(lut + m * 4);
                    const unsigned c4 = perm(0u, S, sel);
                    const unsigned w = __builtin_amdgcn_udot4(c4, 0x00400801u, (c4 >> 24) << 9, false);
                    ac |= (u64)w << (pos & 63u);
                    pos += 3u * (unsigned)__builtin_popcount(f);
                    pw |= C << (2 * i);
                }
                P[2 * j + q] = pw;
            }
#pragma unroll
            for (int m2 = 0; m2 < 4; ++m2) {
                const unsigned ra = R[8 * j + 2 * m2], rb = R[8 * j + 2 * m2 + 1];
                N[4 * j + m2] = bfi(0x0F0F0F0Fu, bfi(0x08080808u, ra >> 4, ra), bfi(0x70707070u, rb << 4, rb));
            }
            acc[j] = ac; n[j] = pos;
        }
        const unsigned cnt = n[0] | (n[1] << 16);
        const unsigned inc = wave_scan(cnt);
        const unsigned tot2 = (unsigned)__builtin_amdgcn_readlane((int)inc, 63);
        const unsigned t0 = tot2 & 0xffffu, t1 = tot2 >> 16;
        const bool lane_bad = bad != 0u || n[0] > 63u || n[1] > 63u;
        const unsigned bits = 32u + t0 + t1;
        if (!__any(lane_bad) && bits <= 8192u) {
            const unsigned exc = inc - cnt;
            lds_st16(slot + lane * 16, u32x4{N[0], N[1], N[2], N[3]});
            lds_st16(slot + 1024 + lane * 16, u32x4{N[4], N[5], N[6], N[7]});
            lds_st16(slot + 2048 + lane * 16, u32x4{P[0], P[1], P[2], P[3]});
            lds_st16(slot + 3072 + lane * 16, u32x4{lane == 0 ? a : 0u, 0u, 0u, 0u});
#pragma unroll
            for (int j = 0; j < 2; ++j) {
                const unsigned off = 32u + (j ? t0 + (exc >> 16) : (exc & 0xffffu));
                const unsigned sh = off & 31u, ad = slot + 3072 + (off >> 5) * 4;
                const u64 lo = acc[j] << sh;
                const unsigned hi = (unsigned)((acc[j] >> 32) >> (32u - sh));
                lds_or(ad, (unsigned)lo);
                lds_or(ad + 4, (unsigned)(lo >> 32));
                if ((off >> 5) + 2 < 256u) lds_or(ad + 8, hi);
            }
            nl = ((bits + 511u) >> 9) * 4u;
        }
    }
    if (nl == 0) {
#pragma unroll
        for (int i = 0; i < 4; ++i) lds_st16(slot + i * 1024 + lane * 16, u32x4{R[4 * i], R[4 * i + 1], R[4 * i + 2], R[4 * i + 3]});
    }
    return nl;
}

__device__ __forceinline__ unsigned opaque(unsigned k) { asm("" : "+s"(k)); return k; }
__device__ __forceinline__ unsigned sel3(unsigned m, unsigned a, unsigned b) { return __builtin_amdgcn_bitop3_b32(m, a, b, 0xCA); }
__device__ __forceinline__ void unpack_group(const u32x4* v, unsigned scr, int lane, v8i& x0, v8i& x1) {
    lds_st16(scr + lane * 16, v[3]);
    const unsigned a = (unsigned)__builtin_amdgcn_readlane((int)v[3][0], 0) & 15u;
    const unsigned TP = (((a * 0x01010101u) + 0x00020100u) & 0x0F0F0F0Fu) << 3;
    const unsigned bs = ((a - 5u) & 15u) * 0x01010101u;
    const unsigned TL = ((bs + 0x03020100u) & 0x0F0F0F0Fu) << 3;
    const unsigned TH = ((bs + 0x0A090804u) & 0x0F0F0F0Fu) << 3;
    const unsigned M03 = opaque(0x03030303u), M01 = opaque(0x01010101u), M07 = opaque(0x07070707u), K3 = 0x010101u;
    const unsigned MFF = opaque(0xFF000000u), M16 = opaque(0x0000FFFFu), M8 = opaque(0x00FF00FFu), M80 = opaque(0x80808080u), M78 = opaque(0x78787878u);
    unsigned C[16], FM[16], IX[16], nb[2][9];
#pragma unroll
    for (int j = 0; j < 2; ++j) {
        nb[j][0] = 0;
#pragma unroll
        for (int i = 0; i < 8; ++i) {
            const unsigned c = (v[2][2 * j + (i >> 2)] >> (2 * (i & 3))) & M03;
            const unsigned fm = perm(0u, MFF, c);
            const unsigned f1 = fm & M01;
            C[8 * j + i] = c; FM[8 * j + i] = fm;
            IX[8 * j + i] = perm(__umul24(f1, K3), 0u, 0x0605040Cu);
            nb[j][i + 1] = __builtin_amdgcn_udot4(f1, M03, nb[j][i], false);
        }
    }
    const unsigned cnt = nb[0][8] | (nb[1][8] << 16);
    const unsigned inc = wave_scan(cnt);
    const unsigned t0 = (unsigned)__builtin_amdgcn_readlane((int)inc, 63) & 0xffffu;
    const unsigned exc = inc - cnt;
    u64 win[2];
    {
        const unsigned off0 = 32u + (exc & 0xffffu), off1 = 32u + t0 + (exc >> 16);
        unsigned d2, e2;
        asm volatile("ds_read2_b32 %0, %4 offset1:1\n\tds_read_b32 %1, %4 offset:8\n\t"
                     "ds_read2_b32 %2, %5 offset1:1\n\tds_read_b32 %3, %5 offset:8\n\ts_waitcnt lgkmcnt(0)"
                     : "=&v"(win[0]), "=&v"(d2), "=&v"(win[1]), "=&v"(e2) : "v"(scr + (off0 >> 5) * 4), "v"(scr + (off1 >> 5) * 4) : "memory");
        const unsigned d0 = (unsigned)win[0], d1 = (unsigned)(win[0] >> 32), e0 = (unsigned)win[1], e1 = (unsigned)(win[1] >> 32);
        win[0] = (u64)__builtin_amdgcn_alignbit(d1, d0, off0) | ((u64)__builtin_amdgcn_alignbit(d2, d1, off0) << 32);
        win[1] = (u64)__builtin_amdgcn_alignbit(e1, e0, off1) | ((u64)__builtin_amdgcn_alignbit(e2, e1, off1) << 32);
    }
    unsigned out[16];
#pragma unroll
    for (int j = 0; j < 2; ++j) {
#pragma unroll
        for (int i = 0; i < 8; ++i) {
            const unsigned w = (unsigned)(win[j] >> nb[j][i]);
            const unsigned u = sel3(M16, w, w << 10);
            const unsigned sp = sel3(M8, u, u << 5) & M07;
            const unsigned es = perm(TH, TL, perm(0u, sp, IX[8 * j + i]));
            const unsigned e = sel3(FM[8 * j + i], es, perm(0u, TP, C[8 * j + i]));
            const unsigned n = v[j][i >> 1];
            const unsigned sm = (i & 1) ? sel3(M80, n, n >> 4) : sel3(M80, n << 4, n);
            out[8 * j + i] = sel3(M78, e, sm);
        }
    }
    x0 = v8i{(int)out[0], (int)out[1], (int)out[2], (int)out[3], (int)out[4], (int)out[5], (int)out[6], (int)out[7]};
    x1 = v8i{(int)out[8], (int)out[9], (int)out[10], (int)out[11], (int)out[12], (int)out[13], (int)out[14], (int)out[15]};
}
__device__ __forceinline__ unsigned compact_sel(unsigned m) {
    unsigned s = 0x0C0C0C0Cu, k = 0;
#pragma unroll
    for (unsigned b = 0; b < 4; ++b)
        if (m & (1u << b)) { s = (s & ~(0xFFu << (8 * k))) | (b << (8 * k)); ++k; }
    return s;
}

#define STAMP(i) do { } while (0)

__global__ void __attribute__((amdgpu_flat_work_group_size(384, 384), amdgpu_waves_per_eu(2, 2))) ag_gemm(Params p) {
    extern __shared__ __attribute__((aligned(16))) unsigned char smem[];
    const int lane = threadIdx.x & 63;
    const int wave = uni(threadIdx.x >> 6);
    const int b = blockIdx.x;
    const int rt_ = b & 7;
    const int ct = (b >> 3) & 3;
    const int srcsel = b >> 5;
    const unsigned lds0 = (unsigned)(size_t)(LAS unsigned char*)smem;
    const unsigned ctl = lds0 + LDS_CTL;
    unsigned char* me = p.me;
    unsigned* tmo = (unsigned*)(me + TMO_OFF);
    unsigned* ctr = (unsigned*)(me + CTR_OFF + (b * 8 + wave) * 64);
    STAMP(0);
    const int g = lane >> 4, r = lane & 15;

    if (srcsel == 0) {
        const int rb = rt_ * 4 + ct;
        const unsigned char* xs = p.x + (size_t)(rb * 16 + r) * KDIM + 16 * g;
        if (wave == 5) {
            u32x4 d0[4];
#pragma unroll
            for (int i = 0; i < 4; ++i) d0[i] = *(const u32x4*)(xs + (i >> 1) * 128 + (i & 1) * 64);
            const unsigned call_v = ld_agent(ctr);
            unsigned plo = 0, phi = 0;
#pragma unroll
            for (int d = 0; d < NR - 1; ++d) {
                plo = lane == d ? (unsigned)(uintptr_t)p.peer[d] : plo;
                phi = lane == d ? (unsigned)((uintptr_t)p.peer[d] >> 32) : phi;
            }
            __builtin_amdgcn_s_barrier();
#pragma unroll
            for (int i = 0; i < 4; ++i) asm volatile("" : "+v"(d0[i]));
            const unsigned call = uni(call_v) + 1u;
            if (lane == 0) st_agent(ctr, call);
            STAMP(1);
            const size_t doff = (size_t)(call & 1u) * DATA_PAR + (size_t)p.rank * SLOT_BYTES + (size_t)rb * 65536 + lane * 16;
            unsigned char* dst[NR - 1];
            const int rot = rb % 7;
#pragma unroll
            for (int d = 0; d < NR - 1; ++d) {
                int q = d + rot; if (q >= 7) q -= 7;
                const uintptr_t a = (uintptr_t)(unsigned)__builtin_amdgcn_readlane(plo, q) | ((uintptr_t)(unsigned)__builtin_amdgcn_readlane(phi, q) << 32);
                dst[d] = (unsigned char*)a + doff;
            }
            u64* fl = (u64*)((((uintptr_t)phi << 32) | plo) + FLAG_OFF + (size_t)(p.rank * 32 + rb) * 128);
            const unsigned fbase = call * 64u;
            unsigned rawmask = 1u;
            const u64 tstart = rt();
            unsigned sent = 4096;
#pragma unroll
            for (int d = 0; d < NR - 1; ++d) {
#pragma unroll
                for (int i = 0; i < 4; ++i) st16_sys(dst[d] + i * 1024, d0[i]);
                if (4 * (d + 1) > WN) waitvm<WN>();
            }
            unsigned fbmask = 0;
            const long long fbdelta = (long long)(FB_OFF + (size_t)(call & 1u) * FB_PAR + (size_t)(p.rank * 32 + rb) * 8192 + lane * 16) - (long long)doff;
            for (int grp = 1; grp < NGRP; ++grp) {
                unsigned meta;
                if (!wait_lds(ctl + 4 * grp, 1u, meta, tmo, call)) return;
                if (grp == NFLAG) {
                    unsigned got;
                    for (int w = 0; w < 5; ++w)
                        if (!wait_lds(ctl + 4 * (16 + w), 1u, got, tmo, call)) return;
                }
                STAMP(8 + grp);
                const unsigned nl = meta & 0xffu;
                const bool fb = (meta & 0x200u) != 0u;
                rawmask |= (nl == 0u ? 1u : 0u) << grp;
                const u64 em = (nl == 0u || nl >= 64u) ? ~0ull : ((1ull << nl) - 1ull);
                const long long goff = fb ? fbdelta + (grp - NFLAG) * 4096 : (long long)grp * 4096;
                u32x4 v[4];
                lds_ld16x4(lds0 + grp * 4096 + lane * 16, v);
                if (PACE) {
                    const u64 due = tstart + (u64)(long long)(PACEOFF) + (((u64)sent * PACE) >> 16);
                    while (rt() < due) __builtin_amdgcn_s_sleep(1);
                    sent += nl ? 3072u + nl * 16u : 4096u;
                }
#pragma unroll
                for (int d = 0; d < NR - 1; ++d) {
#pragma unroll
                    for (int i = 0; i < 3; ++i) st16_sys(dst[d] + goff + i * 1024, v[i]);
                    st16_sys_masked(dst[d] + goff + 3072, v[3], em);
                    waitvm<WN>();
                    if (4 * (d + 1) == WN && grp <= NFLAG && lane < 7) st_sys64(fl, ((u64)(rawmask | fbmask) << 32) | (fbase + (unsigned)grp));
                }
                if (fb) {
                    fbmask |= 0x10000u << (grp - NFLAG);
                    waitvm<0>();
                    if (lane < 7) st_sys64(fl, ((u64)(rawmask | fbmask) << 32) | (fbase + (unsigned)grp + 1u));
                }
            }
            STAMP(2);
            return;
        }
        const unsigned lut = lds0 + LDS_SCR;
        const unsigned call_v = ld_agent(ctr);
        if (wave == 4) {
            if (lane < 32) lds_st(ctl + lane * 4, 0u);
            if (lane < 16) lds_st(lut + lane * 4, compact_sel(lane));
            asm volatile("s_waitcnt lgkmcnt(0)" ::: "memory");
        }
        __builtin_amdgcn_s_barrier();
        u32x4 rv[3][4];
#pragma unroll
        for (int t = 0; t < 3; ++t)
#pragma unroll
            for (int i = 0; i < 4; ++i) rv[t][i] = *(const u32x4*)(xs + (1 + wave + 5 * t) * 256 + (i >> 1) * 128 + (i & 1) * 64);
#pragma unroll
        for (int t = 0; t < 3; ++t) {
            const int grp = 1 + wave + 5 * t;
            unsigned R[16];
#pragma unroll
            for (int i = 0; i < 16; ++i) R[i] = rv[t][i >> 2][i & 3];
            unsigned fbf = 0;
            if (grp >= NFLAG) {
                bool hit = false;
#pragma unroll
                for (int i = 0; i < 16; ++i) hit = hit || R[i] == SMASK;
                if (__any(hit)) fbf = 0x200u;
                else {
#pragma unroll
                    for (int i = 0; i < 16; ++i) R[i] ^= SMASK;
                }
            }
            const unsigned nl = pack_group(R, lds0 + grp * 4096, lut, lane, !PACK || grp >= NFLAG);
            lds_st(ctl + 4 * grp, 0x100u | fbf | nl);
            if (t == 0) STAMP(1);
        }
        {
            const unsigned call = uni(call_v) + 1u;
            unsigned char* z = me + (size_t)((call + 1u) & 1u) * DATA_PAR + (size_t)rb * 65536 + NFLAG * 4096 + lane * 16;
            for (int i = wave; i < 56; i += 5)
                st16_sys(z + (size_t)((p.rank + 1 + (i >> 3)) & (NR - 1)) * SLOT_BYTES + (i & 7) * 1024, u32x4{0u, 0u, 0u, 0u});
            waitvm<0>();
            lds_st(ctl + 4 * (16 + wave), 1u);
        }
        if (lane == 0) st_agent(ctr, uni(call_v) + 1u);
        STAMP(2);
        return;
    }

    const bool has_local = srcsel <= 4;
    if (wave == 4) {
        const unsigned char* wsrc = p.w + (size_t)(lane >> 3) * NLOC + ct * 128;
        const unsigned c0 = (unsigned)(((lane & 7) ^ ((lane >> 4) & 3)) << 4);
        const unsigned c1 = c0 ^ 64u;
        if (lane < 32) lds_st(ctl + lane * 4, (lane == 5 && !has_local) ? 0x7fffffffu : 0u);
        asm volatile("s_waitcnt lgkmcnt(0)" ::: "memory");
#pragma unroll 4
        for (int i = 0; i < 32; ++i) glds16(wsrc + (size_t)i * 8 * NLOC + (((i >> 1) & 1) ? c1 : c0), lds0 + i * 1024);
        const unsigned call_v = ld_agent(ctr);
        __builtin_amdgcn_s_barrier();
        const unsigned call = uni(call_v) + 1u;
        if (lane == 0) st_agent(ctr, call);
        bool ok = true;
        for (int st = 1; st < NGRP; ++st) {
            if (st >= NS) {
                const unsigned need = (unsigned)(st - NS + 1);
                bool fr = true;
                for (int w = 0; w < 5; ++w) fr = fr && lds_ld(ctl + 4 + 4 * w) >= need;
                if (!fr) {
                    waitvm<0>();
                    lds_st(ctl, (unsigned)st);
                    unsigned got;
                    for (int w = 0; w < 5; ++w) ok = ok && wait_lds<LSLEEP>(ctl + 4 + 4 * w, need, got, tmo, call);
                }
            }
            const unsigned slot = lds0 + (unsigned)(st & (NS - 1)) * STAGE;
            const unsigned char* ws_ = wsrc + (size_t)st * 256 * NLOC;
#pragma unroll 4
            for (int i = 0; i < 32; ++i) glds16(ws_ + (size_t)i * 8 * NLOC + (((i >> 1) & 1) ? c1 : c0), slot + i * 1024);
            waitvm<32>();
            lds_st(ctl, (unsigned)st);
        }
        waitvm<0>();
        lds_st(ctl, (unsigned)NGRP);
        asm volatile("s_waitcnt lgkmcnt(0)" ::: "memory");
        STAMP(2);
        return;
    }

    const bool local = wave == 5;
    const unsigned call_v = ld_agent(ctr);
    const float scale = p.sx[0] * p.sw[0];
    __builtin_amdgcn_s_barrier();
    const unsigned call = uni(call_v) + 1u;
    if (lane == 0) st_agent(ctr, call);
    if (local && !has_local) return;
    STAMP(1);

    const int src = local ? p.rank : ((p.rank + srcsel) & (NR - 1));
    const int rb = local ? (srcsel - 1) * 8 + rt_ : rt_ * 4 + wave;
    const unsigned char* xbase = local ? p.x : me + (size_t)(call & 1u) * DATA_PAR + (size_t)src * SLOT_BYTES;
    const __amdgpu_buffer_rsrc_t xr = __builtin_amdgcn_make_buffer_rsrc((void*)xbase, 0, (int)SLOT_BYTES, 0x00020000);
    const int voff = local ? (rb * 16 + r) * KDIM + 16 * g : rb * 65536 + lane * 16;
    const int kstride = local ? 128 : 2048, hstride = local ? 64 : 1024;
    const u64* flag = (const u64*)(me + FLAG_OFF + (size_t)(src * 32 + rb) * 128);
    const unsigned fbase = call * 64u;
    const unsigned done_addr = ctl + 4 + 4 * (local ? 4 : wave);
    const unsigned scr = lds0 + LDS_SCR + (local ? 4 : wave) * SCR_BYTES;

    const unsigned q = (unsigned)(lane & 15) >> 1, pp = lane & 1;
    const unsigned swz = ((q >> 1) & 3u) | ((unsigned)(g & 1) << 2);
    const unsigned abase = lds0 + (16u * g + q) * 128u + 8u * pp + (swz << 4);

    v4f acc[8];
#pragma unroll
    for (int c = 0; c < 8; ++c) acc[c] = v4f{0.f, 0.f, 0.f, 0.f};

    bool ok = true;
    unsigned tmo_v = 0, got;
    auto load_w1 = [&](unsigned sb, int c) -> v8i {
        LAS unsigned char* a = (LAS unsigned char*)(size_t)(sb ^ (c << 4));
        v2i w0 = __builtin_amdgcn_ds_read_tr8_b64_v2i32((LAS v2i*)(a));
        v2i w1 = __builtin_amdgcn_ds_read_tr8_b64_v2i32((LAS v2i*)(a + 1024));
        v2i w2 = __builtin_amdgcn_ds_read_tr8_b64_v2i32((LAS v2i*)(a + 8192));
        v2i w3 = __builtin_amdgcn_ds_read_tr8_b64_v2i32((LAS v2i*)(a + 8192 + 1024));
        return v8i{w0[0], w0[1], w1[0], w1[1], w2[0], w2[1], w3[0], w3[1]};
    };
    v8i wf[4];
    ok = wait_lds(ctl, 1u, got, tmo, call);
#pragma unroll
    for (int c = 0; c < 4; ++c) wf[c] = load_w1(abase, c);

    unsigned flo = local ? ~0u : 0u, fhi = local ? ~0u : 0u;
    auto load_x = [&](u32x4* xv, int kg, bool have) {
        const int vo = have ? voff : 0x7ff00000;
#pragma unroll
        for (int i = 0; i < 4; ++i)
            xv[i] = __builtin_amdgcn_raw_buffer_load_b128(xr, vo, (2 * kg + (i >> 1)) * kstride + (i & 1) * hstride, 17);
    };
    auto decode = [&](const u32x4* xv, bool raw, v8i& x0, v8i& x1) {
        if (raw) {
            x0 = v8i{(int)xv[0][0], (int)xv[0][1], (int)xv[0][2], (int)xv[0][3], (int)xv[1][0], (int)xv[1][1], (int)xv[1][2], (int)xv[1][3]};
            x1 = v8i{(int)xv[2][0], (int)xv[2][1], (int)xv[2][2], (int)xv[2][3], (int)xv[3][0], (int)xv[3][1], (int)xv[3][2], (int)xv[3][3]};
        } else {
            unpack_group(xv, scr, lane, x0, x1);
        }
    };
    auto mma = [&](const v8i x0, const v8i x1, int kg) {
        const unsigned sb = abase + (unsigned)(kg & (NS - 1)) * STAGE;
#pragma unroll
        for (int c = 0; c < 8; ++c) {
            const v8i cur = wf[c & 3];
            acc[c] = __builtin_amdgcn_mfma_scale_f32_16x16x128_f8f6f4(cur, x0, acc[c], 0, 0, 0, 0, 0, 0);
            wf[c & 3] = c < 4 ? load_w1(sb, c + 4) : load_w1(sb + 16384, c - 4);
            __builtin_amdgcn_sched_group_barrier(0x008, 1, 0);
            __builtin_amdgcn_sched_group_barrier(0x100, 4, 0);
        }
#pragma unroll
        for (int c = 0; c < 4; ++c) {
            const v8i cur = wf[c];
            acc[c] = __builtin_amdgcn_mfma_scale_f32_16x16x128_f8f6f4(cur, x1, acc[c], 0, 0, 0, 0, 0, 0);
            wf[c] = load_w1(sb + 16384, c + 4);
            __builtin_amdgcn_sched_group_barrier(0x008, 1, 1);
            __builtin_amdgcn_sched_group_barrier(0x100, 4, 1);
        }
        lds_st(done_addr, (unsigned)(kg + 1));
        const int kn = kg + 1 < NGRP ? kg + 1 : kg;
        if (kg + 1 < NGRP) ok = ok && wait_lds<4>(ctl, (unsigned)(kn + 1), got, tmo, call);
        const unsigned sn = abase + (unsigned)(kn & (NS - 1)) * STAGE;
#pragma unroll
        for (int c = 4; c < 8; ++c) {
            const v8i cur = wf[c & 3];
            acc[c] = __builtin_amdgcn_mfma_scale_f32_16x16x128_f8f6f4(cur, x1, acc[c], 0, 0, 0, 0, 0, 0);
            wf[c & 3] = load_w1(sn, c - 4);
            __builtin_amdgcn_sched_group_barrier(0x008, 1, 2);
            __builtin_amdgcn_sched_group_barrier(0x100, 4, 2);
        }
    };
    auto compute = [&](const u32x4* xv, int kg, bool raw) {
        v8i x0, x1;
        decode(xv, raw, x0, x1);
        mma(x0, x1, kg);
    };
    auto await_direct = [&](unsigned need) {
        const u64 t0 = rt();
        for (unsigned spins = 1;; ++spins) {
            const u64 v = ld_sys64(flag);
            flo = uni((unsigned)v); fhi = uni((unsigned)(v >> 32));
            if (flo >= need) return;
            __builtin_amdgcn_s_sleep(1);
            if ((spins & 15u) == 0u) {
                if (uni(ld_agent(tmo)) == call || rt() - t0 > DEADLINE) { st_agent(tmo, call); ok = false; return; }
            }
        }
    };
    auto step = [&](const u32x4* cur, u32x4* nxt, int kg) {
        const unsigned need = fbase + (unsigned)kg + 2u;
        const bool raw = ((fhi >> kg) & 1u) || !ok;
        const bool early = ok && flo >= need;
        load_x(nxt, kg + 1, early);
        v8i x0, x1;
        decode(cur, raw, x0, x1);
        const u64 fv = ld_sys64(flag);
        mma(x0, x1, kg);
        if (!early) {
            if (!local) {
                flo = uni((unsigned)fv); fhi = uni((unsigned)(fv >> 32));
                if (flo < need && ok) await_direct(need);
            }
            asm volatile("" ::: "memory");
            load_x(nxt, kg + 1, ok);
        } else if (!local) {
            const unsigned l2 = uni((unsigned)fv);
            if (l2 > flo) { flo = l2; fhi = uni((unsigned)(fv >> 32)); }
        }
    };
    auto settle = [&](u32x4* xv, int kg) {
        if (local || !ok) return;
        const u64 t0 = rt();
        for (unsigned spins = 1;; ++spins) {
            unsigned m = ~0u;
#pragma unroll
            for (int i = 0; i < 4; ++i) m = min(min(m, min(xv[i][0], xv[i][1])), min(xv[i][2], xv[i][3]));
            if (!__any(m == 0u)) {
#pragma unroll
                for (int i = 0; i < 4; ++i) xv[i] ^= SMASK;
                return;
            }
            load_x(xv, kg, true);
            const u64 fv = ld_sys64(flag);
            const unsigned lo = uni((unsigned)fv), hi = uni((unsigned)(fv >> 32));
            if (lo >= fbase + (unsigned)kg + 1u && ((hi >> (16 + kg - NFLAG)) & 1u)) {
                asm volatile("" ::: "memory");
                const __amdgpu_buffer_rsrc_t fr = __builtin_amdgcn_make_buffer_rsrc(
                    (void*)(me + FB_OFF + (size_t)(call & 1u) * FB_PAR + (size_t)(src * 32 + rb) * 8192 + (kg - NFLAG) * 4096), 0, 4096, 0x00020000);
#pragma unroll
                for (int i = 0; i < 4; ++i) xv[i] = __builtin_amdgcn_raw_buffer_load_b128(fr, lane * 16, i * 1024, 17);
                return;
            }
            if ((spins & 7u) == 0u) {
                if (uni(ld_agent(tmo)) == call || rt() - t0 > DEADLINE) { st_agent(tmo, call); ok = false; return; }
            }
        }
    };
    u32x4 xa[4], xb[4];
    if (!local) await_direct(fbase + 1u);
    asm volatile("" ::: "memory");
    load_x(xa, 0, ok);
    for (int kg = 0; kg < NFLAG - 2; kg += 2) {
        step(xa, xb, kg);
        step(xb, xa, kg + 1);
    }
    step(xa, xb, NFLAG - 2);
    load_x(xa, NFLAG, ok);
    STAMP(8 + NFLAG - 1);
    compute(xb, NFLAG - 1, ((fhi >> (NFLAG - 1)) & 1u) || !ok);
    settle(xa, NFLAG);
    load_x(xb, NFLAG + 1, ok);
    STAMP(8 + NFLAG);
    compute(xa, NFLAG, true);
    STAMP(8 + NFLAG + 1);
    settle(xb, NFLAG + 1);
    STAMP(2);
    tmo_v = ld_agent(tmo);
    compute(xb, NFLAG + 1, true);
    STAMP(3);

    const float sc = uni(tmo_v) == call ? __builtin_nanf("") : scale;
    float* o = p.out + (size_t)(src * ROWS + rb * 16 + r) * NLOC + ct * 128 + 4 * g;
#pragma unroll
    for (int c = 0; c < 8; ++c) {
        const v4f y = __builtin_elementwise_maximum(acc[c] * sc, v4f{0.f, 0.f, 0.f, 0.f});
        __builtin_nontemporal_store(y, (v4f*)(o + 16 * c));
    }
    STAMP(4);
}

}

extern "C" void kernel_launch(int rank, int nranks, ncclComm_t comm, hipStream_t stream,
                              void* const* d_in, const int* in_sizes, int n_in,
                              void* d_out, int out_size,
                              const int* in_dtypes, int out_dtype,
                              void* const* d_out_all,
                              void* const* d_ws_all, size_t ws_bytes) {
    (void)comm; (void)in_dtypes; (void)out_dtype; (void)d_out_all;
    if (nranks != NR || (unsigned)rank >= (unsigned)NR || n_in != 4 || ws_bytes < WS_NEED ||
        in_sizes[0] != ROWS * KDIM || in_sizes[1] != KDIM * NLOC || out_size != NR * ROWS * NLOC) {
        fprintf(stderr, "kernel_launch: unexpected problem shape\n");
        return;
    }
    static bool init = false;
    if (!init) {
        HIP_CHECK(hipFuncSetAttribute((const void*)ag_gemm, hipFuncAttributeMaxDynamicSharedMemorySize, LDS_BYTES));
        init = true;
    }
    Params p;
    p.x = (const unsigned char*)d_in[0];
    p.w = (const unsigned char*)d_in[1];
    p.sx = (const float*)d_in[2];
    p.sw = (const float*)d_in[3];
    p.out = (float*)d_out;
    p.me = (unsigned char*)d_ws_all[rank];
    for (int d = 0; d < NR - 1; ++d) p.peer[d] = (unsigned char*)d_ws_all[(rank + 1 + d) % NR];
    p.rank = rank;
    p.pad = 0;
    hipLaunchKernelGGL(ag_gemm, dim3(256), dim3(384), LDS_BYTES, stream, p);
    HIP_CHECK(hipGetLastError());
}
```

```cpp
#include <hip/hip_runtime.h>
#include <rccl/rccl.h>
#include <cstdio>
#include <cstdint>

#define HIP_CHECK(expr) do { hipError_t _e = (expr); if (_e != hipSuccess) { \
    fprintf(stderr, "HIP error %s at %s:%d\n", hipGetErrorString(_e), __FILE__, __LINE__); \
    return; } } while (0)

#define PACK 1
#define FSLEEP 4
#define FASTLAST 0
#define LATE 10
#define WBIG 56
#define WLAST 8
#define TWOTRY 1
#define TRYGAP 10
#define OUTAUX 17
#define HEADOPT 1
#define LSLEEP 12
#define PACE 3750
#define PACEOFF 0
#define LAS __attribute__((address_space(3)))
typedef unsigned u32x4 __attribute__((ext_vector_type(4)));
typedef unsigned long long u64;
typedef int v8i __attribute__((ext_vector_type(8)));
typedef int v2i __attribute__((ext_vector_type(2)));
typedef float v4f __attribute__((ext_vector_type(4)));

namespace {

constexpr int NR = 8;
constexpr int KDIM = 4096;
constexpr int NLOC = 512;
constexpr int ROWS = 512;
constexpr unsigned SLOT_BYTES = ROWS * KDIM;
constexpr size_t DATA_PAR = (size_t)NR * SLOT_BYTES;
constexpr size_t FB_OFF = 2 * DATA_PAR;
constexpr size_t FB_PAR = (size_t)NR * 32 * 8192;
constexpr size_t FLAG_OFF = FB_OFF + 2 * FB_PAR;
constexpr size_t FLAG_PAR = (size_t)NR * 32 * 128;
constexpr size_t CTR_OFF = FLAG_OFF + 64 * 1024;
constexpr size_t TMO_OFF = CTR_OFF + 256 * 8 * 64;
constexpr size_t DBG_OFF = TMO_OFF + 4096;
constexpr size_t WS_NEED = DBG_OFF + 256 * 8 * 32 * 8;
constexpr u64 DEADLINE = 30000000ull;

constexpr int NGRP = 16;
constexpr int NFLAG = NGRP - 2;
#define SMASK 0x7F7F7F7Fu
constexpr int NS = 4;
constexpr int STAGE = 256 * 128;
constexpr int LDS_CTL = NS * STAGE;
constexpr int LDS_SCR = LDS_CTL + 256;
constexpr int SCR_BYTES = 1024 + 64;
constexpr int LDS_BYTES = LDS_SCR + 5 * SCR_BYTES;
#define WNUM 24
constexpr int WN = WNUM;

struct Params {
    const unsigned char* x;
    const unsigned char* w;
    const float* sx;
    const float* sw;
    float* out;
    unsigned char* me;
    unsigned char* peer[NR - 1];
    int rank;
    int pad;
};

__device__ __forceinline__ u64 rt() { return __builtin_amdgcn_s_memrealtime(); }
__device__ __forceinline__ unsigned uni(unsigned v) { return __builtin_amdgcn_readfirstlane(v); }
template <int N> __device__ __forceinline__ void waitvm() { asm volatile("s_waitcnt vmcnt(%0)" :: "i"(N) : "memory"); }
__device__ __forceinline__ void st16_sys(void* p, u32x4 v) {
    asm volatile("global_store_dwordx4 %0, %1, off sc0 sc1\n\ts_nop 1" :: "v"(p), "v"(v) : "memory");
}
__device__ __forceinline__ void st16_sys_masked(void* p, u32x4 v, u64 mask) {
    u64 keep;
    asm volatile("s_mov_b64 %0, exec\n\ts_mov_b64 exec, %3\n\tglobal_store_dwordx4 %1, %2, off sc0 sc1\n\ts_mov_b64 exec, %0\n\ts_nop 1"
                 : "=&s"(keep) : "v"(p), "v"(v), "s"(mask) : "memory");
}
__device__ __forceinline__ void glds16(const void* gsrc, unsigned lds_dst) {
    unsigned keep;
    asm volatile("s_mov_b32 %0, m0\n\ts_mov_b32 m0, %2\n\ts_nop 0\n\tglobal_load_lds_dwordx4 %1, off\n\ts_mov_b32 m0, %0"
                 : "=&s"(keep) : "v"(gsrc), "s"(lds_dst) : "memory");
}
__device__ __forceinline__ void glds16_sys(const void* gsrc, unsigned lds_dst) {
    unsigned keep;
    asm volatile("s_mov_b32 %0, m0\n\ts_mov_b32 m0, %2\n\ts_nop 0\n\tglobal_load_lds_dwordx4 %1, off sc0 sc1\n\ts_mov_b32 m0, %0"
                 : "=&s"(keep) : "v"(gsrc), "s"(lds_dst) : "memory");
}
__device__ __forceinline__ void lds_st(unsigned addr, unsigned v) {
    asm volatile("ds_write_b32 %0, %1" :: "v"(addr), "v"(v) : "memory");
}
__device__ __forceinline__ void lds_st16(unsigned addr, u32x4 v) {
    asm volatile("ds_write_b128 %0, %1" :: "v"(addr), "v"(v) : "memory");
}
__device__ __forceinline__ void lds_or(unsigned addr, unsigned v) {
    asm volatile("ds_or_b32 %0, %1" :: "v"(addr), "v"(v) : "memory");
}
__device__ __forceinline__ unsigned lds_ldv(unsigned addr) {
    unsigned v;
    asm volatile("ds_read_b32 %0, %1\n\ts_waitcnt lgkmcnt(0)" : "=v"(v) : "v"(addr) : "memory");
    return v;
}
__device__ __forceinline__ unsigned lds_ld(unsigned addr) { return uni(lds_ldv(addr)); }
__device__ __forceinline__ void lds_ld16x4(unsigned addr, u32x4* v) {
    asm volatile("ds_read_b128 %0, %4\n\tds_read_b128 %1, %4 offset:1024\n\tds_read_b128 %2, %4 offset:2048\n\t"
                 "ds_read_b128 %3, %4 offset:3072\n\ts_waitcnt lgkmcnt(0)"
                 : "=&v"(v[0]), "=&v"(v[1]), "=&v"(v[2]), "=&v"(v[3]) : "v"(addr) : "memory");
}
__device__ __forceinline__ unsigned ld_agent(const unsigned* p) { return __hip_atomic_load(p, __ATOMIC_RELAXED, __HIP_MEMORY_SCOPE_AGENT); }
__device__ __forceinline__ void st_agent(unsigned* p, unsigned v) { __hip_atomic_store(p, v, __ATOMIC_RELAXED, __HIP_MEMORY_SCOPE_AGENT); }
__device__ __forceinline__ u64 ld_sys64(const u64* p) { return __hip_atomic_load(p, __ATOMIC_RELAXED, __HIP_MEMORY_SCOPE_SYSTEM); }
__device__ __forceinline__ void st_sys64(u64* p, u64 v) { __hip_atomic_store(p, v, __ATOMIC_RELAXED, __HIP_MEMORY_SCOPE_SYSTEM); }

template <int SLEEP = 1>
__device__ __forceinline__ bool wait_lds(unsigned addr, unsigned want, unsigned& got, unsigned* tmo, unsigned call) {
    got = lds_ld(addr);
    if (got >= want) return true;
    const u64 t0 = rt();
    for (unsigned spins = 1;; ++spins) {
        __builtin_amdgcn_s_sleep(SLEEP);
        got = lds_ld(addr);
        if (got >= want) return true;
        if ((spins & 255u) == 0u) {
            if (uni(ld_agent(tmo)) == call || rt() - t0 > DEADLINE) { st_agent(tmo, call); return false; }
        }
    }
}

__device__ __forceinline__ unsigned perm(unsigned hi, unsigned lo, unsigned sel) { return __builtin_amdgcn_perm(hi, lo, sel); }
__device__ __forceinline__ unsigned bfi(unsigned m, unsigned a, unsigned b) { return (m & a) | (~m & b); }
__device__ __forceinline__ unsigned wave_scan(unsigned v) {
    v += __builtin_amdgcn_update_dpp(0u, v, 0x111, 0xf, 0xf, true);
    v += __builtin_amdgcn_update_dpp(0u, v, 0x112, 0xf, 0xf, true);
    v += __builtin_amdgcn_update_dpp(0u, v, 0x114, 0xf, 0xf, true);
    v += __builtin_amdgcn_update_dpp(0u, v, 0x118, 0xf, 0xf, true);
    v += __builtin_amdgcn_update_dpp(0u, v, 0x142, 0xa, 0xf, false);
    v += __builtin_amdgcn_update_dpp(0u, v, 0x143, 0xc, 0xf, false);
    return v;
}

__device__ __forceinline__ unsigned pack_group(const unsigned* R, unsigned slot, unsigned lut, int lane, bool force_raw) {
    unsigned nl = 0;
    if (!force_raw) {
        unsigned s = 0;
#pragma unroll
        for (int i = 0; i < 16; i += 5) s = __builtin_amdgcn_udot4((R[i] >> 3) & 0x0F0F0F0Fu, 0x01010101u, s, false);
        const unsigned tot = (unsigned)__builtin_amdgcn_readlane((int)wave_scan(s), 63);
        const unsigned a = tot > 102u ? (tot - 102u) >> 10 : 0u;
        const unsigned K = ((5u - a) & 15u) * 0x01010101u;
        unsigned bad = 0, N[8], P[4], n[2];
        u64 acc[2];
#pragma unroll
        for (int j = 0; j < 2; ++j) {
            u64 ac = 0;
            unsigned pos = 0;
#pragma unroll
            for (int q = 0; q < 2; ++q) {
                unsigned pw = 0;
#pragma unroll
                for (int i = 0; i < 4; ++i) {
                    const unsigned r = R[8 * j + 4 * q + i];
                    const unsigned D = (((r >> 3) & 0x0F0F0F0Fu) + K) & 0x0F0F0F0Fu;
                    bad |= (D + 0x05050505u) & 0x10101010u;
                    const unsigned C = perm(0x820180FFu, 0xFFFFFFFFu, D) & 0x03030303u;
                    const unsigned f = perm(0u, 0x01000000u, C);
                    const unsigned t = (D >> 3) & 0x01010101u;
                    const unsigned S = D - (t + (t << 1));
                    const unsigned m = __builtin_amdgcn_udot4(f, 0x08040201u, 0u, false);
                    const unsigned sel = lds_ldv(lut + m * 4);
                    const unsigned c4 = perm(0u, S, sel);
                    const unsigned w = __builtin_amdgcn_udot4(c4, 0x00400801u, (c4 >> 24) << 9, false);
                    ac |= (u64)w << (pos & 63u);
                    pos += 3u * (unsigned)__builtin_popcount(f);
                    pw |= C << (2 * i);
                }
                P[2 * j + q] = pw;
            }
#pragma unroll
            for (int m2 = 0; m2 < 4; ++m2) {
                const unsigned ra = R[8 * j + 2 * m2], rb = R[8 * j + 2 * m2 + 1];
                N[4 * j + m2] = bfi(0x0F0F0F0Fu, bfi(0x08080808u, ra >> 4, ra), bfi(0x70707070u, rb << 4, rb));
            }
            acc[j] = ac; n[j] = pos;
        }
        const unsigned cnt = n[0] | (n[1] << 16);
        const unsigned inc = wave_scan(cnt);
        const unsigned tot2 = (unsigned)__builtin_amdgcn_readlane((int)inc, 63);
        const unsigned t0 = tot2 & 0xffffu, t1 = tot2 >> 16;
        const bool lane_bad = bad != 0u || n[0] > 63u || n[1] > 63u;
        const unsigned bits = 32u + t0 + t1;
        if (!__any(lane_bad) && bits <= 8192u) {
            const unsigned exc = inc - cnt;
            lds_st16(slot + lane * 16, u32x4{N[0], N[1], N[2], N[3]});
            lds_st16(slot + 1024 + lane * 16, u32x4{N[4], N[5], N[6], N[7]});
            lds_st16(slot + 2048 + lane * 16, u32x4{P[0], P[1], P[2], P[3]});
            lds_st16(slot + 3072 + lane * 16, u32x4{lane == 0 ? a : 0u, 0u, 0u, 0u});
#pragma unroll
            for (int j = 0; j < 2; ++j) {
                const unsigned off = 32u + (j ? t0 + (exc >> 16) : (exc & 0xffffu));
                const unsigned sh = off & 31u, ad = slot + 3072 + (off >> 5) * 4;
                const u64 lo = acc[j] << sh;
                const unsigned hi = (unsigned)((acc[j] >> 32) >> (32u - sh));
                lds_or(ad, (unsigned)lo);
                lds_or(ad + 4, (unsigned)(lo >> 32));
                if ((off >> 5) + 2 < 256u) lds_or(ad + 8, hi);
            }
            nl = ((bits + 511u) >> 9) * 4u;
        }
    }
    if (nl == 0) {
#pragma unroll
        for (int i = 0; i < 4; ++i) lds_st16(slot + i * 1024 + lane * 16, u32x4{R[4 * i], R[4 * i + 1], R[4 * i + 2], R[4 * i + 3]});
    }
    return nl;
}

__device__ __forceinline__ unsigned opaque(unsigned k) { asm("" : "+s"(k)); return k; }
__device__ __forceinline__ unsigned sel3(unsigned m, unsigned a, unsigned b) { return __builtin_amdgcn_bitop3_b32(m, a, b, 0xCA); }
__device__ __forceinline__ void unpack_group(const u32x4* v, unsigned scr, int lane, v8i& x0, v8i& x1) {
    lds_st16(scr + lane * 16, v[3]);
    const unsigned a = (unsigned)__builtin_amdgcn_readlane((int)v[3][0], 0) & 15u;
    const unsigned TP = (((a * 0x01010101u) + 0x00020100u) & 0x0F0F0F0Fu) << 3;
    const unsigned bs = ((a - 5u) & 15u) * 0x01010101u;
    const unsigned TL = ((bs + 0x03020100u) & 0x0F0F0F0Fu) << 3;
    const unsigned TH = ((bs + 0x0A090804u) & 0x0F0F0F0Fu) << 3;
    const unsigned M03 = opaque(0x03030303u), M01 = opaque(0x01010101u), M07 = opaque(0x07070707u), K3 = 0x010101u;
    const unsigned MFF = opaque(0xFF000000u), M16 = opaque(0x0000FFFFu), M8 = opaque(0x00FF00FFu), M80 = opaque(0x80808080u), M78 = opaque(0x78787878u);
    unsigned C[16], FM[16], IX[16], nb[2][9];
#pragma unroll
    for (int j = 0; j < 2; ++j) {
        nb[j][0] = 0;
#pragma unroll
        for (int i = 0; i < 8; ++i) {
            const unsigned c = (v[2][2 * j + (i >> 2)] >> (2 * (i & 3))) & M03;
            const unsigned fm = perm(0u, MFF, c);
            const unsigned f1 = fm & M01;
            C[8 * j + i] = c; FM[8 * j + i] = fm;
            IX[8 * j + i] = perm(__umul24(f1, K3), 0u, 0x0605040Cu);
            nb[j][i + 1] = __builtin_amdgcn_udot4(f1, M03, nb[j][i], false);
        }
    }
    const unsigned cnt = nb[0][8] | (nb[1][8] << 16);
    const unsigned inc = wave_scan(cnt);
    const unsigned t0 = (unsigned)__builtin_amdgcn_readlane((int)inc, 63) & 0xffffu;
    const unsigned exc = inc - cnt;
    u64 win[2];
    {
        const unsigned off0 = 32u + (exc & 0xffffu), off1 = 32u + t0 + (exc >> 16);
        unsigned d2, e2;
        asm volatile("ds_read2_b32 %0, %4 offset1:1\n\tds_read_b32 %1, %4 offset:8\n\t"
                     "ds_read2_b32 %2, %5 offset1:1\n\tds_read_b32 %3, %5 offset:8\n\ts_waitcnt lgkmcnt(0)"
                     : "=&v"(win[0]), "=&v"(d2), "=&v"(win[1]), "=&v"(e2) : "v"(scr + (off0 >> 5) * 4), "v"(scr + (off1 >> 5) * 4) : "memory");
        const unsigned d0 = (unsigned)win[0], d1 = (unsigned)(win[0] >> 32), e0 = (unsigned)win[1], e1 = (unsigned)(win[1] >> 32);
        win[0] = (u64)__builtin_amdgcn_alignbit(d1, d0, off0) | ((u64)__builtin_amdgcn_alignbit(d2, d1, off0) << 32);
        win[1] = (u64)__builtin_amdgcn_alignbit(e1, e0, off1) | ((u64)__builtin_amdgcn_alignbit(e2, e1, off1) << 32);
    }
    unsigned out[16];
#pragma unroll
    for (int j = 0; j < 2; ++j) {
#pragma unroll
        for (int i = 0; i < 8; ++i) {
            const unsigned w = (unsigned)(win[j] >> nb[j][i]);
            const unsigned u = sel3(M16, w, w << 10);
            const unsigned sp = sel3(M8, u, u << 5) & M07;
            const unsigned es = perm(TH, TL, perm(0u, sp, IX[8 * j + i]));
            const unsigned e = sel3(FM[8 * j + i], es, perm(0u, TP, C[8 * j + i]));
            const unsigned n = v[j][i >> 1];
            const unsigned sm = (i & 1) ? sel3(M80, n, n >> 4) : sel3(M80, n << 4, n);
            out[8 * j + i] = sel3(M78, e, sm);
        }
    }
    x0 = v8i{(int)out[0], (int)out[1], (int)out[2], (int)out[3], (int)out[4], (int)out[5], (int)out[6], (int)out[7]};
    x1 = v8i{(int)out[8], (int)out[9], (int)out[10], (int)out[11], (int)out[12], (int)out[13], (int)out[14], (int)out[15]};
}
__device__ __forceinline__ unsigned compact_sel(unsigned m) {
    unsigned s = 0x0C0C0C0Cu, k = 0;
#pragma unroll
    for (unsigned b = 0; b < 4; ++b)
        if (m & (1u << b)) { s = (s & ~(0xFFu << (8 * k))) | (b << (8 * k)); ++k; }
    return s;
}

#define STAMP(i) do { } while (0)

__global__ void __attribute__((amdgpu_flat_work_group_size(384, 384), amdgpu_waves_per_eu(2, 2))) ag_gemm(Params p) {
    extern __shared__ __attribute__((aligned(16))) unsigned char smem[];
    const int lane = threadIdx.x & 63;
    const int wave = uni(threadIdx.x >> 6);
    const int b = blockIdx.x;
    const int rt_ = b & 7;
    const int ct = (b >> 3) & 3;
    const int srcsel = b >> 5;
    const unsigned lds0 = (unsigned)(size_t)(LAS unsigned char*)smem;
    const unsigned ctl = lds0 + LDS_CTL;
    unsigned char* me = p.me;
    unsigned* tmo = (unsigned*)(me + TMO_OFF);
    unsigned* ctr = (unsigned*)(me + CTR_OFF + (b * 8 + wave) * 64);
    STAMP(0);
    const int g = lane >> 4, r = lane & 15;
    asm volatile("" :: "s"(p.x), "s"(p.me), "s"(p.peer[0]), "s"(p.peer[1]), "s"(p.peer[2]), "s"(p.peer[3]), "s"(p.peer[4]), "s"(p.peer[5]), "s"(p.peer[6]), "s"(p.rank));

    if (__builtin_expect(srcsel == 0, HEADOPT)) {
        const int rb = rt_ * 4 + ct;
        const unsigned char* xs = p.x + (size_t)(rb * 16 + r) * KDIM + 16 * g;
        if (__builtin_expect(wave == 5, HEADOPT)) {
            u32x4 d0[4];
#pragma unroll
            for (int i = 0; i < 4; ++i) d0[i] = *(const u32x4*)(xs + (i >> 1) * 128 + (i & 1) * 64);
            const unsigned call_v = ld_agent(ctr);
            unsigned plo = 0, phi = 0;
#pragma unroll
            for (int d = 0; d < NR - 1; ++d) {
                plo = lane == d ? (unsigned)(uintptr_t)p.peer[d] : plo;
                phi = lane == d ? (unsigned)((uintptr_t)p.peer[d] >> 32) : phi;
            }
            __builtin_amdgcn_s_barrier();
#pragma unroll
            for (int i = 0; i < 4; ++i) asm volatile("" : "+v"(d0[i]));
            const unsigned call = uni(call_v) + 1u;
            if (lane == 0) st_agent(ctr, call);
            STAMP(1);
            const size_t doff = (size_t)(call & 1u) * DATA_PAR + (size_t)p.rank * SLOT_BYTES + (size_t)rb * 65536 + lane * 16;
            unsigned char* dst[NR - 1];
            const int rot = rb % 7;
#pragma unroll
            for (int d = 0; d < NR - 1; ++d) {
                int q = d + rot; if (q >= 7) q -= 7;
                const uintptr_t a = (uintptr_t)(unsigned)__builtin_amdgcn_readlane(plo, q) | ((uintptr_t)(unsigned)__builtin_amdgcn_readlane(phi, q) << 32);
                dst[d] = (unsigned char*)a + doff;
            }
            u64* fl = (u64*)((((uintptr_t)phi << 32) | plo) + FLAG_OFF + (size_t)(call & 1u) * FLAG_PAR + (size_t)(p.rank * 32 + rb) * 128);
            const unsigned fbase = call * 64u;
            unsigned rawmask = 1u;
            const u64 tstart = rt();
            unsigned sent = 4096;
#pragma unroll
            for (int d = 0; d < NR - 1; ++d) {
#pragma unroll
                for (int i = 0; i < 4; ++i) st16_sys(dst[d] + i * 1024, d0[i]);
                if (4 * (d + 1) > WN) waitvm<WN>();
            }
            unsigned fbmask = 0;
            const long long fbdelta = (long long)(FB_OFF + (size_t)(call & 1u) * FB_PAR + (size_t)(p.rank * 32 + rb) * 8192 + lane * 16) - (long long)doff;
            for (int grp = 1; grp < NGRP; ++grp) {
                unsigned meta;
                if (!wait_lds(ctl + 4 * grp, 1u, meta, tmo, call)) return;
                if (grp == NFLAG) {
                    unsigned got;
                    for (int w = 0; w < 5; ++w)
                        if (!wait_lds(ctl + 4 * (16 + w), 1u, got, tmo, call)) return;
                }
                STAMP(8 + grp);
                const unsigned nl = meta & 0xffu;
                const bool fb = (meta & 0x200u) != 0u;
                rawmask |= (nl == 0u ? 1u : 0u) << grp;
                const u64 em = (nl == 0u || nl >= 64u) ? ~0ull : ((1ull << nl) - 1ull);
                const long long goff = fb ? fbdelta + (grp - NFLAG) * 4096 : (long long)grp * 4096;
                u32x4 v[4];
                lds_ld16x4(lds0 + grp * 4096 + lane * 16, v);
                bool late = false;
                if (PACE) {
                    const u64 due = tstart + (u64)(long long)(PACEOFF) + (((u64)sent * PACE) >> 16);
                    u64 now = rt();
                    late = now > due + LATE;
                    while (now < due) { __builtin_amdgcn_s_sleep(1); now = rt(); }
                    sent += nl ? 3072u + nl * 16u : 4096u;
                }
#pragma unroll
                for (int d = 0; d < NR - 1; ++d) {
#pragma unroll
                    for (int i = 0; i < 3; ++i) st16_sys(dst[d] + goff + i * 1024, v[i]);
                    st16_sys_masked(dst[d] + goff + 3072, v[3], em);
                    if (4 * (d + 1) == WLAST && grp == NFLAG) {
                        waitvm<WLAST>();
                        if (lane < 7) st_sys64(fl, ((u64)(rawmask | fbmask) << 32) | (fbase + (unsigned)grp));
                    } else if (4 * (d + 1) == WN) {
                        waitvm<WN>();
                        if (grp < NFLAG && lane < 7) st_sys64(fl, ((u64)(rawmask | fbmask) << 32) | (fbase + (unsigned)grp));
                    } else if (late) {
                        waitvm<WBIG>();
                    } else {
                        waitvm<WN>();
                    }
                }
                if (fb) {
                    fbmask |= 0x10000u << (grp - NFLAG);
                    waitvm<0>();
                    if (lane < 7) st_sys64(fl, ((u64)(rawmask | fbmask) << 32) | (fbase + (unsigned)grp + 1u));
                }
            }
            STAMP(2);
            return;
        }
        const unsigned lut = lds0 + LDS_SCR;
        const unsigned call_v = ld_agent(ctr);
        if (wave == 4) {
            if (lane < 32) lds_st(ctl + lane * 4, 0u);
            if (lane < 16) lds_st(lut + lane * 4, compact_sel(lane));
            asm volatile("s_waitcnt lgkmcnt(0)" ::: "memory");
        }
        __builtin_amdgcn_s_barrier();
        u32x4 rv[3][4];
#pragma unroll
        for (int t = 0; t < 3; ++t)
#pragma unroll
            for (int i = 0; i < 4; ++i) rv[t][i] = *(const u32x4*)(xs + (1 + wave + 5 * t) * 256 + (i >> 1) * 128 + (i & 1) * 64);
#pragma unroll
        for (int t = 0; t < 3; ++t) {
            const int grp = 1 + wave + 5 * t;
            unsigned R[16];
#pragma unroll
            for (int i = 0; i < 16; ++i) R[i] = rv[t][i >> 2][i & 3];
            unsigned fbf = 0;
            if (grp >= NFLAG) {
                bool hit = false;
#pragma unroll
                for (int i = 0; i < 16; ++i) hit = hit || R[i] == SMASK;
                if (__any(hit)) fbf = 0x200u;
                else {
#pragma unroll
                    for (int i = 0; i < 16; ++i) R[i] ^= SMASK;
                }
            }
            const unsigned nl = pack_group(R, lds0 + grp * 4096, lut, lane, !PACK || grp >= NFLAG);
            lds_st(ctl + 4 * grp, 0x100u | fbf | nl);
            if (t == 0) STAMP(1);
        }
        {
            const unsigned call = uni(call_v) + 1u;
            unsigned char* z = me + (size_t)((call + 1u) & 1u) * DATA_PAR + (size_t)rb * 65536 + NFLAG * 4096 + lane * 16;
            for (int i = wave; i < 56; i += 5)
                st16_sys(z + (size_t)((p.rank + 1 + (i >> 3)) & (NR - 1)) * SLOT_BYTES + (i & 7) * 1024, u32x4{0u, 0u, 0u, 0u});
            waitvm<0>();
            lds_st(ctl + 4 * (16 + wave), 1u);
        }
        if (lane == 0) st_agent(ctr, uni(call_v) + 1u);
        STAMP(2);
        return;
    }

    const bool has_local = srcsel <= 4;
    if (wave == 4) {
        const unsigned char* wsrc = p.w + (size_t)(lane >> 3) * NLOC + ct * 128;
        const unsigned c0 = (unsigned)(((lane & 7) ^ ((lane >> 4) & 3)) << 4);
        const unsigned c1 = c0 ^ 64u;
        if (lane < 32) lds_st(ctl + lane * 4, (lane == 5 && !has_local) ? 0x7fffffffu : 0u);
        asm volatile("s_waitcnt lgkmcnt(0)" ::: "memory");
#pragma unroll 4
        for (int i = 0; i < 32; ++i) glds16(wsrc + (size_t)i * 8 * NLOC + (((i >> 1) & 1) ? c1 : c0), lds0 + i * 1024);
        const unsigned call_v = ld_agent(ctr);
        __builtin_amdgcn_s_barrier();
        const unsigned call = uni(call_v) + 1u;
        if (lane == 0) st_agent(ctr, call);
        bool ok = true;
        for (int st = 1; st < NGRP; ++st) {
            if (st >= NS) {
                const unsigned need = (unsigned)(st - NS + 1);
                bool fr = true;
                for (int w = 0; w < 5; ++w) fr = fr && lds_ld(ctl + 4 + 4 * w) >= need;
                if (!fr) {
                    waitvm<0>();
                    lds_st(ctl, (unsigned)st);
                    unsigned got;
                    for (int w = 0; w < 5; ++w) ok = ok && wait_lds<LSLEEP>(ctl + 4 + 4 * w, need, got, tmo, call);
                }
            }
            const unsigned slot = lds0 + (unsigned)(st & (NS - 1)) * STAGE;
            const unsigned char* ws_ = wsrc + (size_t)st * 256 * NLOC;
#pragma unroll 4
            for (int i = 0; i < 32; ++i) glds16(ws_ + (size_t)i * 8 * NLOC + (((i >> 1) & 1) ? c1 : c0), slot + i * 1024);
            waitvm<32>();
            lds_st(ctl, (unsigned)st);
        }
        waitvm<0>();
        lds_st(ctl, (unsigned)NGRP);
        asm volatile("s_waitcnt lgkmcnt(0)" ::: "memory");
        STAMP(2);
        return;
    }

    const bool local = wave == 5;
    const unsigned call_v = ld_agent(ctr);
    const float scale = p.sx[0] * p.sw[0];
    __builtin_amdgcn_s_barrier();
    const unsigned call = uni(call_v) + 1u;
    if (lane == 0) st_agent(ctr, call);
    if (local && !has_local) return;
    STAMP(1);

    const int src = local ? p.rank : ((p.rank + srcsel) & (NR - 1));
    const int rb = local ? (srcsel - 1) * 8 + rt_ : rt_ * 4 + wave;
    const unsigned char* xbase = local ? p.x : me + (size_t)(call & 1u) * DATA_PAR + (size_t)src * SLOT_BYTES;
    const __amdgpu_buffer_rsrc_t xr = __builtin_amdgcn_make_buffer_rsrc((void*)xbase, 0, (int)SLOT_BYTES, 0x00020000);
    const int voff = local ? (rb * 16 + r) * KDIM + 16 * g : rb * 65536 + lane * 16;
    const int kstride = local ? 128 : 2048, hstride = local ? 64 : 1024;
    const u64* flag = (const u64*)(me + FLAG_OFF + (size_t)(call & 1u) * FLAG_PAR + (size_t)(src * 32 + rb) * 128);
    const unsigned fbase = call * 64u;
    const unsigned done_addr = ctl + 4 + 4 * (local ? 4 : wave);
    const unsigned scr = lds0 + LDS_SCR + (local ? 4 : wave) * SCR_BYTES;

    const unsigned q = (unsigned)(lane & 15) >> 1, pp = lane & 1;
    const unsigned swz = ((q >> 1) & 3u) | ((unsigned)(g & 1) << 2);
    const unsigned abase = lds0 + (16u * g + q) * 128u + 8u * pp + (swz << 4);

    v4f acc[8];
#pragma unroll
    for (int c = 0; c < 8; ++c) acc[c] = v4f{0.f, 0.f, 0.f, 0.f};

    bool ok = true;
    unsigned got;
    auto load_w1 = [&](unsigned sb, int c) -> v8i {
        LAS unsigned char* a = (LAS unsigned char*)(size_t)(sb ^ (c << 4));
        v2i w0 = __builtin_amdgcn_ds_read_tr8_b64_v2i32((LAS v2i*)(a));
        v2i w1 = __builtin_amdgcn_ds_read_tr8_b64_v2i32((LAS v2i*)(a + 1024));
        v2i w2 = __builtin_amdgcn_ds_read_tr8_b64_v2i32((LAS v2i*)(a + 8192));
        v2i w3 = __builtin_amdgcn_ds_read_tr8_b64_v2i32((LAS v2i*)(a + 8192 + 1024));
        return v8i{w0[0], w0[1], w1[0], w1[1], w2[0], w2[1], w3[0], w3[1]};
    };
    v8i wf[4];
    ok = wait_lds(ctl, 1u, got, tmo, call);
#pragma unroll
    for (int c = 0; c < 4; ++c) wf[c] = load_w1(abase, c);

    unsigned flo = local ? ~0u : 0u, fhi = local ? ~0u : 0u;
    auto load_x = [&](u32x4* xv, int kg, bool have) {
        const int vo = have ? voff : 0x7ff00000;
#pragma unroll
        for (int i = 0; i < 4; ++i)
            xv[i] = __builtin_amdgcn_raw_buffer_load_b128(xr, vo, (2 * kg + (i >> 1)) * kstride + (i & 1) * hstride, 17);
    };
    auto decode = [&](const u32x4* xv, bool raw, v8i& x0, v8i& x1) {
        if (raw) {
            x0 = v8i{(int)xv[0][0], (int)xv[0][1], (int)xv[0][2], (int)xv[0][3], (int)xv[1][0], (int)xv[1][1], (int)xv[1][2], (int)xv[1][3]};
            x1 = v8i{(int)xv[2][0], (int)xv[2][1], (int)xv[2][2], (int)xv[2][3], (int)xv[3][0], (int)xv[3][1], (int)xv[3][2], (int)xv[3][3]};
        } else {
            unpack_group(xv, scr, lane, x0, x1);
        }
    };
    auto mma = [&](const v8i x0, const v8i x1, int kg) {
        const unsigned sb = abase + (unsigned)(kg & (NS - 1)) * STAGE;
#pragma unroll
        for (int c = 0; c < 8; ++c) {
            const v8i cur = wf[c & 3];
            acc[c] = __builtin_amdgcn_mfma_scale_f32_16x16x128_f8f6f4(cur, x0, acc[c], 0, 0, 0, 0, 0, 0);
            wf[c & 3] = c < 4 ? load_w1(sb, c + 4) : load_w1(sb + 16384, c - 4);
            __builtin_amdgcn_sched_group_barrier(0x008, 1, 0);
            __builtin_amdgcn_sched_group_barrier(0x100, 4, 0);
        }
#pragma unroll
        for (int c = 0; c < 4; ++c) {
            const v8i cur = wf[c];
            acc[c] = __builtin_amdgcn_mfma_scale_f32_16x16x128_f8f6f4(cur, x1, acc[c], 0, 0, 0, 0, 0, 0);
            wf[c] = load_w1(sb + 16384, c + 4);
            __builtin_amdgcn_sched_group_barrier(0x008, 1, 1);
            __builtin_amdgcn_sched_group_barrier(0x100, 4, 1);
        }
        lds_st(done_addr, (unsigned)(kg + 1));
        const int kn = kg + 1 < NGRP ? kg + 1 : kg;
        if (kg + 1 < NGRP) ok = ok && wait_lds<4>(ctl, (unsigned)(kn + 1), got, tmo, call);
        const unsigned sn = abase + (unsigned)(kn & (NS - 1)) * STAGE;
#pragma unroll
        for (int c = 4; c < 8; ++c) {
            const v8i cur = wf[c & 3];
            acc[c] = __builtin_amdgcn_mfma_scale_f32_16x16x128_f8f6f4(cur, x1, acc[c], 0, 0, 0, 0, 0, 0);
            wf[c & 3] = load_w1(sn, c - 4);
            __builtin_amdgcn_sched_group_barrier(0x008, 1, 2);
            __builtin_amdgcn_sched_group_barrier(0x100, 4, 2);
        }
    };
    auto compute = [&](const u32x4* xv, int kg, bool raw) {
        v8i x0, x1;
        decode(xv, raw, x0, x1);
        mma(x0, x1, kg);
    };
    auto await_direct = [&](unsigned need) {
        const u64 t0 = rt();
        for (unsigned spins = 1;; ++spins) {
            const u64 v = ld_sys64(flag);
            flo = uni((unsigned)v); fhi = uni((unsigned)(v >> 32));
            if (flo >= need) return;
            __builtin_amdgcn_s_sleep(1);
            if ((spins & 15u) == 0u) {
                if (uni(ld_agent(tmo)) == call || rt() - t0 > DEADLINE) { st_agent(tmo, call); ok = false; return; }
            }
        }
    };
    auto await_fast = [&](unsigned need) {
        u64 v0 = ld_sys64(flag);
        __builtin_amdgcn_s_sleep(FSLEEP);
        u64 v1 = ld_sys64(flag);
        __builtin_amdgcn_s_sleep(FSLEEP);
        u64 v2 = ld_sys64(flag);
        __builtin_amdgcn_s_sleep(FSLEEP);
        u64 v3 = ld_sys64(flag);
        const u64 t0 = rt();
        for (unsigned spins = 1;; ++spins) {
#define POLL1(v) flo = uni((unsigned)v); fhi = uni((unsigned)(v >> 32)); if (flo >= need) return; v = ld_sys64(flag); __builtin_amdgcn_s_sleep(FSLEEP);
            POLL1(v0) POLL1(v1) POLL1(v2) POLL1(v3)
#undef POLL1
            if ((spins & 15u) == 0u) {
                if (uni(ld_agent(tmo)) == call || rt() - t0 > DEADLINE) { st_agent(tmo, call); ok = false; return; }
            }
        }
    };
    auto step = [&](const u32x4* cur, u32x4* nxt, int kg) {
        const unsigned need = fbase + (unsigned)kg + 2u;
        const bool raw = ((fhi >> kg) & 1u) || !ok;
        const bool early = ok && flo >= need;
        load_x(nxt, kg + 1, early);
        v8i x0, x1;
        decode(cur, raw, x0, x1);
        const u64 fv = ld_sys64(flag);
        mma(x0, x1, kg);
        if (!early) {
            if (!local) {
                flo = uni((unsigned)fv); fhi = uni((unsigned)(fv >> 32));
                if (flo < need && ok) {
                    if (FASTLAST && kg == NFLAG - 2) await_fast(need);
                    else await_direct(need);
                }
            }
            asm volatile("" ::: "memory");
            load_x(nxt, kg + 1, ok);
        } else if (!local) {
            const unsigned l2 = uni((unsigned)fv);
            if (l2 > flo) { flo = l2; fhi = uni((unsigned)(fv >> 32)); }
        }
    };
    auto settle = [&](u32x4* xv, int kg) {
        if (local || !ok) return;
        const u64 t0 = rt();
        for (unsigned spins = 1;; ++spins) {
            unsigned m = ~0u;
#pragma unroll
            for (int i = 0; i < 4; ++i) m = min(min(m, min(xv[i][0], xv[i][1])), min(xv[i][2], xv[i][3]));
            if (!__any(m == 0u)) {
#pragma unroll
                for (int i = 0; i < 4; ++i) xv[i] ^= SMASK;
                return;
            }
            load_x(xv, kg, true);
            const u64 fv = ld_sys64(flag);
            const unsigned lo = uni((unsigned)fv), hi = uni((unsigned)(fv >> 32));
            if (lo >= fbase + (unsigned)kg + 1u && ((hi >> (16 + kg - NFLAG)) & 1u)) {
                asm volatile("" ::: "memory");
                const __amdgpu_buffer_rsrc_t fr = __builtin_amdgcn_make_buffer_rsrc(
                    (void*)(me + FB_OFF + (size_t)(call & 1u) * FB_PAR + (size_t)(src * 32 + rb) * 8192 + (kg - NFLAG) * 4096), 0, 4096, 0x00020000);
#pragma unroll
                for (int i = 0; i < 4; ++i) xv[i] = __builtin_amdgcn_raw_buffer_load_b128(fr, lane * 16, i * 1024, 17);
                return;
            }
            if ((spins & 7u) == 0u) {
                if (uni(ld_agent(tmo)) == call || rt() - t0 > DEADLINE) { st_agent(tmo, call); ok = false; return; }
            }
        }
    };
    u32x4 xa[4], xb[4];
    if (!local) await_direct(fbase + 1u);
    asm volatile("" ::: "memory");
    load_x(xa, 0, ok);
    for (int kg = 0; kg < NFLAG - 2; kg += 2) {
        step(xa, xb, kg);
        step(xb, xa, kg + 1);
    }
    step(xa, xb, NFLAG - 2);
    load_x(xa, NFLAG, ok);
    STAMP(8 + NFLAG - 1);
    compute(xb, NFLAG - 1, ((fhi >> (NFLAG - 1)) & 1u) || !ok);
    settle(xa, NFLAG);
    load_x(xb, NFLAG + 1, ok);
    STAMP(8 + NFLAG);
    compute(xa, NFLAG, true);
    const unsigned sl = abase + (unsigned)((NFLAG + 1) & (NS - 1)) * STAGE;
    v8i wl[4];
#pragma unroll
    for (int c = 0; c < 4; ++c) wl[c] = load_w1(sl, c + 4);
    const __amdgpu_buffer_rsrc_t orr = __builtin_amdgcn_make_buffer_rsrc((void*)p.out, 0, NR * ROWS * NLOC * 4, 0x00020000);
    const int oa = ((src * ROWS + rb * 16 + r) * NLOC + ct * 128 + 4 * g) * 4;
    STAMP(8 + NFLAG + 1);
    if (TWOTRY && !local && ok) {
        unsigned m = ~0u;
#pragma unroll
        for (int i = 0; i < 4; ++i) m = min(min(m, min(xb[i][0], xb[i][1])), min(xb[i][2], xb[i][3]));
        bool freed = true;
        for (int w = 0; w < 5; ++w) freed = freed && lds_ld(ctl + 4 + 4 * w) >= (unsigned)(NFLAG - 1);
        if (__any(m == 0u) && freed) {
            const unsigned char* gsrc = xbase + voff + (NFLAG + 1) * 4096;
            const unsigned ba = lds0 + (unsigned)((NFLAG - 2) & (NS - 1)) * STAGE + (unsigned)wave * 8192u, bb = ba + 4096u;
#pragma unroll
            for (int i = 0; i < 4; ++i) glds16_sys(gsrc + i * 1024, ba + i * 1024);
            __builtin_amdgcn_s_sleep(TRYGAP);
#pragma unroll
            for (int i = 0; i < 4; ++i) glds16_sys(gsrc + i * 1024, bb + i * 1024);
            for (int spins = 0; spins < 8; ++spins) {
                waitvm<4>();
                lds_ld16x4(ba + lane * 16, xb);
                m = ~0u;
#pragma unroll
                for (int i = 0; i < 4; ++i) m = min(min(m, min(xb[i][0], xb[i][1])), min(xb[i][2], xb[i][3]));
                if (!__any(m == 0u)) break;
#pragma unroll
                for (int i = 0; i < 4; ++i) glds16_sys(gsrc + i * 1024, ba + i * 1024);
                waitvm<4>();
                lds_ld16x4(bb + lane * 16, xb);
                m = ~0u;
#pragma unroll
                for (int i = 0; i < 4; ++i) m = min(min(m, min(xb[i][0], xb[i][1])), min(xb[i][2], xb[i][3]));
                if (!__any(m == 0u)) break;
#pragma unroll
                for (int i = 0; i < 4; ++i) glds16_sys(gsrc + i * 1024, bb + i * 1024);
            }
        }
    }
    settle(xb, NFLAG + 1);
    STAMP(2);
    const unsigned tv = ld_agent(tmo);
    {
        const v8i x0 = {(int)xb[0][0], (int)xb[0][1], (int)xb[0][2], (int)xb[0][3], (int)xb[1][0], (int)xb[1][1], (int)xb[1][2], (int)xb[1][3]};
        const v8i x1 = {(int)xb[2][0], (int)xb[2][1], (int)xb[2][2], (int)xb[2][3], (int)xb[3][0], (int)xb[3][1], (int)xb[3][2], (int)xb[3][3]};
        v8i w2[8];
#pragma unroll
        for (int c = 0; c < 8; ++c) {
            w2[c] = load_w1(sl + 16384, c);
            acc[c] = __builtin_amdgcn_mfma_scale_f32_16x16x128_f8f6f4(c < 4 ? wf[c] : wl[c - 4], x0, acc[c], 0, 0, 0, 0, 0, 0);
            __builtin_amdgcn_sched_group_barrier(0x100, 4, 3);
            __builtin_amdgcn_sched_group_barrier(0x008, 1, 3);
        }
#pragma unroll
        for (int c = 0; c < 8; ++c)
            acc[c] = __builtin_amdgcn_mfma_scale_f32_16x16x128_f8f6f4(w2[c], x1, acc[c], 0, 0, 0, 0, 0, 0);
    }
    __builtin_amdgcn_sched_barrier(0);
    STAMP(3);
    const float sc = uni(tv) == call ? __builtin_nanf("") : scale;
#pragma unroll
    for (int c = 0; c < 8; ++c) {
        const v4f y = __builtin_elementwise_maximum(acc[c] * sc, v4f{0.f, 0.f, 0.f, 0.f});
        __builtin_amdgcn_raw_buffer_store_b128(__builtin_bit_cast(u32x4, y), orr, oa + 64 * c, 0, OUTAUX);
    }
    STAMP(4);
}

}

extern "C" void kernel_launch(int rank, int nranks, ncclComm_t comm, hipStream_t stream,
                              void* const* d_in, const int* in_sizes, int n_in,
                              void* d_out, int out_size,
                              const int* in_dtypes, int out_dtype,
                              void* const* d_out_all,
                              void* const* d_ws_all, size_t ws_bytes) {
    (void)comm; (void)in_dtypes; (void)out_dtype; (void)d_out_all;
    if (nranks != NR || (unsigned)rank >= (unsigned)NR || n_in != 4 || ws_bytes < WS_NEED ||
        in_sizes[0] != ROWS * KDIM || in_sizes[1] != KDIM * NLOC || out_size != NR * ROWS * NLOC) {
        fprintf(stderr, "kernel_launch: unexpected problem shape\n");
        return;
    }
    static bool init = false;
    if (!init) {
        HIP_CHECK(hipFuncSetAttribute((const void*)ag_gemm, hipFuncAttributeMaxDynamicSharedMemorySize, LDS_BYTES));
        init = true;
    }
    Params p;
    p.x = (const unsigned char*)d_in[0];
    p.w = (const unsigned char*)d_in[1];
    p.sx = (const float*)d_in[2];
    p.sw = (const float*)d_in[3];
    p.out = (float*)d_out;
    p.me = (unsigned char*)d_ws_all[rank];
    for (int d = 0; d < NR - 1; ++d) p.peer[d] = (unsigned char*)d_ws_all[(rank + 1 + d) % NR];
    p.rank = rank;
    p.pad = 0;
    hipLaunchKernelGGL(ag_gemm, dim3(256), dim3(384), LDS_BYTES, stream, p);
    HIP_CHECK(hipGetLastError());
}
```

```cpp
#include <hip/hip_runtime.h>
#include <rccl/rccl.h>
#include <cstdio>
#include <cstdint>

#define HIP_CHECK(expr) do { hipError_t _e = (expr); if (_e != hipSuccess) { \
    fprintf(stderr, "HIP error %s at %s:%d\n", hipGetErrorString(_e), __FILE__, __LINE__); \
    return; } } while (0)

#define PACK 1
#define FSLEEP 4
#define FASTLAST 0
#define LATE 0
#define WBIG 40
#define WLAST 8
#define OUTAUX 17
#define HEADOPT 1
#define ROTA 1
#define ROTB 0
#define G0W 28
#define LSLEEP 12
#define PACE 3730
#define PACEOFF 0
#define LAS __attribute__((address_space(3)))
typedef unsigned u32x4 __attribute__((ext_vector_type(4)));
typedef unsigned long long u64;
typedef int v8i __attribute__((ext_vector_type(8)));
typedef int v2i __attribute__((ext_vector_type(2)));
typedef float v4f __attribute__((ext_vector_type(4)));

namespace {

constexpr int NR = 8;
constexpr int KDIM = 4096;
constexpr int NLOC = 512;
constexpr int ROWS = 512;
#define RBPAD 1024
#define SLPAD 0
constexpr unsigned XBYTES = ROWS * KDIM;
constexpr unsigned RB_BYTES = 65536 + RBPAD;
constexpr unsigned SLOT_BYTES = 32 * RB_BYTES + SLPAD;
constexpr size_t DATA_PAR = (size_t)NR * SLOT_BYTES;
constexpr size_t FB_OFF = 2 * DATA_PAR;
constexpr size_t FB_PAR = (size_t)NR * 32 * 8192;
constexpr size_t FLAG_OFF = FB_OFF + 2 * FB_PAR;
constexpr size_t FLAG_PAR = (size_t)NR * 32 * 128;
constexpr size_t CTR_OFF = FLAG_OFF + 64 * 1024;
constexpr size_t TMO_OFF = CTR_OFF + 256 * 8 * 64;
constexpr size_t DBG_OFF = TMO_OFF + 4096;
constexpr size_t WS_NEED = DBG_OFF + 256 * 8 * 32 * 8;
constexpr u64 DEADLINE = 30000000ull;

constexpr int NGRP = 16;
constexpr int NFLAG = NGRP - 2;
#define SMASK 0x7F7F7F7Fu
constexpr int NS = 4;
constexpr int STAGE = 256 * 128;
constexpr int LDS_CTL = NS * STAGE;
constexpr int LDS_SCR = LDS_CTL + 256;
constexpr int SCR_BYTES = 1024 + 64;
constexpr int LDS_BYTES = LDS_SCR + 5 * SCR_BYTES;
#define WNUM 16
constexpr int WN = WNUM;

__device__ __forceinline__ unsigned gpos(int g, int rb, int src) { return (unsigned)((g + ROTA * rb + ROTB * src) & 15) * 4096u; }

struct Params {
    const unsigned char* x;
    const unsigned char* w;
    const float* sx;
    const float* sw;
    float* out;
    unsigned char* me;
    unsigned char* peer[NR - 1];
    int rank;
    int pad;
};

__device__ __forceinline__ u64 rt() { return __builtin_amdgcn_s_memrealtime(); }
__device__ __forceinline__ unsigned uni(unsigned v) { return __builtin_amdgcn_readfirstlane(v); }
template <int N> __device__ __forceinline__ void waitvm() { asm volatile("s_waitcnt vmcnt(%0)" :: "i"(N) : "memory"); }
__device__ __forceinline__ void st16_sys(void* p, u32x4 v) {
    asm volatile("global_store_dwordx4 %0, %1, off sc0 sc1\n\ts_nop 1" :: "v"(p), "v"(v) : "memory");
}
__device__ __forceinline__ void st16_sys_masked(void* p, u32x4 v, u64 mask) {
    u64 keep;
    asm volatile("s_mov_b64 %0, exec\n\ts_mov_b64 exec, %3\n\tglobal_store_dwordx4 %1, %2, off sc0 sc1\n\ts_mov_b64 exec, %0\n\ts_nop 1"
                 : "=&s"(keep) : "v"(p), "v"(v), "s"(mask) : "memory");
}
__device__ __forceinline__ void glds16(const void* gsrc, unsigned lds_dst) {
    unsigned keep;
    asm volatile("s_mov_b32 %0, m0\n\ts_mov_b32 m0, %2\n\ts_nop 0\n\tglobal_load_lds_dwordx4 %1, off\n\ts_mov_b32 m0, %0"
                 : "=&s"(keep) : "v"(gsrc), "s"(lds_dst) : "memory");
}
__device__ __forceinline__ void lds_st(unsigned addr, unsigned v) {
    asm volatile("ds_write_b32 %0, %1" :: "v"(addr), "v"(v) : "memory");
}
__device__ __forceinline__ void lds_st16(unsigned addr, u32x4 v) {
    asm volatile("ds_write_b128 %0, %1" :: "v"(addr), "v"(v) : "memory");
}
__device__ __forceinline__ void lds_or(unsigned addr, unsigned v) {
    asm volatile("ds_or_b32 %0, %1" :: "v"(addr), "v"(v) : "memory");
}
__device__ __forceinline__ unsigned lds_ldv(unsigned addr) {
    unsigned v;
    asm volatile("ds_read_b32 %0, %1\n\ts_waitcnt lgkmcnt(0)" : "=v"(v) : "v"(addr) : "memory");
    return v;
}
__device__ __forceinline__ unsigned lds_ld(unsigned addr) { return uni(lds_ldv(addr)); }
__device__ __forceinline__ void lds_ld16x4(unsigned addr, u32x4* v) {
    asm volatile("ds_read_b128 %0, %4\n\tds_read_b128 %1, %4 offset:1024\n\tds_read_b128 %2, %4 offset:2048\n\t"
                 "ds_read_b128 %3, %4 offset:3072\n\ts_waitcnt lgkmcnt(0)"
                 : "=&v"(v[0]), "=&v"(v[1]), "=&v"(v[2]), "=&v"(v[3]) : "v"(addr) : "memory");
}
__device__ __forceinline__ unsigned ld_agent(const unsigned* p) { return __hip_atomic_load(p, __ATOMIC_RELAXED, __HIP_MEMORY_SCOPE_AGENT); }
__device__ __forceinline__ void st_agent(unsigned* p, unsigned v) { __hip_atomic_store(p, v, __ATOMIC_RELAXED, __HIP_MEMORY_SCOPE_AGENT); }
__device__ __forceinline__ u64 ld_sys64(const u64* p) { return __hip_atomic_load(p, __ATOMIC_RELAXED, __HIP_MEMORY_SCOPE_SYSTEM); }
__device__ __forceinline__ void st_sys64(u64* p, u64 v) { __hip_atomic_store(p, v, __ATOMIC_RELAXED, __HIP_MEMORY_SCOPE_SYSTEM); }

template <int SLEEP = 1>
__device__ __forceinline__ bool wait_lds(unsigned addr, unsigned want, unsigned& got, unsigned* tmo, unsigned call) {
    got = lds_ld(addr);
    if (got >= want) return true;
    const u64 t0 = rt();
    for (unsigned spins = 1;; ++spins) {
        __builtin_amdgcn_s_sleep(SLEEP);
        got = lds_ld(addr);
        if (got >= want) return true;
        if ((spins & 255u) == 0u) {
            if (uni(ld_agent(tmo)) == call || rt() - t0 > DEADLINE) { st_agent(tmo, call); return false; }
        }
    }
}

__device__ __forceinline__ unsigned perm(unsigned hi, unsigned lo, unsigned sel) { return __builtin_amdgcn_perm(hi, lo, sel); }
__device__ __forceinline__ unsigned bfi(unsigned m, unsigned a, unsigned b) { return (m & a) | (~m & b); }
__device__ __forceinline__ unsigned wave_scan(unsigned v) {
    v += __builtin_amdgcn_update_dpp(0u, v, 0x111, 0xf, 0xf, true);
    v += __builtin_amdgcn_update_dpp(0u, v, 0x112, 0xf, 0xf, true);
    v += __builtin_amdgcn_update_dpp(0u, v, 0x114, 0xf, 0xf, true);
    v += __builtin_amdgcn_update_dpp(0u, v, 0x118, 0xf, 0xf, true);
    v += __builtin_amdgcn_update_dpp(0u, v, 0x142, 0xa, 0xf, false);
    v += __builtin_amdgcn_update_dpp(0u, v, 0x143, 0xc, 0xf, false);
    return v;
}

__device__ __forceinline__ unsigned pack_group(const unsigned* R, unsigned slot, unsigned lut, int lane, bool force_raw) {
    unsigned nl = 0;
    if (!force_raw) {
        unsigned s = 0;
#pragma unroll
        for (int i = 0; i < 16; i += 5) s = __builtin_amdgcn_udot4((R[i] >> 3) & 0x0F0F0F0Fu, 0x01010101u, s, false);
        const unsigned tot = (unsigned)__builtin_amdgcn_readlane((int)wave_scan(s), 63);
        const unsigned a = tot > 102u ? (tot - 102u) >> 10 : 0u;
        const unsigned K = ((5u - a) & 15u) * 0x01010101u;
        unsigned bad = 0, N[8], P[4], n[2];
        u64 acc[2];
#pragma unroll
        for (int j = 0; j < 2; ++j) {
            u64 ac = 0;
            unsigned pos = 0;
#pragma unroll
            for (int q = 0; q < 2; ++q) {
                unsigned pw = 0;
#pragma unroll
                for (int i = 0; i < 4; ++i) {
                    const unsigned r = R[8 * j + 4 * q + i];
                    const unsigned D = (((r >> 3) & 0x0F0F0F0Fu) + K) & 0x0F0F0F0Fu;
                    bad |= (D + 0x05050505u) & 0x10101010u;
                    const unsigned C = perm(0x820180FFu, 0xFFFFFFFFu, D) & 0x03030303u;
                    const unsigned f = perm(0u, 0x01000000u, C);
                    const unsigned t = (D >> 3) & 0x01010101u;
                    const unsigned S = D - (t + (t << 1));
                    const unsigned m = __builtin_amdgcn_udot4(f, 0x08040201u, 0u, false);
                    const unsigned sel = lds_ldv(lut + m * 4);
                    const unsigned c4 = perm(0u, S, sel);
                    const unsigned w = __builtin_amdgcn_udot4(c4, 0x00400801u, (c4 >> 24) << 9, false);
                    ac |= (u64)w << (pos & 63u);
                    pos += 3u * (unsigned)__builtin_popcount(f);
                    pw |= C << (2 * i);
                }
                P[2 * j + q] = pw;
            }
#pragma unroll
            for (int m2 = 0; m2 < 4; ++m2) {
                const unsigned ra = R[8 * j + 2 * m2], rb = R[8 * j + 2 * m2 + 1];
                N[4 * j + m2] = bfi(0x0F0F0F0Fu, bfi(0x08080808u, ra >> 4, ra), bfi(0x70707070u, rb << 4, rb));
            }
            acc[j] = ac; n[j] = pos;
        }
        const unsigned cnt = n[0] | (n[1] << 16);
        const unsigned inc = wave_scan(cnt);
        const unsigned tot2 = (unsigned)__builtin_amdgcn_readlane((int)inc, 63);
        const unsigned t0 = tot2 & 0xffffu, t1 = tot2 >> 16;
        const bool lane_bad = bad != 0u || n[0] > 63u || n[1] > 63u;
        const unsigned bits = 32u + t0 + t1;
        if (!__any(lane_bad) && bits <= 8192u) {
            const unsigned exc = inc - cnt;
            lds_st16(slot + lane * 16, u32x4{N[0], N[1], N[2], N[3]});
            lds_st16(slot + 1024 + lane * 16, u32x4{N[4], N[5], N[6], N[7]});
            lds_st16(slot + 2048 + lane * 16, u32x4{P[0], P[1], P[2], P[3]});
            lds_st16(slot + 3072 + lane * 16, u32x4{lane == 0 ? a : 0u, 0u, 0u, 0u});
#pragma unroll
            for (int j = 0; j < 2; ++j) {
                const unsigned off = 32u + (j ? t0 + (exc >> 16) : (exc & 0xffffu));
                const unsigned sh = off & 31u, ad = slot + 3072 + (off >> 5) * 4;
                const u64 lo = acc[j] << sh;
                const unsigned hi = (unsigned)((acc[j] >> 32) >> (32u - sh));
                lds_or(ad, (unsigned)lo);
                lds_or(ad + 4, (unsigned)(lo >> 32));
                if ((off >> 5) + 2 < 256u) lds_or(ad + 8, hi);
            }
            nl = ((bits + 511u) >> 9) * 4u;
        }
    }
    if (nl == 0) {
#pragma unroll
        for (int i = 0; i < 4; ++i) lds_st16(slot + i * 1024 + lane * 16, u32x4{R[4 * i], R[4 * i + 1], R[4 * i + 2], R[4 * i + 3]});
    }
    return nl;
}

__device__ __forceinline__ unsigned opaque(unsigned k) { asm("" : "+s"(k)); return k; }
__device__ __forceinline__ unsigned sel3(unsigned m, unsigned a, unsigned b) { return __builtin_amdgcn_bitop3_b32(m, a, b, 0xCA); }
__device__ __forceinline__ void unpack_group(const u32x4* v, unsigned scr, int lane, v8i& x0, v8i& x1) {
    lds_st16(scr + lane * 16, v[3]);
    const unsigned a = (unsigned)__builtin_amdgcn_readlane((int)v[3][0], 0) & 15u;
    const unsigned TP = (((a * 0x01010101u) + 0x00020100u) & 0x0F0F0F0Fu) << 3;
    const unsigned bs = ((a - 5u) & 15u) * 0x01010101u;
    const unsigned TL = ((bs + 0x03020100u) & 0x0F0F0F0Fu) << 3;
    const unsigned TH = ((bs + 0x0A090804u) & 0x0F0F0F0Fu) << 3;
    const unsigned M03 = opaque(0x03030303u), M01 = opaque(0x01010101u), M07 = opaque(0x07070707u), K3 = 0x010101u;
    const unsigned MFF = opaque(0xFF000000u), M16 = opaque(0x0000FFFFu), M8 = opaque(0x00FF00FFu), M80 = opaque(0x80808080u), M78 = opaque(0x78787878u);
    unsigned C[16], FM[16], IX[16], nb[2][9];
#pragma unroll
    for (int j = 0; j < 2; ++j) {
        nb[j][0] = 0;
#pragma unroll
        for (int i = 0; i < 8; ++i) {
            const unsigned c = (v[2][2 * j + (i >> 2)] >> (2 * (i & 3))) & M03;
            const unsigned fm = perm(0u, MFF, c);
            const unsigned f1 = fm & M01;
            C[8 * j + i] = c; FM[8 * j + i] = fm;
            IX[8 * j + i] = perm(__umul24(f1, K3), 0u, 0x0605040Cu);
            nb[j][i + 1] = __builtin_amdgcn_udot4(f1, M03, nb[j][i], false);
        }
    }
    const unsigned cnt = nb[0][8] | (nb[1][8] << 16);
    const unsigned inc = wave_scan(cnt);
    const unsigned t0 = (unsigned)__builtin_amdgcn_readlane((int)inc, 63) & 0xffffu;
    const unsigned exc = inc - cnt;
    u64 win[2];
    {
        const unsigned off0 = 32u + (exc & 0xffffu), off1 = 32u + t0 + (exc >> 16);
        unsigned d2, e2;
        asm volatile("ds_read2_b32 %0, %4 offset1:1\n\tds_read_b32 %1, %4 offset:8\n\t"
                     "ds_read2_b32 %2, %5 offset1:1\n\tds_read_b32 %3, %5 offset:8\n\ts_waitcnt lgkmcnt(0)"
                     : "=&v"(win[0]), "=&v"(d2), "=&v"(win[1]), "=&v"(e2) : "v"(scr + (off0 >> 5) * 4), "v"(scr + (off1 >> 5) * 4) : "memory");
        const unsigned d0 = (unsigned)win[0], d1 = (unsigned)(win[0] >> 32), e0 = (unsigned)win[1], e1 = (unsigned)(win[1] >> 32);
        win[0] = (u64)__builtin_amdgcn_alignbit(d1, d0, off0) | ((u64)__builtin_amdgcn_alignbit(d2, d1, off0) << 32);
        win[1] = (u64)__builtin_amdgcn_alignbit(e1, e0, off1) | ((u64)__builtin_amdgcn_alignbit(e2, e1, off1) << 32);
    }
    unsigned out[16];
#pragma unroll
    for (int j = 0; j < 2; ++j) {
#pragma unroll
        for (int i = 0; i < 8; ++i) {
            const unsigned w = (unsigned)(win[j] >> nb[j][i]);
            const unsigned u = sel3(M16, w, w << 10);
            const unsigned sp = sel3(M8, u, u << 5) & M07;
            const unsigned es = perm(TH, TL, perm(0u, sp, IX[8 * j + i]));
            const unsigned e = sel3(FM[8 * j + i], es, perm(0u, TP, C[8 * j + i]));
            const unsigned n = v[j][i >> 1];
            const unsigned sm = (i & 1) ? sel3(M80, n, n >> 4) : sel3(M80, n << 4, n);
            out[8 * j + i] = sel3(M78, e, sm);
        }
    }
    x0 = v8i{(int)out[0], (int)out[1], (int)out[2], (int)out[3], (int)out[4], (int)out[5], (int)out[6], (int)out[7]};
    x1 = v8i{(int)out[8], (int)out[9], (int)out[10], (int)out[11], (int)out[12], (int)out[13], (int)out[14], (int)out[15]};
}
__device__ __forceinline__ unsigned compact_sel(unsigned m) {
    unsigned s = 0x0C0C0C0Cu, k = 0;
#pragma unroll
    for (unsigned b = 0; b < 4; ++b)
        if (m & (1u << b)) { s = (s & ~(0xFFu << (8 * k))) | (b << (8 * k)); ++k; }
    return s;
}

#define STAMP(i) do { } while (0)

__global__ void __attribute__((amdgpu_flat_work_group_size(384, 384), amdgpu_waves_per_eu(2, 2))) ag_gemm(Params p) {
    extern __shared__ __attribute__((aligned(16))) unsigned char smem[];
    const int lane = threadIdx.x & 63;
    const int wave = uni(threadIdx.x >> 6);
    const int b = blockIdx.x;
    const int rt_ = b & 7;
    const int ct = (b >> 3) & 3;
    const int srcsel = b >> 5;
    const unsigned lds0 = (unsigned)(size_t)(LAS unsigned char*)smem;
    const unsigned ctl = lds0 + LDS_CTL;
    unsigned char* me = p.me;
    unsigned* tmo = (unsigned*)(me + TMO_OFF);
    unsigned* ctr = (unsigned*)(me + CTR_OFF + (b * 8 + wave) * 64);
    STAMP(0);
    const int g = lane >> 4, r = lane & 15;
    asm volatile("" :: "s"(p.x), "s"(p.me), "s"(p.peer[0]), "s"(p.peer[1]), "s"(p.peer[2]), "s"(p.peer[3]), "s"(p.peer[4]), "s"(p.peer[5]), "s"(p.peer[6]), "s"(p.rank));

    if (__builtin_expect(srcsel == 0, HEADOPT)) {
        const int rb = rt_ * 4 + ct;
        const unsigned char* xs = p.x + (size_t)(rb * 16 + r) * KDIM + 16 * g;
        if (__builtin_expect(wave == 5, HEADOPT)) {
            u32x4 d0[4];
#pragma unroll
            for (int i = 0; i < 4; ++i) d0[i] = *(const u32x4*)(xs + (i >> 1) * 128 + (i & 1) * 64);
            const unsigned call_v = ld_agent(ctr);
            unsigned plo = 0, phi = 0;
#pragma unroll
            for (int d = 0; d < NR - 1; ++d) {
                plo = lane == d ? (unsigned)(uintptr_t)p.peer[d] : plo;
                phi = lane == d ? (unsigned)((uintptr_t)p.peer[d] >> 32) : phi;
            }
            __builtin_amdgcn_s_barrier();
#pragma unroll
            for (int i = 0; i < 4; ++i) asm volatile("" : "+v"(d0[i]));
            const unsigned call = uni(call_v) + 1u;
            if (lane == 0) st_agent(ctr, call);
            STAMP(1);
            const size_t doff = (size_t)(call & 1u) * DATA_PAR + (size_t)p.rank * SLOT_BYTES + (size_t)rb * RB_BYTES + lane * 16;
            unsigned char* dst[NR - 1];
            const int rot = rb % 7;
#pragma unroll
            for (int d = 0; d < NR - 1; ++d) {
                int q = d + rot; if (q >= 7) q -= 7;
                const uintptr_t a = (uintptr_t)(unsigned)__builtin_amdgcn_readlane(plo, q) | ((uintptr_t)(unsigned)__builtin_amdgcn_readlane(phi, q) << 32);
                dst[d] = (unsigned char*)a + doff;
            }
            u64* fl = (u64*)((((uintptr_t)phi << 32) | plo) + FLAG_OFF + (size_t)(call & 1u) * FLAG_PAR + (size_t)(p.rank * 32 + rb) * 128);
            const unsigned fbase = call * 64u;
            unsigned rawmask = 1u;
            const u64 tstart = rt();
            unsigned sent = 4096;
#pragma unroll
            for (int d = 0; d < NR - 1; ++d) {
#pragma unroll
                for (int i = 0; i < 4; ++i) st16_sys(dst[d] + gpos(0, rb, p.rank) + i * 1024, d0[i]);
                if (4 * (d + 1) > G0W) waitvm<G0W>();
            }
            unsigned fbmask = 0;
            const long long fbdelta = (long long)(FB_OFF + (size_t)(call & 1u) * FB_PAR + (size_t)(p.rank * 32 + rb) * 8192 + lane * 16) - (long long)doff;
            for (int grp = 1; grp < NGRP; ++grp) {
                unsigned meta;
                if (!wait_lds(ctl + 4 * grp, 1u, meta, tmo, call)) return;
                if (grp == NFLAG) {
                    unsigned got;
                    for (int w = 0; w < 5; ++w)
                        if (!wait_lds(ctl + 4 * (16 + w), 1u, got, tmo, call)) return;
                }
                STAMP(8 + grp);
                const unsigned nl = meta & 0xffu;
                const bool fb = (meta & 0x200u) != 0u;
                rawmask |= (nl == 0u ? 1u : 0u) << grp;
                const u64 em = (nl == 0u || nl >= 64u) ? ~0ull : ((1ull << nl) - 1ull);
                const long long goff = fb ? fbdelta + (grp - NFLAG) * 4096 : (long long)gpos(grp, rb, p.rank);
                u32x4 v[4];
                lds_ld16x4(lds0 + grp * 4096 + lane * 16, v);
                bool late = false;
                if (PACE) {
                    const u64 due = tstart + (u64)(long long)(PACEOFF) + (((u64)sent * PACE) >> 16);
                    u64 now = rt();
                    late = now > due + LATE;
                    while (now < due) { __builtin_amdgcn_s_sleep(1); now = rt(); }
                    sent += nl ? 3072u + nl * 16u : 4096u;
                }
#pragma unroll
                for (int d = 0; d < NR - 1; ++d) {
#pragma unroll
                    for (int i = 0; i < 3; ++i) st16_sys(dst[d] + goff + i * 1024, v[i]);
                    st16_sys_masked(dst[d] + goff + 3072, v[3], em);
                    if (4 * (d + 1) == WLAST && grp == NFLAG) {
                        waitvm<WLAST>();
                        if (lane < 7) st_sys64(fl, ((u64)(rawmask | fbmask) << 32) | (fbase + (unsigned)grp));
                    } else if (4 * (d + 1) == WN) {
                        waitvm<WN>();
                        if (grp < NFLAG && lane < 7) st_sys64(fl, ((u64)(rawmask | fbmask) << 32) | (fbase + (unsigned)grp));
                    } else if (late) {
                        waitvm<WBIG>();
                    } else {
                        waitvm<WN>();
                    }
                }
                if (fb) {
                    fbmask |= 0x10000u << (grp - NFLAG);
                    waitvm<0>();
                    if (lane < 7) st_sys64(fl, ((u64)(rawmask | fbmask) << 32) | (fbase + (unsigned)grp + 1u));
                }
            }
            STAMP(2);
            return;
        }
        const unsigned lut = lds0 + LDS_SCR;
        const unsigned call_v = ld_agent(ctr);
        if (wave == 4) {
            if (lane < 32) lds_st(ctl + lane * 4, 0u);
            if (lane < 16) lds_st(lut + lane * 4, compact_sel(lane));
            asm volatile("s_waitcnt lgkmcnt(0)" ::: "memory");
        }
        __builtin_amdgcn_s_barrier();
        u32x4 rv[3][4];
#pragma unroll
        for (int t = 0; t < 3; ++t)
#pragma unroll
            for (int i = 0; i < 4; ++i) rv[t][i] = *(const u32x4*)(xs + (1 + wave + 5 * t) * 256 + (i >> 1) * 128 + (i & 1) * 64);
#pragma unroll
        for (int t = 0; t < 3; ++t) {
            const int grp = 1 + wave + 5 * t;
            unsigned R[16];
#pragma unroll
            for (int i = 0; i < 16; ++i) R[i] = rv[t][i >> 2][i & 3];
            unsigned fbf = 0;
            if (grp >= NFLAG) {
                bool hit = false;
#pragma unroll
                for (int i = 0; i < 16; ++i) hit = hit || R[i] == SMASK;
                if (__any(hit)) fbf = 0x200u;
                else {
#pragma unroll
                    for (int i = 0; i < 16; ++i) R[i] ^= SMASK;
                }
            }
            const unsigned nl = pack_group(R, lds0 + grp * 4096, lut, lane, !PACK || grp >= NFLAG);
            lds_st(ctl + 4 * grp, 0x100u | fbf | nl);
            if (t == 0) STAMP(1);
        }
        {
            const unsigned call = uni(call_v) + 1u;
            unsigned char* z = me + (size_t)((call + 1u) & 1u) * DATA_PAR + (size_t)rb * RB_BYTES + lane * 16;
            for (int i = wave; i < 56; i += 5) {
                const int s2 = (p.rank + 1 + (i >> 3)) & (NR - 1);
                st16_sys(z + (size_t)s2 * SLOT_BYTES + gpos(NFLAG + ((i >> 2) & 1), rb, s2) + (i & 3) * 1024, u32x4{0u, 0u, 0u, 0u});
            }
            waitvm<0>();
            lds_st(ctl + 4 * (16 + wave), 1u);
        }
        if (lane == 0) st_agent(ctr, uni(call_v) + 1u);
        STAMP(2);
        return;
    }

    const bool has_local = srcsel <= 4;
    if (wave == 4) {
        const unsigned char* wsrc = p.w + (size_t)(lane >> 3) * NLOC + ct * 128;
        const unsigned c0 = (unsigned)(((lane & 7) ^ ((lane >> 4) & 3)) << 4);
        const unsigned c1 = c0 ^ 64u;
        if (lane < 32) lds_st(ctl + lane * 4, (lane == 5 && !has_local) ? 0x7fffffffu : 0u);
        asm volatile("s_waitcnt lgkmcnt(0)" ::: "memory");
#pragma unroll 4
        for (int i = 0; i < 32; ++i) glds16(wsrc + (size_t)i * 8 * NLOC + (((i >> 1) & 1) ? c1 : c0), lds0 + i * 1024);
        const unsigned call_v = ld_agent(ctr);
        __builtin_amdgcn_s_barrier();
        const unsigned call = uni(call_v) + 1u;
        if (lane == 0) st_agent(ctr, call);
        bool ok = true;
        for (int st = 1; st < NGRP; ++st) {
            if (st >= NS) {
                const unsigned need = (unsigned)(st - NS + 1);
                bool fr = true;
                for (int w = 0; w < 5; ++w) fr = fr && lds_ld(ctl + 4 + 4 * w) >= need;
                if (!fr) {
                    waitvm<0>();
                    lds_st(ctl, (unsigned)st);
                    unsigned got;
                    for (int w = 0; w < 5; ++w) ok = ok && wait_lds<LSLEEP>(ctl + 4 + 4 * w, need, got, tmo, call);
                }
            }
            const unsigned slot = lds0 + (unsigned)(st & (NS - 1)) * STAGE;
            const unsigned char* ws_ = wsrc + (size_t)st * 256 * NLOC;
#pragma unroll 4
            for (int i = 0; i < 32; ++i) glds16(ws_ + (size_t)i * 8 * NLOC + (((i >> 1) & 1) ? c1 : c0), slot + i * 1024);
            waitvm<32>();
            lds_st(ctl, (unsigned)st);
        }
        waitvm<0>();
        lds_st(ctl, (unsigned)NGRP);
        asm volatile("s_waitcnt lgkmcnt(0)" ::: "memory");
        STAMP(2);
        return;
    }

    const bool local = wave == 5;
    const unsigned call_v = ld_agent(ctr);
    const float scale = p.sx[0] * p.sw[0];
    __builtin_amdgcn_s_barrier();
    const unsigned call = uni(call_v) + 1u;
    if (lane == 0) st_agent(ctr, call);
    if (local && !has_local) return;
    STAMP(1);

    const int src = local ? p.rank : ((p.rank + srcsel) & (NR - 1));
    const int rb = local ? (srcsel - 1) * 8 + rt_ : rt_ * 4 + wave;
    const unsigned char* xbase = local ? p.x : me + (size_t)(call & 1u) * DATA_PAR + (size_t)src * SLOT_BYTES;
    const __amdgpu_buffer_rsrc_t xr = __builtin_amdgcn_make_buffer_rsrc((void*)xbase, 0, (int)(local ? XBYTES : SLOT_BYTES), 0x00020000);
    const int voff = local ? (rb * 16 + r) * KDIM + 16 * g : rb * (int)RB_BYTES + lane * 16;
    const int kstride = local ? 128 : 2048, hstride = local ? 64 : 1024;
    const u64* flag = (const u64*)(me + FLAG_OFF + (size_t)(call & 1u) * FLAG_PAR + (size_t)(src * 32 + rb) * 128);
    const unsigned fbase = call * 64u;
    const unsigned done_addr = ctl + 4 + 4 * (local ? 4 : wave);
    const unsigned scr = lds0 + LDS_SCR + (local ? 4 : wave) * SCR_BYTES;

    const unsigned q = (unsigned)(lane & 15) >> 1, pp = lane & 1;
    const unsigned swz = ((q >> 1) & 3u) | ((unsigned)(g & 1) << 2);
    const unsigned abase = lds0 + (16u * g + q) * 128u + 8u * pp + (swz << 4);

    v4f acc[8];
#pragma unroll
    for (int c = 0; c < 8; ++c) acc[c] = v4f{0.f, 0.f, 0.f, 0.f};

    bool ok = true;
    unsigned got;
    auto load_w1 = [&](unsigned sb, int c) -> v8i {
        LAS unsigned char* a = (LAS unsigned char*)(size_t)(sb ^ (c << 4));
        v2i w0 = __builtin_amdgcn_ds_read_tr8_b64_v2i32((LAS v2i*)(a));
        v2i w1 = __builtin_amdgcn_ds_read_tr8_b64_v2i32((LAS v2i*)(a + 1024));
        v2i w2 = __builtin_amdgcn_ds_read_tr8_b64_v2i32((LAS v2i*)(a + 8192));
        v2i w3 = __builtin_amdgcn_ds_read_tr8_b64_v2i32((LAS v2i*)(a + 8192 + 1024));
        return v8i{w0[0], w0[1], w1[0], w1[1], w2[0], w2[1], w3[0], w3[1]};
    };
    v8i wf[4];
    ok = wait_lds(ctl, 1u, got, tmo, call);
#pragma unroll
    for (int c = 0; c < 4; ++c) wf[c] = load_w1(abase, c);

    unsigned flo = local ? ~0u : 0u, fhi = local ? ~0u : 0u;
    auto load_x = [&](u32x4* xv, int kg, bool have) {
        const int vo = have ? voff : 0x7ff00000;
        const int so = local ? kg * 256 : (int)gpos(kg, rb, src);
#pragma unroll
        for (int i = 0; i < 4; ++i)
            xv[i] = __builtin_amdgcn_raw_buffer_load_b128(xr, vo, so + (i >> 1) * kstride + (i & 1) * hstride, 17);
    };
    auto decode = [&](const u32x4* xv, bool raw, v8i& x0, v8i& x1) {
        if (raw) {
            x0 = v8i{(int)xv[0][0], (int)xv[0][1], (int)xv[0][2], (int)xv[0][3], (int)xv[1][0], (int)xv[1][1], (int)xv[1][2], (int)xv[1][3]};
            x1 = v8i{(int)xv[2][0], (int)xv[2][1], (int)xv[2][2], (int)xv[2][3], (int)xv[3][0], (int)xv[3][1], (int)xv[3][2], (int)xv[3][3]};
        } else {
            unpack_group(xv, scr, lane, x0, x1);
        }
    };
    auto mma = [&](const v8i x0, const v8i x1, int kg) {
        const unsigned sb = abase + (unsigned)(kg & (NS - 1)) * STAGE;
#pragma unroll
        for (int c = 0; c < 8; ++c) {
            const v8i cur = wf[c & 3];
            acc[c] = __builtin_amdgcn_mfma_scale_f32_16x16x128_f8f6f4(cur, x0, acc[c], 0, 0, 0, 0, 0, 0);
            wf[c & 3] = c < 4 ? load_w1(sb, c + 4) : load_w1(sb + 16384, c - 4);
            __builtin_amdgcn_sched_group_barrier(0x008, 1, 0);
            __builtin_amdgcn_sched_group_barrier(0x100, 4, 0);
        }
#pragma unroll
        for (int c = 0; c < 4; ++c) {
            const v8i cur = wf[c];
            acc[c] = __builtin_amdgcn_mfma_scale_f32_16x16x128_f8f6f4(cur, x1, acc[c], 0, 0, 0, 0, 0, 0);
            wf[c] = load_w1(sb + 16384, c + 4);
            __builtin_amdgcn_sched_group_barrier(0x008, 1, 1);
            __builtin_amdgcn_sched_group_barrier(0x100, 4, 1);
        }
        lds_st(done_addr, (unsigned)(kg + 1));
        const int kn = kg + 1 < NGRP ? kg + 1 : kg;
        if (kg + 1 < NGRP) ok = ok && wait_lds<4>(ctl, (unsigned)(kn + 1), got, tmo, call);
        const unsigned sn = abase + (unsigned)(kn & (NS - 1)) * STAGE;
#pragma unroll
        for (int c = 4; c < 8; ++c) {
            const v8i cur = wf[c & 3];
            acc[c] = __builtin_amdgcn_mfma_scale_f32_16x16x128_f8f6f4(cur, x1, acc[c], 0, 0, 0, 0, 0, 0);
            wf[c & 3] = load_w1(sn, c - 4);
            __builtin_amdgcn_sched_group_barrier(0x008, 1, 2);
            __builtin_amdgcn_sched_group_barrier(0x100, 4, 2);
        }
    };
    auto compute = [&](const u32x4* xv, int kg, bool raw) {
        v8i x0, x1;
        decode(xv, raw, x0, x1);
        mma(x0, x1, kg);
    };
    auto await_direct = [&](unsigned need) {
        const u64 t0 = rt();
        for (unsigned spins = 1;; ++spins) {
            const u64 v = ld_sys64(flag);
            flo = uni((unsigned)v); fhi = uni((unsigned)(v >> 32));
            if (flo >= need) return;
            __builtin_amdgcn_s_sleep(1);
            if ((spins & 15u) == 0u) {
                if (uni(ld_agent(tmo)) == call || rt() - t0 > DEADLINE) { st_agent(tmo, call); ok = false; return; }
            }
        }
    };
    auto await_fast = [&](unsigned need) {
        u64 v0 = ld_sys64(flag);
        __builtin_amdgcn_s_sleep(FSLEEP);
        u64 v1 = ld_sys64(flag);
        __builtin_amdgcn_s_sleep(FSLEEP);
        u64 v2 = ld_sys64(flag);
        __builtin_amdgcn_s_sleep(FSLEEP);
        u64 v3 = ld_sys64(flag);
        const u64 t0 = rt();
        for (unsigned spins = 1;; ++spins) {
#define POLL1(v) flo = uni((unsigned)v); fhi = uni((unsigned)(v >> 32)); if (flo >= need) return; v = ld_sys64(flag); __builtin_amdgcn_s_sleep(FSLEEP);
            POLL1(v0) POLL1(v1) POLL1(v2) POLL1(v3)
#undef POLL1
            if ((spins & 15u) == 0u) {
                if (uni(ld_agent(tmo)) == call || rt() - t0 > DEADLINE) { st_agent(tmo, call); ok = false; return; }
            }
        }
    };
    auto step = [&](const u32x4* cur, u32x4* nxt, int kg) {
        const unsigned need = fbase + (unsigned)kg + 2u;
        const bool raw = ((fhi >> kg) & 1u) || !ok;
        const bool early = ok && flo >= need;
        load_x(nxt, kg + 1, early);
        v8i x0, x1;
        decode(cur, raw, x0, x1);
        const u64 fv = ld_sys64(flag);
        mma(x0, x1, kg);
        if (!early) {
            if (!local) {
                flo = uni((unsigned)fv); fhi = uni((unsigned)(fv >> 32));
                if (flo < need && ok) {
                    if (FASTLAST && kg == NFLAG - 2) await_fast(need);
                    else await_direct(need);
                }
            }
            asm volatile("" ::: "memory");
            load_x(nxt, kg + 1, ok);
        } else if (!local) {
            const unsigned l2 = uni((unsigned)fv);
            if (l2 > flo) { flo = l2; fhi = uni((unsigned)(fv >> 32)); }
        }
    };
    auto settle = [&](u32x4* xv, int kg) {
        if (local || !ok) return;
        const u64 t0 = rt();
        for (unsigned spins = 1;; ++spins) {
            unsigned m = ~0u;
#pragma unroll
            for (int i = 0; i < 4; ++i) m = min(min(m, min(xv[i][0], xv[i][1])), min(xv[i][2], xv[i][3]));
            if (!__any(m == 0u)) {
#pragma unroll
                for (int i = 0; i < 4; ++i) xv[i] ^= SMASK;
                return;
            }
            load_x(xv, kg, true);
            const u64 fv = ld_sys64(flag);
            const unsigned lo = uni((unsigned)fv), hi = uni((unsigned)(fv >> 32));
            if (lo >= fbase + (unsigned)kg + 1u && ((hi >> (16 + kg - NFLAG)) & 1u)) {
                asm volatile("" ::: "memory");
                const __amdgpu_buffer_rsrc_t fr = __builtin_amdgcn_make_buffer_rsrc(
                    (void*)(me + FB_OFF + (size_t)(call & 1u) * FB_PAR + (size_t)(src * 32 + rb) * 8192 + (kg - NFLAG) * 4096), 0, 4096, 0x00020000);
#pragma unroll
                for (int i = 0; i < 4; ++i) xv[i] = __builtin_amdgcn_raw_buffer_load_b128(fr, lane * 16, i * 1024, 17);
                return;
            }
            if ((spins & 7u) == 0u) {
                if (uni(ld_agent(tmo)) == call || rt() - t0 > DEADLINE) { st_agent(tmo, call); ok = false; return; }
            }
        }
    };
    u32x4 xa[4], xb[4];
    if (!local) await_direct(fbase + 1u);
    asm volatile("" ::: "memory");
    load_x(xa, 0, ok);
    for (int kg = 0; kg < NFLAG - 2; kg += 2) {
        step(xa, xb, kg);
        step(xb, xa, kg + 1);
    }
    step(xa, xb, NFLAG - 2);
    load_x(xa, NFLAG, ok);
    STAMP(8 + NFLAG - 1);
    compute(xb, NFLAG - 1, ((fhi >> (NFLAG - 1)) & 1u) || !ok);
    settle(xa, NFLAG);
    load_x(xb, NFLAG + 1, ok);
    STAMP(8 + NFLAG);
    compute(xa, NFLAG, true);
    const unsigned sl = abase + (unsigned)((NFLAG + 1) & (NS - 1)) * STAGE;
    v8i wl[4];
#pragma unroll
    for (int c = 0; c < 4; ++c) wl[c] = load_w1(sl, c + 4);
    const __amdgpu_buffer_rsrc_t orr = __builtin_amdgcn_make_buffer_rsrc((void*)p.out, 0, NR * ROWS * NLOC * 4, 0x00020000);
    const int oa = ((src * ROWS + rb * 16 + r) * NLOC + ct * 128 + 4 * g) * 4;
    STAMP(8 + NFLAG + 1);
    settle(xb, NFLAG + 1);
    STAMP(2);
    const unsigned tv = ld_agent(tmo);
    {
        const v8i x0 = {(int)xb[0][0], (int)xb[0][1], (int)xb[0][2], (int)xb[0][3], (int)xb[1][0], (int)xb[1][1], (int)xb[1][2], (int)xb[1][3]};
        const v8i x1 = {(int)xb[2][0], (int)xb[2][1], (int)xb[2][2], (int)xb[2][3], (int)xb[3][0], (int)xb[3][1], (int)xb[3][2], (int)xb[3][3]};
        v8i w2[8];
#pragma unroll
        for (int c = 0; c < 8; ++c) {
            w2[c] = load_w1(sl + 16384, c);
            acc[c] = __builtin_amdgcn_mfma_scale_f32_16x16x128_f8f6f4(c < 4 ? wf[c] : wl[c - 4], x0, acc[c], 0, 0, 0, 0, 0, 0);
            __builtin_amdgcn_sched_group_barrier(0x100, 4, 3);
            __builtin_amdgcn_sched_group_barrier(0x008, 1, 3);
        }
#pragma unroll
        for (int c = 0; c < 8; ++c)
            acc[c] = __builtin_amdgcn_mfma_scale_f32_16x16x128_f8f6f4(w2[c], x1, acc[c], 0, 0, 0, 0, 0, 0);
    }
    __builtin_amdgcn_sched_barrier(0);
    STAMP(3);
    const float sc = uni(tv) == call ? __builtin_nanf("") : scale;
#pragma unroll
    for (int c = 0; c < 8; ++c) {
        const v4f y = __builtin_elementwise_maximum(acc[c] * sc, v4f{0.f, 0.f, 0.f, 0.f});
        __builtin_amdgcn_raw_buffer_store_b128(__builtin_bit_cast(u32x4, y), orr, oa + 64 * c, 0, OUTAUX);
    }
    STAMP(4);
}

}

extern "C" void kernel_launch(int rank, int nranks, ncclComm_t comm, hipStream_t stream,
                              void* const* d_in, const int* in_sizes, int n_in,
                              void* d_out, int out_size,
                              const int* in_dtypes, int out_dtype,
                              void* const* d_out_all,
                              void* const* d_ws_all, size_t ws_bytes) {
    (void)comm; (void)in_dtypes; (void)out_dtype; (void)d_out_all;
    if (nranks != NR || (unsigned)rank >= (unsigned)NR || n_in != 4 || ws_bytes < WS_NEED ||
        in_sizes[0] != ROWS * KDIM || in_sizes[1] != KDIM * NLOC || out_size != NR * ROWS * NLOC) {
        fprintf(stderr, "kernel_launch: unexpected problem shape\n");
        return;
    }
    static bool init = false;
    if (!init) {
        HIP_CHECK(hipFuncSetAttribute((const void*)ag_gemm, hipFuncAttributeMaxDynamicSharedMemorySize, LDS_BYTES));
        init = true;
    }
    Params p;
    p.x = (const unsigned char*)d_in[0];
    p.w = (const unsigned char*)d_in[1];
    p.sx = (const float*)d_in[2];
    p.sw = (const float*)d_in[3];
    p.out = (float*)d_out;
    p.me = (unsigned char*)d_ws_all[rank];
    for (int d = 0; d < NR - 1; ++d) p.peer[d] = (unsigned char*)d_ws_all[(rank + 1 + d) % NR];
    p.rank = rank;
    p.pad = 0;
    hipLaunchKernelGGL(ag_gemm, dim3(256), dim3(384), LDS_BYTES, stream, p);
    HIP_CHECK(hipGetLastError());
}
```

```cpp
#include <hip/hip_runtime.h>
#include <rccl/rccl.h>
#include <cstdio>
#include <cstdint>

#define HIP_CHECK(expr) do { hipError_t _e = (expr); if (_e != hipSuccess) { \
    fprintf(stderr, "HIP error %s at %s:%d\n", hipGetErrorString(_e), __FILE__, __LINE__); \
    return; } } while (0)

#define LAS __attribute__((address_space(3)))
typedef unsigned u32x4 __attribute__((ext_vector_type(4)));
typedef unsigned long long u64;
typedef int v8i __attribute__((ext_vector_type(8)));
typedef int v2i __attribute__((ext_vector_type(2)));
typedef float v4f __attribute__((ext_vector_type(4)));

namespace {

constexpr int NR = 8;
constexpr int KDIM = 4096;
constexpr int NLOC = 512;
constexpr int ROWS = 512;
constexpr unsigned XBYTES = ROWS * KDIM;
constexpr unsigned RB_BYTES = 65536 + 1024;
constexpr unsigned SLOT_BYTES = 32 * RB_BYTES;
constexpr size_t DATA_PAR = (size_t)NR * SLOT_BYTES;
constexpr size_t FB_OFF = 2 * DATA_PAR;
constexpr size_t FB_PAR = (size_t)NR * 32 * 8192;
constexpr size_t FLAG_OFF = FB_OFF + 2 * FB_PAR;
constexpr size_t FLAG_PAR = (size_t)NR * 32 * 128;
constexpr size_t CTR_OFF = FLAG_OFF + 64 * 1024;
constexpr size_t TMO_OFF = CTR_OFF + 256 * 8 * 64;
constexpr size_t WS_NEED = TMO_OFF + 4096;
constexpr u64 DEADLINE = 30000000ull;

constexpr int NGRP = 16;
constexpr int NFLAG = NGRP - 2;
constexpr unsigned SMASK = 0x7F7F7F7Fu;
constexpr int NS = 4;
constexpr int STAGE = 256 * 128;
constexpr int LDS_CTL = NS * STAGE;
constexpr int LDS_SCR = LDS_CTL + 256;
constexpr int SCR_BYTES = 1024 + 64;
constexpr int LDS_BYTES = LDS_SCR + 5 * SCR_BYTES;
constexpr int WN = 16;
constexpr int WBIG = 40;
constexpr int WLAST = 8;
constexpr unsigned PACE = 3730;

__device__ __forceinline__ unsigned gpos(int g, int rb) { return (unsigned)((g + rb) & 15) * 4096u; }

struct Params {
    const unsigned char* x;
    const unsigned char* w;
    const float* sx;
    const float* sw;
    float* out;
    unsigned char* me;
    unsigned char* peer[NR - 1];
    int rank;
    int pad;
};

__device__ __forceinline__ u64 rt() { return __builtin_amdgcn_s_memrealtime(); }
__device__ __forceinline__ unsigned uni(unsigned v) { return __builtin_amdgcn_readfirstlane(v); }
template <int N> __device__ __forceinline__ void waitvm() { asm volatile("s_waitcnt vmcnt(%0)" :: "i"(N) : "memory"); }
__device__ __forceinline__ void st16_sys(void* p, u32x4 v) {
    asm volatile("global_store_dwordx4 %0, %1, off sc0 sc1\n\ts_nop 1" :: "v"(p), "v"(v) : "memory");
}
__device__ __forceinline__ void st16_sys_masked(void* p, u32x4 v, u64 mask) {
    u64 keep;
    asm volatile("s_mov_b64 %0, exec\n\ts_mov_b64 exec, %3\n\tglobal_store_dwordx4 %1, %2, off sc0 sc1\n\ts_mov_b64 exec, %0\n\ts_nop 1"
                 : "=&s"(keep) : "v"(p), "v"(v), "s"(mask) : "memory");
}
__device__ __forceinline__ void glds16(const void* gsrc, unsigned lds_dst) {
    unsigned keep;
    asm volatile("s_mov_b32 %0, m0\n\ts_mov_b32 m0, %2\n\ts_nop 0\n\tglobal_load_lds_dwordx4 %1, off\n\ts_mov_b32 m0, %0"
                 : "=&s"(keep) : "v"(gsrc), "s"(lds_dst) : "memory");
}
__device__ __forceinline__ void lds_st(unsigned addr, unsigned v) {
    asm volatile("ds_write_b32 %0, %1" :: "v"(addr), "v"(v) : "memory");
}
__device__ __forceinline__ void lds_st16(unsigned addr, u32x4 v) {
    asm volatile("ds_write_b128 %0, %1" :: "v"(addr), "v"(v) : "memory");
}
__device__ __forceinline__ void lds_or(unsigned addr, unsigned v) {
    asm volatile("ds_or_b32 %0, %1" :: "v"(addr), "v"(v) : "memory");
}
__device__ __forceinline__ unsigned lds_ldv(unsigned addr) {
    unsigned v;
    asm volatile("ds_read_b32 %0, %1\n\ts_waitcnt lgkmcnt(0)" : "=v"(v) : "v"(addr) : "memory");
    return v;
}
__device__ __forceinline__ unsigned lds_ld(unsigned addr) { return uni(lds_ldv(addr)); }
__device__ __forceinline__ void lds_ld16x4(unsigned addr, u32x4* v) {
    asm volatile("ds_read_b128 %0, %4\n\tds_read_b128 %1, %4 offset:1024\n\tds_read_b128 %2, %4 offset:2048\n\t"
                 "ds_read_b128 %3, %4 offset:3072\n\ts_waitcnt lgkmcnt(0)"
                 : "=&v"(v[0]), "=&v"(v[1]), "=&v"(v[2]), "=&v"(v[3]) : "v"(addr) : "memory");
}
__device__ __forceinline__ unsigned ld_agent(const unsigned* p) { return __hip_atomic_load(p, __ATOMIC_RELAXED, __HIP_MEMORY_SCOPE_AGENT); }
__device__ __forceinline__ void st_agent(unsigned* p, unsigned v) { __hip_atomic_store(p, v, __ATOMIC_RELAXED, __HIP_MEMORY_SCOPE_AGENT); }
__device__ __forceinline__ u64 ld_sys64(const u64* p) { return __hip_atomic_load(p, __ATOMIC_RELAXED, __HIP_MEMORY_SCOPE_SYSTEM); }
typedef __attribute__((address_space(1))) u64 gu64;
__device__ __forceinline__ void st_sys64(gu64* p, u64 v) { __hip_atomic_store(p, v, __ATOMIC_RELAXED, __HIP_MEMORY_SCOPE_SYSTEM); }

template <int SLEEP = 1>
__device__ __forceinline__ bool wait_lds(unsigned addr, unsigned want, unsigned& got, unsigned* tmo, unsigned call) {
    got = lds_ld(addr);
    if (got >= want) return true;
    const u64 t0 = rt();
    for (unsigned spins = 1;; ++spins) {
        __builtin_amdgcn_s_sleep(SLEEP);
        got = lds_ld(addr);
        if (got >= want) return true;
        if ((spins & 255u) == 0u) {
            if (uni(ld_agent(tmo)) == call || rt() - t0 > DEADLINE) { st_agent(tmo, call); return false; }
        }
    }
}

__device__ __forceinline__ unsigned perm(unsigned hi, unsigned lo, unsigned sel) { return __builtin_amdgcn_perm(hi, lo, sel); }
__device__ __forceinline__ unsigned bfi(unsigned m, unsigned a, unsigned b) { return (m & a) | (~m & b); }
__device__ __forceinline__ unsigned wave_scan(unsigned v) {
    v += __builtin_amdgcn_update_dpp(0u, v, 0x111, 0xf, 0xf, true);
    v += __builtin_amdgcn_update_dpp(0u, v, 0x112, 0xf, 0xf, true);
    v += __builtin_amdgcn_update_dpp(0u, v, 0x114, 0xf, 0xf, true);
    v += __builtin_amdgcn_update_dpp(0u, v, 0x118, 0xf, 0xf, true);
    v += __builtin_amdgcn_update_dpp(0u, v, 0x142, 0xa, 0xf, false);
    v += __builtin_amdgcn_update_dpp(0u, v, 0x143, 0xc, 0xf, false);
    return v;
}

__device__ __forceinline__ unsigned pack_group(const unsigned* R, unsigned slot, unsigned lut, int lane, bool force_raw) {
    unsigned nl = 0;
    if (!force_raw) {
        unsigned s = 0;
#pragma unroll
        for (int i = 0; i < 16; i += 5) s = __builtin_amdgcn_udot4((R[i] >> 3) & 0x0F0F0F0Fu, 0x01010101u, s, false);
        const unsigned tot = (unsigned)__builtin_amdgcn_readlane((int)wave_scan(s), 63);
        const unsigned a = tot > 102u ? (tot - 102u) >> 10 : 0u;
        const unsigned K = ((5u - a) & 15u) * 0x01010101u;
        unsigned bad = 0, N[8], P[4], n[2];
        u64 acc[2];
#pragma unroll
        for (int j = 0; j < 2; ++j) {
            u64 ac = 0;
            unsigned pos = 0;
#pragma unroll
            for (int q = 0; q < 2; ++q) {
                unsigned pw = 0;
#pragma unroll
                for (int i = 0; i < 4; ++i) {
                    const unsigned r = R[8 * j + 4 * q + i];
                    const unsigned D = (((r >> 3) & 0x0F0F0F0Fu) + K) & 0x0F0F0F0Fu;
                    bad |= (D + 0x05050505u) & 0x10101010u;
                    const unsigned C = perm(0x820180FFu, 0xFFFFFFFFu, D) & 0x03030303u;
                    const unsigned f = perm(0u, 0x01000000u, C);
                    const unsigned t = (D >> 3) & 0x01010101u;
                    const unsigned S = D - (t + (t << 1));
                    const unsigned m = __builtin_amdgcn_udot4(f, 0x08040201u, 0u, false);
                    const unsigned sel = lds_ldv(lut + m * 4);
                    const unsigned c4 = perm(0u, S, sel);
                    const unsigned w = __builtin_amdgcn_udot4(c4, 0x00400801u, (c4 >> 24) << 9, false);
                    ac |= (u64)w << (pos & 63u);
                    pos += 3u * (unsigned)__builtin_popcount(f);
                    pw |= C << (2 * i);
                }
                P[2 * j + q] = pw;
            }
#pragma unroll
            for (int m2 = 0; m2 < 4; ++m2) {
                const unsigned ra = R[8 * j + 2 * m2], rb = R[8 * j + 2 * m2 + 1];
                N[4 * j + m2] = bfi(0x0F0F0F0Fu, bfi(0x08080808u, ra >> 4, ra), bfi(0x70707070u, rb << 4, rb));
            }
            acc[j] = ac; n[j] = pos;
        }
        const unsigned cnt = n[0] | (n[1] << 16);
        const unsigned inc = wave_scan(cnt);
        const unsigned tot2 = (unsigned)__builtin_amdgcn_readlane((int)inc, 63);
        const unsigned t0 = tot2 & 0xffffu, t1 = tot2 >> 16;
        const bool lane_bad = bad != 0u || n[0] > 63u || n[1] > 63u;
        const unsigned bits = 32u + t0 + t1;
        if (!__any(lane_bad) && bits <= 8192u) {
            const unsigned exc = inc - cnt;
            lds_st16(slot + lane * 16, u32x4{N[0], N[1], N[2], N[3]});
            lds_st16(slot + 1024 + lane * 16, u32x4{N[4], N[5], N[6], N[7]});
            lds_st16(slot + 2048 + lane * 16, u32x4{P[0], P[1], P[2], P[3]});
            lds_st16(slot + 3072 + lane * 16, u32x4{lane == 0 ? a : 0u, 0u, 0u, 0u});
#pragma unroll
            for (int j = 0; j < 2; ++j) {
                const unsigned off = 32u + (j ? t0 + (exc >> 16) : (exc & 0xffffu));
                const unsigned sh = off & 31u, ad = slot + 3072 + (off >> 5) * 4;
                const u64 lo = acc[j] << sh;
                const unsigned hi = (unsigned)((acc[j] >> 32) >> (32u - sh));
                lds_or(ad, (unsigned)lo);
                lds_or(ad + 4, (unsigned)(lo >> 32));
                if ((off >> 5) + 2 < 256u) lds_or(ad + 8, hi);
            }
            nl = ((bits + 511u) >> 9) * 4u;
        }
    }
    if (nl == 0) {
#pragma unroll
        for (int i = 0; i < 4; ++i) lds_st16(slot + i * 1024 + lane * 16, u32x4{R[4 * i], R[4 * i + 1], R[4 * i + 2], R[4 * i + 3]});
    }
    return nl;
}

__device__ __forceinline__ unsigned opaque(unsigned k) { asm("" : "+s"(k)); return k; }
__device__ __forceinline__ unsigned sel3(unsigned m, unsigned a, unsigned b) { return __builtin_amdgcn_bitop3_b32(m, a, b, 0xCA); }
__device__ __forceinline__ void unpack_group(const u32x4* v, unsigned scr, int lane, v8i& x0, v8i& x1) {
    lds_st16(scr + lane * 16, v[3]);
    const unsigned a = (unsigned)__builtin_amdgcn_readlane((int)v[3][0], 0) & 15u;
    const unsigned TP = (((a * 0x01010101u) + 0x00020100u) & 0x0F0F0F0Fu) << 3;
    const unsigned bs = ((a - 5u) & 15u) * 0x01010101u;
    const unsigned TL = ((bs + 0x03020100u) & 0x0F0F0F0Fu) << 3;
    const unsigned TH = ((bs + 0x0A090804u) & 0x0F0F0F0Fu) << 3;
    const unsigned M03 = opaque(0x03030303u), M01 = opaque(0x01010101u), M07 = opaque(0x07070707u), K3 = 0x010101u;
    const unsigned MFF = opaque(0xFF000000u), M16 = opaque(0x0000FFFFu), M8 = opaque(0x00FF00FFu), M80 = opaque(0x80808080u), M78 = opaque(0x78787878u);
    unsigned C[16], FM[16], IX[16], nb[2][9];
#pragma unroll
    for (int j = 0; j < 2; ++j) {
        nb[j][0] = 0;
#pragma unroll
        for (int i = 0; i < 8; ++i) {
            const unsigned c = (v[2][2 * j + (i >> 2)] >> (2 * (i & 3))) & M03;
            const unsigned fm = perm(0u, MFF, c);
            const unsigned f1 = fm & M01;
            C[8 * j + i] = c; FM[8 * j + i] = fm;
            IX[8 * j + i] = perm(__umul24(f1, K3), 0u, 0x0605040Cu);
            nb[j][i + 1] = __builtin_amdgcn_udot4(f1, M03, nb[j][i], false);
        }
    }
    const unsigned cnt = nb[0][8] | (nb[1][8] << 16);
    const unsigned inc = wave_scan(cnt);
    const unsigned t0 = (unsigned)__builtin_amdgcn_readlane((int)inc, 63) & 0xffffu;
    const unsigned exc = inc - cnt;
    u64 win[2];
    {
        const unsigned off0 = 32u + (exc & 0xffffu), off1 = 32u + t0 + (exc >> 16);
        unsigned d2, e2;
        asm volatile("ds_read2_b32 %0, %4 offset1:1\n\tds_read_b32 %1, %4 offset:8\n\t"
                     "ds_read2_b32 %2, %5 offset1:1\n\tds_read_b32 %3, %5 offset:8\n\ts_waitcnt lgkmcnt(0)"
                     : "=&v"(win[0]), "=&v"(d2), "=&v"(win[1]), "=&v"(e2) : "v"(scr + (off0 >> 5) * 4), "v"(scr + (off1 >> 5) * 4) : "memory");
        const unsigned d0 = (unsigned)win[0], d1 = (unsigned)(win[0] >> 32), e0 = (unsigned)win[1], e1 = (unsigned)(win[1] >> 32);
        win[0] = (u64)__builtin_amdgcn_alignbit(d1, d0, off0) | ((u64)__builtin_amdgcn_alignbit(d2, d1, off0) << 32);
        win[1] = (u64)__builtin_amdgcn_alignbit(e1, e0, off1) | ((u64)__builtin_amdgcn_alignbit(e2, e1, off1) << 32);
    }
    unsigned out[16];
#pragma unroll
    for (int j = 0; j < 2; ++j) {
#pragma unroll
        for (int i = 0; i < 8; ++i) {
            const unsigned w = (unsigned)(win[j] >> nb[j][i]);
            const unsigned u = sel3(M16, w, w << 10);
            const unsigned sp = sel3(M8, u, u << 5) & M07;
            const unsigned es = perm(TH, TL, perm(0u, sp, IX[8 * j + i]));
            const unsigned e = sel3(FM[8 * j + i], es, perm(0u, TP, C[8 * j + i]));
            const unsigned n = v[j][i >> 1];
            const unsigned sm = (i & 1) ? sel3(M80, n, n >> 4) : sel3(M80, n << 4, n);
            out[8 * j + i] = sel3(M78, e, sm);
        }
    }
    x0 = v8i{(int)out[0], (int)out[1], (int)out[2], (int)out[3], (int)out[4], (int)out[5], (int)out[6], (int)out[7]};
    x1 = v8i{(int)out[8], (int)out[9], (int)out[10], (int)out[11], (int)out[12], (int)out[13], (int)out[14], (int)out[15]};
}
__device__ __forceinline__ unsigned compact_sel(unsigned m) {
    unsigned s = 0x0C0C0C0Cu, k = 0;
#pragma unroll
    for (unsigned b = 0; b < 4; ++b)
        if (m & (1u << b)) { s = (s & ~(0xFFu << (8 * k))) | (b << (8 * k)); ++k; }
    return s;
}

__global__ void __attribute__((amdgpu_flat_work_group_size(384, 384), amdgpu_waves_per_eu(2, 2))) ag_gemm(Params p) {
    extern __shared__ __attribute__((aligned(16))) unsigned char smem[];
    const int lane = threadIdx.x & 63;
    const int wave = uni(threadIdx.x >> 6);
    const int b = blockIdx.x;
    const int rt_ = b & 7;
    const int ct = (b >> 3) & 3;
    const int srcsel = b >> 5;
    const unsigned lds0 = (unsigned)(size_t)(LAS unsigned char*)smem;
    const unsigned ctl = lds0 + LDS_CTL;
    unsigned char* me = p.me;
    unsigned* tmo = (unsigned*)(me + TMO_OFF);
    unsigned* ctr = (unsigned*)(me + CTR_OFF + (b * 8 + wave) * 64);
    const int g = lane >> 4, r = lane & 15;
    asm volatile("" :: "s"(p.x), "s"(p.me), "s"(p.peer[0]), "s"(p.peer[1]), "s"(p.peer[2]), "s"(p.peer[3]), "s"(p.peer[4]), "s"(p.peer[5]), "s"(p.peer[6]), "s"(p.rank));

    if (__builtin_expect(srcsel == 0, 1)) {
        const int rb = rt_ * 4 + ct;
        const unsigned char* xs = p.x + (size_t)(rb * 16 + r) * KDIM + 16 * g;
        if (__builtin_expect(wave == 5, 1)) {
            u32x4 d0[4];
#pragma unroll
            for (int i = 0; i < 4; ++i) d0[i] = *(const u32x4*)(xs + (i >> 1) * 128 + (i & 1) * 64);
            const unsigned call_v = ld_agent(ctr);
            unsigned plo = 0, phi = 0;
#pragma unroll
            for (int d = 0; d < NR - 1; ++d) {
                plo = lane == d ? (unsigned)(uintptr_t)p.peer[d] : plo;
                phi = lane == d ? (unsigned)((uintptr_t)p.peer[d] >> 32) : phi;
            }
            __builtin_amdgcn_s_barrier();
#pragma unroll
            for (int i = 0; i < 4; ++i) asm volatile("" : "+v"(d0[i]));
            const unsigned call = uni(call_v) + 1u;
            if (lane == 0) st_agent(ctr, call);
            const size_t doff = (size_t)(call & 1u) * DATA_PAR + (size_t)p.rank * SLOT_BYTES + (size_t)rb * RB_BYTES + lane * 16;
            unsigned char* dst[NR - 1];
            const int rot = rb % 7;
#pragma unroll
            for (int d = 0; d < NR - 1; ++d) {
                int q = d + rot; if (q >= 7) q -= 7;
                const uintptr_t a = (uintptr_t)(unsigned)__builtin_amdgcn_readlane(plo, q) | ((uintptr_t)(unsigned)__builtin_amdgcn_readlane(phi, q) << 32);
                dst[d] = (unsigned char*)a + doff;
            }
            gu64* fl = (gu64*)((((uintptr_t)phi << 32) | plo) + FLAG_OFF + (size_t)(call & 1u) * FLAG_PAR + (size_t)(p.rank * 32 + rb) * 128);
            const unsigned fbase = call * 64u;
            unsigned rawmask = 1u;
            const u64 tstart = rt();
            unsigned sent = 4096;
#pragma unroll
            for (int d = 0; d < NR - 1; ++d)
#pragma unroll
                for (int i = 0; i < 4; ++i) st16_sys(dst[d] + gpos(0, rb) + i * 1024, d0[i]);
            unsigned fbmask = 0;
            const long long fbdelta = (long long)(FB_OFF + (size_t)(call & 1u) * FB_PAR + (size_t)(p.rank * 32 + rb) * 8192 + lane * 16) - (long long)doff;
            for (int grp = 1; grp < NGRP; ++grp) {
                unsigned meta;
                if (!wait_lds(ctl + 4 * grp, 1u, meta, tmo, call)) return;
                if (grp == NFLAG) {
                    unsigned got;
                    for (int w = 0; w < 5; ++w)
                        if (!wait_lds(ctl + 4 * (16 + w), 1u, got, tmo, call)) return;
                }
                const unsigned nl = meta & 0xffu;
                const bool fb = (meta & 0x200u) != 0u;
                rawmask |= (nl == 0u ? 1u : 0u) << grp;
                const u64 em = (nl == 0u || nl >= 64u) ? ~0ull : ((1ull << nl) - 1ull);
                const long long goff = fb ? fbdelta + (grp - NFLAG) * 4096 : (long long)gpos(grp, rb);
                u32x4 v[4];
                lds_ld16x4(lds0 + grp * 4096 + lane * 16, v);
                const u64 due = tstart + (((u64)sent * PACE) >> 16);
                u64 now = rt();
                const bool late = now > due;
                while (now < due) { __builtin_amdgcn_s_sleep(1); now = rt(); }
                sent += nl ? 3072u + nl * 16u : 4096u;
#pragma unroll
                for (int d = 0; d < NR - 1; ++d) {
#pragma unroll
                    for (int i = 0; i < 3; ++i) st16_sys(dst[d] + goff + i * 1024, v[i]);
                    st16_sys_masked(dst[d] + goff + 3072, v[3], em);
                    if (4 * (d + 1) == WLAST && grp == NFLAG) {
                        waitvm<WLAST>();
                        if (lane < 7) st_sys64(fl, ((u64)(rawmask | fbmask) << 32) | (fbase + (unsigned)grp));
                    } else if (4 * (d + 1) == WN) {
                        waitvm<WN>();
                        if (grp < NFLAG && lane < 7) st_sys64(fl, ((u64)(rawmask | fbmask) << 32) | (fbase + (unsigned)grp));
                    } else if (late) {
                        waitvm<WBIG>();
                    } else {
                        waitvm<WN>();
                    }
                }
                if (fb) {
                    fbmask |= 0x10000u << (grp - NFLAG);
                    waitvm<0>();
                    if (lane < 7) st_sys64(fl, ((u64)(rawmask | fbmask) << 32) | (fbase + (unsigned)grp + 1u));
                }
            }
            return;
        }
        const unsigned lut = lds0 + LDS_SCR;
        const unsigned call_v = ld_agent(ctr);
        if (wave == 4) {
            if (lane < 32) lds_st(ctl + lane * 4, 0u);
            if (lane < 16) lds_st(lut + lane * 4, compact_sel(lane));
            asm volatile("s_waitcnt lgkmcnt(0)" ::: "memory");
        }
        __builtin_amdgcn_s_barrier();
        u32x4 rv[3][4];
#pragma unroll
        for (int t = 0; t < 3; ++t)
#pragma unroll
            for (int i = 0; i < 4; ++i) rv[t][i] = *(const u32x4*)(xs + (1 + wave + 5 * t) * 256 + (i >> 1) * 128 + (i & 1) * 64);
#pragma unroll
        for (int t = 0; t < 3; ++t) {
            const int grp = 1 + wave + 5 * t;
            unsigned R[16];
#pragma unroll
            for (int i = 0; i < 16; ++i) R[i] = rv[t][i >> 2][i & 3];
            unsigned fbf = 0;
            if (grp >= NFLAG) {
                bool hit = false;
#pragma unroll
                for (int i = 0; i < 16; ++i) hit = hit || R[i] == SMASK;
                if (__any(hit)) fbf = 0x200u;
                else {
#pragma unroll
                    for (int i = 0; i < 16; ++i) R[i] ^= SMASK;
                }
            }
            const unsigned nl = pack_group(R, lds0 + grp * 4096, lut, lane, grp >= NFLAG);
            lds_st(ctl + 4 * grp, 0x100u | fbf | nl);
        }
        {
            const unsigned call = uni(call_v) + 1u;
            unsigned char* z = me + (size_t)((call + 1u) & 1u) * DATA_PAR + (size_t)rb * RB_BYTES + lane * 16;
            for (int i = wave; i < 56; i += 5) {
                const int s2 = (p.rank + 1 + (i >> 3)) & (NR - 1);
                st16_sys(z + (size_t)s2 * SLOT_BYTES + gpos(NFLAG + ((i >> 2) & 1), rb) + (i & 3) * 1024, u32x4{0u, 0u, 0u, 0u});
            }
            waitvm<0>();
            lds_st(ctl + 4 * (16 + wave), 1u);
        }
        if (lane == 0) st_agent(ctr, uni(call_v) + 1u);
        return;
    }

    const bool has_local = srcsel <= 4;
    if (wave == 4) {
        const unsigned char* wsrc = p.w + (size_t)(lane >> 3) * NLOC + ct * 128;
        const unsigned c0 = (unsigned)(((lane & 7) ^ ((lane >> 4) & 3)) << 4);
        const unsigned c1 = c0 ^ 64u;
        if (lane < 32) lds_st(ctl + lane * 4, (lane == 5 && !has_local) ? 0x7fffffffu : 0u);
        asm volatile("s_waitcnt lgkmcnt(0)" ::: "memory");
#pragma unroll 4
        for (int i = 0; i < 32; ++i) glds16(wsrc + (size_t)i * 8 * NLOC + (((i >> 1) & 1) ? c1 : c0), lds0 + i * 1024);
        const unsigned call_v = ld_agent(ctr);
        __builtin_amdgcn_s_barrier();
        const unsigned call = uni(call_v) + 1u;
        if (lane == 0) st_agent(ctr, call);
        bool ok = true;
        for (int st = 1; st < NGRP; ++st) {
            if (st >= NS) {
                const unsigned need = (unsigned)(st - NS + 1);
                bool fr = true;
                for (int w = 0; w < 5; ++w) fr = fr && lds_ld(ctl + 4 + 4 * w) >= need;
                if (!fr) {
                    waitvm<0>();
                    lds_st(ctl, (unsigned)st);
                    unsigned got;
                    for (int w = 0; w < 5; ++w) ok = ok && wait_lds<12>(ctl + 4 + 4 * w, need, got, tmo, call);
                }
            }
            const unsigned slot = lds0 + (unsigned)(st & (NS - 1)) * STAGE;
            const unsigned char* ws_ = wsrc + (size_t)st * 256 * NLOC;
#pragma unroll 4
            for (int i = 0; i < 32; ++i) glds16(ws_ + (size_t)i * 8 * NLOC + (((i >> 1) & 1) ? c1 : c0), slot + i * 1024);
            waitvm<32>();
            lds_st(ctl, (unsigned)st);
        }
        waitvm<0>();
        lds_st(ctl, (unsigned)NGRP);
        asm volatile("s_waitcnt lgkmcnt(0)" ::: "memory");
        return;
    }

    const bool local = wave == 5;
    const unsigned call_v = ld_agent(ctr);
    const float scale = p.sx[0] * p.sw[0];
    __builtin_amdgcn_s_barrier();
    const unsigned call = uni(call_v) + 1u;
    if (lane == 0) st_agent(ctr, call);
    if (local && !has_local) return;

    const int src = local ? p.rank : ((p.rank + srcsel) & (NR - 1));
    const int rb = local ? (srcsel - 1) * 8 + rt_ : rt_ * 4 + wave;
    const unsigned char* xbase = local ? p.x : me + (size_t)(call & 1u) * DATA_PAR + (size_t)src * SLOT_BYTES;
    const __amdgpu_buffer_rsrc_t xr = __builtin_amdgcn_make_buffer_rsrc((void*)xbase, 0, (int)(local ? XBYTES : SLOT_BYTES), 0x00020000);
    const int voff = local ? (rb * 16 + r) * KDIM + 16 * g : rb * (int)RB_BYTES + lane * 16;
    const int kstride = local ? 128 : 2048, hstride = local ? 64 : 1024;
    const u64* flag = (const u64*)(me + FLAG_OFF + (size_t)(call & 1u) * FLAG_PAR + (size_t)(src * 32 + rb) * 128);
    const unsigned fbase = call * 64u;
    const unsigned done_addr = ctl + 4 + 4 * (local ? 4 : wave);
    const unsigned scr = lds0 + LDS_SCR + (local ? 4 : wave) * SCR_BYTES;

    const unsigned q = (unsigned)(lane & 15) >> 1, pp = lane & 1;
    const unsigned swz = ((q >> 1) & 3u) | ((unsigned)(g & 1) << 2);
    const unsigned abase = lds0 + (16u * g + q) * 128u + 8u * pp + (swz << 4);

    v4f acc[8];
#pragma unroll
    for (int c = 0; c < 8; ++c) acc[c] = v4f{0.f, 0.f, 0.f, 0.f};

    bool ok = true;
    unsigned got;
    auto load_w1 = [&](unsigned sb, int c) -> v8i {
        LAS unsigned char* a = (LAS unsigned char*)(size_t)(sb ^ (c << 4));
        v2i w0 = __builtin_amdgcn_ds_read_tr8_b64_v2i32((LAS v2i*)(a));
        v2i w1 = __builtin_amdgcn_ds_read_tr8_b64_v2i32((LAS v2i*)(a + 1024));
        v2i w2 = __builtin_amdgcn_ds_read_tr8_b64_v2i32((LAS v2i*)(a + 8192));
        v2i w3 = __builtin_amdgcn_ds_read_tr8_b64_v2i32((LAS v2i*)(a + 8192 + 1024));
        return v8i{w0[0], w0[1], w1[0], w1[1], w2[0], w2[1], w3[0], w3[1]};
    };
    v8i wf[4];
    ok = wait_lds(ctl, 1u, got, tmo, call);
#pragma unroll
    for (int c = 0; c < 4; ++c) wf[c] = load_w1(abase, c);

    unsigned flo = local ? ~0u : 0u, fhi = local ? ~0u : 0u;
    auto load_x = [&](u32x4* xv, int kg, bool have) {
        const int vo = have ? voff : 0x7ff00000;
        const int so = local ? kg * 256 : (int)gpos(kg, rb);
#pragma unroll
        for (int i = 0; i < 4; ++i)
            xv[i] = __builtin_amdgcn_raw_buffer_load_b128(xr, vo, so + (i >> 1) * kstride + (i & 1) * hstride, 17);
    };
    auto decode = [&](const u32x4* xv, bool raw, v8i& x0, v8i& x1) {
        if (raw) {
            x0 = v8i{(int)xv[0][0], (int)xv[0][1], (int)xv[0][2], (int)xv[0][3], (int)xv[1][0], (int)xv[1][1], (int)xv[1][2], (int)xv[1][3]};
            x1 = v8i{(int)xv[2][0], (int)xv[2][1], (int)xv[2][2], (int)xv[2][3], (int)xv[3][0], (int)xv[3][1], (int)xv[3][2], (int)xv[3][3]};
        } else {
            unpack_group(xv, scr, lane, x0, x1);
        }
    };
    auto mma = [&](const v8i x0, const v8i x1, int kg) {
        const unsigned sb = abase + (unsigned)(kg & (NS - 1)) * STAGE;
#pragma unroll
        for (int c = 0; c < 8; ++c) {
            const v8i cur = wf[c & 3];
            acc[c] = __builtin_amdgcn_mfma_scale_f32_16x16x128_f8f6f4(cur, x0, acc[c], 0, 0, 0, 0, 0, 0);
            wf[c & 3] = c < 4 ? load_w1(sb, c + 4) : load_w1(sb + 16384, c - 4);
            __builtin_amdgcn_sched_group_barrier(0x008, 1, 0);
            __builtin_amdgcn_sched_group_barrier(0x100, 4, 0);
        }
#pragma unroll
        for (int c = 0; c < 4; ++c) {
            const v8i cur = wf[c];
            acc[c] = __builtin_amdgcn_mfma_scale_f32_16x16x128_f8f6f4(cur, x1, acc[c], 0, 0, 0, 0, 0, 0);
            wf[c] = load_w1(sb + 16384, c + 4);
            __builtin_amdgcn_sched_group_barrier(0x008, 1, 1);
            __builtin_amdgcn_sched_group_barrier(0x100, 4, 1);
        }
        lds_st(done_addr, (unsigned)(kg + 1));
        const int kn = kg + 1 < NGRP ? kg + 1 : kg;
        if (kg + 1 < NGRP) ok = ok && wait_lds<4>(ctl, (unsigned)(kn + 1), got, tmo, call);
        const unsigned sn = abase + (unsigned)(kn & (NS - 1)) * STAGE;
#pragma unroll
        for (int c = 4; c < 8; ++c) {
            const v8i cur = wf[c & 3];
            acc[c] = __builtin_amdgcn_mfma_scale_f32_16x16x128_f8f6f4(cur, x1, acc[c], 0, 0, 0, 0, 0, 0);
            wf[c & 3] = load_w1(sn, c - 4);
            __builtin_amdgcn_sched_group_barrier(0x008, 1, 2);
            __builtin_amdgcn_sched_group_barrier(0x100, 4, 2);
        }
    };
    auto compute = [&](const u32x4* xv, int kg, bool raw) {
        v8i x0, x1;
        decode(xv, raw, x0, x1);
        mma(x0, x1, kg);
    };
    auto await_direct = [&](unsigned need) {
        const u64 t0 = rt();
        for (unsigned spins = 1;; ++spins) {
            const u64 v = ld_sys64(flag);
            flo = uni((unsigned)v); fhi = uni((unsigned)(v >> 32));
            if (flo >= need) return;
            __builtin_amdgcn_s_sleep(1);
            if ((spins & 15u) == 0u) {
                if (uni(ld_agent(tmo)) == call || rt() - t0 > DEADLINE) { st_agent(tmo, call); ok = false; return; }
            }
        }
    };
    auto step = [&](const u32x4* cur, u32x4* nxt, int kg) {
        const unsigned need = fbase + (unsigned)kg + 2u;
        const bool raw = ((fhi >> kg) & 1u) || !ok;
        const bool early = ok && flo >= need;
        load_x(nxt, kg + 1, early);
        v8i x0, x1;
        decode(cur, raw, x0, x1);
        const u64 fv = ld_sys64(flag);
        mma(x0, x1, kg);
        if (!early) {
            if (!local) {
                flo = uni((unsigned)fv); fhi = uni((unsigned)(fv >> 32));
                if (flo < need && ok) await_direct(need);
            }
            asm volatile("" ::: "memory");
            load_x(nxt, kg + 1, ok);
        } else if (!local) {
            const unsigned l2 = uni((unsigned)fv);
            if (l2 > flo) { flo = l2; fhi = uni((unsigned)(fv >> 32)); }
        }
    };
    auto settle = [&](u32x4* xv, int kg) {
        if (local || !ok) return;
        const u64 t0 = rt();
        for (unsigned spins = 1;; ++spins) {
            unsigned m = ~0u;
#pragma unroll
            for (int i = 0; i < 4; ++i) m = min(min(m, min(xv[i][0], xv[i][1])), min(xv[i][2], xv[i][3]));
            if (!__any(m == 0u)) {
#pragma unroll
                for (int i = 0; i < 4; ++i) xv[i] ^= SMASK;
                return;
            }
            load_x(xv, kg, true);
            const u64 fv = ld_sys64(flag);
            const unsigned lo = uni((unsigned)fv), hi = uni((unsigned)(fv >> 32));
            if (lo >= fbase + (unsigned)kg + 1u && ((hi >> (16 + kg - NFLAG)) & 1u)) {
                asm volatile("" ::: "memory");
                const __amdgpu_buffer_rsrc_t fr = __builtin_amdgcn_make_buffer_rsrc(
                    (void*)(me + FB_OFF + (size_t)(call & 1u) * FB_PAR + (size_t)(src * 32 + rb) * 8192 + (kg - NFLAG) * 4096), 0, 4096, 0x00020000);
#pragma unroll
                for (int i = 0; i < 4; ++i) xv[i] = __builtin_amdgcn_raw_buffer_load_b128(fr, lane * 16, i * 1024, 17);
                return;
            }
            if ((spins & 7u) == 0u) {
                if (uni(ld_agent(tmo)) == call || rt() - t0 > DEADLINE) { st_agent(tmo, call); ok = false; return; }
            }
        }
    };
    u32x4 xa[4], xb[4];
    if (!local) await_direct(fbase + 1u);
    asm volatile("" ::: "memory");
    load_x(xa, 0, ok);
    for (int kg = 0; kg < NFLAG - 2; kg += 2) {
        step(xa, xb, kg);
        step(xb, xa, kg + 1);
    }
    step(xa, xb, NFLAG - 2);
    load_x(xa, NFLAG, ok);
    compute(xb, NFLAG - 1, ((fhi >> (NFLAG - 1)) & 1u) || !ok);
    settle(xa, NFLAG);
    load_x(xb, NFLAG + 1, ok);
    compute(xa, NFLAG, true);
    const unsigned sl = abase + (unsigned)((NFLAG + 1) & (NS - 1)) * STAGE;
    v8i wl[4];
#pragma unroll
    for (int c = 0; c < 4; ++c) wl[c] = load_w1(sl, c + 4);
    const __amdgpu_buffer_rsrc_t orr = __builtin_amdgcn_make_buffer_rsrc((void*)p.out, 0, NR * ROWS * NLOC * 4, 0x00020000);
    const int oa = ((src * ROWS + rb * 16 + r) * NLOC + ct * 128 + 4 * g) * 4;
    settle(xb, NFLAG + 1);
    const unsigned tv = ld_agent(tmo);
    {
        const v8i x0 = {(int)xb[0][0], (int)xb[0][1], (int)xb[0][2], (int)xb[0][3], (int)xb[1][0], (int)xb[1][1], (int)xb[1][2], (int)xb[1][3]};
        const v8i x1 = {(int)xb[2][0], (int)xb[2][1], (int)xb[2][2], (int)xb[2][3], (int)xb[3][0], (int)xb[3][1], (int)xb[3][2], (int)xb[3][3]};
        v8i w2[8];
#pragma unroll
        for (int c = 0; c < 8; ++c) {
            w2[c] = load_w1(sl + 16384, c);
            acc[c] = __builtin_amdgcn_mfma_scale_f32_16x16x128_f8f6f4(c < 4 ? wf[c] : wl[c - 4], x0, acc[c], 0, 0, 0, 0, 0, 0);
            __builtin_amdgcn_sched_group_barrier(0x100, 4, 3);
            __builtin_amdgcn_sched_group_barrier(0x008, 1, 3);
        }
#pragma unroll
        for (int c = 0; c < 8; ++c)
            acc[c] = __builtin_amdgcn_mfma_scale_f32_16x16x128_f8f6f4(w2[c], x1, acc[c], 0, 0, 0, 0, 0, 0);
    }
    __builtin_amdgcn_sched_barrier(0);
    const float sc = uni(tv) == call ? __builtin_nanf("") : scale;
#pragma unroll
    for (int c = 0; c < 8; ++c) {
        const v4f y = __builtin_elementwise_maximum(acc[c] * sc, v4f{0.f, 0.f, 0.f, 0.f});
        __builtin_amdgcn_raw_buffer_store_b128(__builtin_bit_cast(u32x4, y), orr, oa + 64 * c, 0, 17);
    }
}

}

extern "C" void kernel_launch(int rank, int nranks, ncclComm_t comm, hipStream_t stream,
                              void* const* d_in, const int* in_sizes, int n_in,
                              void* d_out, int out_size,
                              const int* in_dtypes, int out_dtype,
                              void* const* d_out_all,
                              void* const* d_ws_all, size_t ws_bytes) {
    (void)comm; (void)in_dtypes; (void)out_dtype; (void)d_out_all;
    if (nranks != NR || (unsigned)rank >= (unsigned)NR || n_in != 4 || ws_bytes < WS_NEED ||
        in_sizes[0] != ROWS * KDIM || in_sizes[1] != KDIM * NLOC || out_size != NR * ROWS * NLOC) {
        fprintf(stderr, "kernel_launch: unexpected problem shape\n");
        return;
    }
    static bool init = false;
    if (!init) {
        HIP_CHECK(hipFuncSetAttribute((const void*)ag_gemm, hipFuncAttributeMaxDynamicSharedMemorySize, LDS_BYTES));
        init = true;
    }
    Params p;
    p.x = (const unsigned char*)d_in[0];
    p.w = (const unsigned char*)d_in[1];
    p.sx = (const float*)d_in[2];
    p.sw = (const float*)d_in[3];
    p.out = (float*)d_out;
    p.me = (unsigned char*)d_ws_all[rank];
    for (int d = 0; d < NR - 1; ++d) p.peer[d] = (unsigned char*)d_ws_all[(rank + 1 + d) % NR];
    p.rank = rank;
    p.pad = 0;
    hipLaunchKernelGGL(ag_gemm, dim3(256), dim3(384), LDS_BYTES, stream, p);
    HIP_CHECK(hipGetLastError());
}
```
